```python
import numpy as np

kernel_name = "gpu_gemm_rs_m4096_k4096_n4096_bf16_reluq8"


ABSMAX_RTOL = 0.02

COMPUTE_DTYPE = "bfloat16"
EPILOGUE = "relu_quant_int8"
IN_QUANT = None
OUT_QUANT = "int8"
IN_DTYPES = {"x": "bfloat16", "w_mat": "bfloat16"}
WORLD_SIZE = 8

def _to_bf16(x):
    u = np.ascontiguousarray(x, dtype=np.float32).view(np.uint32)
    return ((u + 0x7FFF + ((u >> 16) & 1)) & 0xFFFF0000).view(np.float32)

def setup_inputs():

    x = _to_bf16(np.random.randn(4096, 4096).astype(np.float32))
    w_mat = _to_bf16((np.random.randn(4096, 4096) / np.sqrt(4096)).astype(np.float32))
    return {"x": x, "w_mat": w_mat}

def _quant_dequant(y, amax):
    scale = amax / 127.0
    q = np.clip(np.round(y / scale), -127.0, 127.0)
    return (q * scale).astype(np.float32)
def _epilogue(y):
    y = np.maximum(y, 0.0)
    return _quant_dequant(y, np.abs(y).max())

def reference(x, w_mat):
    y = (x @ w_mat).astype(np.float32)
    return _epilogue(y)
```

```cpp
#include <hip/hip_runtime.h>
#include <rccl/rccl.h>
#include <cstdio>
#include <cstring>

typedef unsigned int u32;
typedef unsigned long long u64;
typedef __attribute__((ext_vector_type(4))) unsigned int u32x4;
typedef __attribute__((ext_vector_type(2))) unsigned int u32x2;
typedef __attribute__((ext_vector_type(4))) short s16x4;
typedef __attribute__((ext_vector_type(8))) short s16x8;
typedef __attribute__((ext_vector_type(8))) __bf16 bf16x8;
typedef __attribute__((ext_vector_type(2))) __bf16 bf16x2;
typedef __attribute__((ext_vector_type(2))) float f32x2;
typedef __attribute__((ext_vector_type(16))) float f32x16;
#define LDSP __attribute__((address_space(3)))
#define GLBP __attribute__((address_space(1)))
#define CSTP __attribute__((address_space(4)))
typedef __amdgpu_buffer_rsrc_t rsrc_t;

namespace {

constexpr int NR = 8;
constexpr u32 X_BYTES = 4096u * 512u * 2u;
constexpr u32 W_BYTES = 512u * 4096u * 2u;
constexpr u32 OUT_BYTES = 512u * 4096u;
constexpr u32 CH = 65536;
constexpr u32 YT = 32768;
constexpr u32 F_CALL = 0;
constexpr u32 F_TMO = 16384;
constexpr u32 F_TICKET = F_TMO + 1024;
constexpr u32 F_YRDY = F_TICKET + 1024;
constexpr u32 F_AMAX = 20480;
constexpr u32 F_LINK = 32768;
constexpr u32 F_DBG = F_LINK + 8 * 64 * 128;
constexpr u32 OFF_XS = 1u << 20;
constexpr u32 OFF_WS = OFF_XS + 6 * 16 * CH;
constexpr u32 OFF_YOUT = OFF_WS + 6 * 16 * CH;
constexpr u32 OFF_YX = OFF_YOUT + 4 * 32 * YT;
constexpr u32 OFF_YT = OFF_YX + 4 * 32 * YT;
constexpr u32 WS_USED = OFF_YT + 64 * YT;
constexpr u32 OOB = 0x80000000u;

constexpr int NSLOT = 4;
constexpr u32 STAGE = 32768;
constexpr u32 L_CTRL = NSLOT * STAGE;
constexpr u32 LDS_BYTES = L_CTRL + 1024;
constexpr u64 DEADLINE = 50000000ull;
constexpr int NPUSH = 6;
constexpr int NGEMM = 32 - NPUSH;
#define WIN_T 3

struct Params {
  const char* x;
  const char* w;
  char* out;
  char* ws[NR];
  u32 rank;
  u32 seqw[8];
  u32 posw[8];
  u32 yw[4][2][4];
};

__device__ __forceinline__ rsrc_t mk_rsrc(const void* p, u32 bytes) {
  return __builtin_amdgcn_make_buffer_rsrc((void*)p, 0, bytes, 0x00020000);
}
__device__ __forceinline__ u32 rfl(u32 v) { return __builtin_amdgcn_readfirstlane(v); }
__device__ __forceinline__ u32 ld_sys(const char* p) { return __hip_atomic_load((const GLBP u32*)p, __ATOMIC_RELAXED, __HIP_MEMORY_SCOPE_SYSTEM); }
__device__ __forceinline__ u32 ld_agt(const char* p) { return __hip_atomic_load((const GLBP u32*)p, __ATOMIC_RELAXED, __HIP_MEMORY_SCOPE_AGENT); }
__device__ __forceinline__ void st_sys(char* p, u32 v) { __hip_atomic_store((GLBP u32*)p, v, __ATOMIC_RELAXED, __HIP_MEMORY_SCOPE_SYSTEM); }
__device__ __forceinline__ void st_agt(char* p, u32 v) { __hip_atomic_store((GLBP u32*)p, v, __ATOMIC_RELAXED, __HIP_MEMORY_SCOPE_AGENT); }
#define WAIT_VM(n) asm volatile("s_waitcnt vmcnt(" #n ")" ::: "memory")
#define WAIT_LGKM0() asm volatile("s_waitcnt lgkmcnt(0)" ::: "memory")
#define BAR() __builtin_amdgcn_s_barrier()
#define SCHED() __builtin_amdgcn_sched_barrier(0)

__device__ __forceinline__ u32 row_base(u32 q, u32 r) {
  if (r < 4) return 512u * (q ^ 1u) + 128u * r;
  if (r < 12) return 512u * (q ^ (4u + (r & 3u))) + 256u * ((q >> 1) & 1u) + 128u * ((r - 4u) >> 2);
  return 512u * q + 128u * (r - 12u);
}
__device__ __forceinline__ u32 k_half(u32 q, u32 p) { return (q ^ ((q >> 1) & p)) & 1u; }

__device__ __forceinline__ void wave_wait(const char* addr, bool want, u32 call, char* tmo) {
  const u64 t0 = __builtin_amdgcn_s_memrealtime();
  for (;;) {
    const u32 v = want ? ld_sys(addr) : call;
    if (__all((int)(v - call) >= 0)) break;
    if (rfl(ld_agt(tmo)) == call) break;
    if (__builtin_amdgcn_s_memrealtime() - t0 > DEADLINE) { st_agt(tmo, call); break; }
    __builtin_amdgcn_s_sleep(4);
  }
  asm volatile("" ::: "memory");
}

#define STAMP(i) do {} while (0)

}

__global__ void __launch_bounds__(512, 2) fused_kernel(Params params_by_value) {
  (void)params_by_value;
  const CSTP Params* const K = (const CSTP Params*)__builtin_amdgcn_kernarg_segment_ptr();
  const char* const gx = K->x;
  const char* const gw = K->w;
  extern __shared__ __attribute__((aligned(16))) char smem[];
  LDSP char* lds = (LDSP char*)smem;
  volatile LDSP u32* ctrl = (volatile LDSP u32*)(lds + L_CTRL);
  const u32 tid = threadIdx.x, lane = tid & 63u, wid = rfl(tid >> 6);
  const u32 q = K->rank & 7u;
  char* const me = K->ws[q];
  char* const tmo = me + F_TMO;
  const u32 grp = blockIdx.x & 7u, li = blockIdx.x >> 3;
  char* const my_ctr = me + F_CALL + blockIdx.x * 64u;

  if (li < (u32)NPUSH) {
    const u32 callv = ld_agt(my_ctr);
    if (tid < 64) ctrl[tid] = 0;
    WAIT_LGKM0();
    BAR();
    if (wid == 0) STAMP(0);
    const u32 d = li + 2u;
    const u32 dst_rank = (q ^ d) & 7u;
    char* const peer = K->ws[dst_rank];
    const rsrc_t r_dst = mk_rsrc(peer, WS_USED);
    const u32 h = k_half(dst_rank, q);
    const u32 u = grp * 8u + wid, kb = u >> 4, pu = u & 15u;
    const u32 vx = (pu * 8u + (lane >> 3)) * 1024u + h * 512u + kb * 128u + (((lane & 7u) ^ (((pu & 1u) * 4u + (lane >> 4)) & 7u)) << 4);
    const u32 vw = (h * 256u + kb * 64u + pu * 4u + ((lane >> 2) & 3u)) * 8192u + (32u * (lane >> 4) + 8u * (lane & 3u)) * 2u;
    const u32 vin = u * 1024u + lane * 16u;
    const u32 vy = ((grp & 3u) * 8u + wid) * 1024u + lane * 16u;
    const rsrc_t r_me = mk_rsrc(me, WS_USED);
    u32 code;
    {
      const u32 wsel = lane >> 2;
      u32 word = 0;
#pragma unroll
      for (u32 i = 0; i < 8; ++i) { const u32 t = rfl(K->seqw[i]); word = wsel == i ? t : word; }
#pragma unroll
      for (u32 i = 0; i < 4; ++i) { const u32 t = rfl(K->yw[(d - 4u) & 3u][grp >> 2][i]); word = wsel == 8u + i ? t : word; }
      code = (word >> ((lane & 3u) * 8u)) & 31u;
    }
    u32 d_src, d_dst, d_inf, d_gate = OOB;
    bool valid;
    if (lane < 32u) {
      if (code < 16u) { d_src = row_base(dst_rank, code) * 1024u; d_dst = OFF_XS + ((d - 2u) * 16u + code) * CH; d_inf = 0u | lane << 2; }
      else { d_src = ((dst_rank >> 2) * 2048u + (code - 16u) * 128u) * 2u; d_dst = OFF_WS + ((d - 2u) * 16u + (code - 16u)) * CH; d_inf = 1u | lane << 2; }
      valid = !(d >= 4u && code >= 4u && code < 12u && (code & 3u) == d - 4u);
    } else {
      const u32 ti = ((d - 4u) & 3u) * 32u + code;
      d_src = OFF_YOUT + ti * YT; d_dst = OFF_YX + ti * YT; d_inf = 2u | (32u + code) << 2;
      valid = d >= 4u && lane < 48u;
      if (valid) d_gate = F_YRDY + ti * 4u;
    }
    u64 todo = __ballot(valid);
    const u64 free_mask = __ballot(d_gate == OOB);
    u64 ready = free_mask;
    const u64 t_start = __builtin_amdgcn_s_memrealtime();
    u32 call = 0, pollv = 0;
    int prev = -1;
    int id0 = -1, id1 = -1, id2 = -1;
    u32 do0 = OOB, do1 = OOB, do2 = OOB;
    u32x4 b0 = {}, b1 = {}, b2 = {};

    auto fetch = [&](u32x4& bl, int& idl, u32& dol) {
      const u64 avail = todo & ready;
      const u64 ax_ = (u64)gx, aw_ = (u64)gw, am_ = (u64)me;
      const u32 vx_ = vx, vw_ = vw, vy_ = vy;
      u64 src = am_; u32 sbytes = WS_USED, soff = OOB, kind = 2u;
      idl = -1; dol = OOB;
      if (avail) {
        const int it = __builtin_ctzll(avail);
        todo &= ~(1ull << it);
        const u32 inf = __builtin_amdgcn_readlane(d_inf, it);
        soff = __builtin_amdgcn_readlane(d_src, it);
        dol = __builtin_amdgcn_readlane(d_dst, it);
        kind = inf & 3u; idl = (int)(inf >> 2);
        src = kind == 0u ? ax_ : kind == 1u ? aw_ : am_;
        sbytes = kind == 2u ? WS_USED : X_BYTES;
      }
      bl = __builtin_amdgcn_raw_buffer_load_b128(mk_rsrc((const void*)src, sbytes), kind == 0u ? vx_ : kind == 1u ? vw_ : vy_, soff, 16);
    };
    auto step = [&](u32x4& bl, int& idl, u32& dol, u32x4& bs, int& ids, u32& dos) {
      WAIT_VM(1);
      ready |= __ballot((int)(pollv - call) >= 0);
      const u32 vy_ = vy, vin_ = vin, g_ = grp;
      __builtin_amdgcn_raw_buffer_store_b128(bs, r_dst, (ids >= 32 ? vy_ : vin_) + dos, 0, 17);
      if (prev >= 0) {
        u32 old = 0;
        if (lane == 0) old = __hip_atomic_fetch_add((LDSP u32*)(lds + L_CTRL) + prev, 1u, __ATOMIC_RELAXED, __HIP_MEMORY_SCOPE_WORKGROUP);
        if (rfl(old) == 7u && lane == 0)
          st_sys(peer + F_LINK + (d * 64u + (u32)prev) * 128u + (prev >= 32 ? (g_ & 3u) : g_) * 4u, call);
      }
      prev = ids;
      ids = -1;
      if (todo && !(todo & ready) && __builtin_amdgcn_s_memrealtime() - t_start > DEADLINE) { st_agt(tmo, call); ready = ~0ull; }
      const bool need_poll = (todo & ~ready) && __builtin_popcountll(todo & ready) <= 4;
      pollv = __builtin_amdgcn_raw_buffer_load_b32(r_me, d_gate, need_poll ? 0u : 0x40000000u, 17);
      fetch(bl, idl, dol);
    };
    fetch(b1, id1, do1);
    fetch(b2, id2, do2);
    call = rfl(callv) + 1u;
    pollv = call - 1u;
    for (;;) {
      step(b0, id0, do0, b1, id1, do1);
      step(b1, id1, do1, b2, id2, do2);
      step(b2, id2, do2, b0, id0, do0);
      if (!todo && (id0 & id1 & id2 & prev) < 0) break;
    }
    if (wid == 0) STAMP(1);
    BAR();
    if (tid == 0) st_agt(my_ctr, call);
    return;
  }

  const u32 kidx = li - NPUSH;
  const u32 ga = grp >> 1, gb = grp & 1u;
  const bool loader = wid >= 4u;
  const u32 lw = wid & 3u;
  u32 call = 0;
  if (wid == 4u) {
    __builtin_amdgcn_fence(__ATOMIC_ACQUIRE, "");
    call = rfl(ld_agt(my_ctr)) + 1u;
    STAMP(0);
  }
  const rsrc_t r_me = mk_rsrc(me, WS_USED);

  u32 kind, rt, ct;
  u32 fin = 0;
  bool have = kidx < 24u;
  if (have) { rt = ga + 4u * (kidx >> 3); ct = 2u * (kidx & 7u) + gb; kind = rt < 4u ? 0u : 1u; }
  else { rt = 0; ct = 0; kind = 0; }

  for (int round = 0; round < 2; ++round) {
    if (round == 1) {
      if (wid == 4u) {
        u32 t = 0;
        if (lane == 0) t = __hip_atomic_fetch_add((GLBP u32*)(me + F_TICKET + grp * 128u), 1u, __ATOMIC_RELAXED, __HIP_MEMORY_SCOPE_AGENT);
        t = rfl(t) - (call - 1u) * (u32)NGEMM;
        if (lane == 0) { ctrl[0] = t; st_agt(my_ctr, call); }
        WAIT_LGKM0();
      }
      BAR();
      const u32 t = rfl(ctrl[0]);
      if (t >= 16u) return;
      have = true; fin = grp * 16u + t;
      kind = t < 8u ? 2u : 3u; rt = 12u + ga; ct = 2u * (t & 7u) + gb;
    }
    if (!have) continue;

    const u32 grow = row_base(q, rt);
    const u32 gcol = ((q >> 2) ^ (kind == 3u ? 1u : 0u)) * 2048u + ct * 128u;
    const u32 skip = kind == 1u ? 2u + (rt & 3u) : 99u;
    const u32 nrem = kind == 3u ? 0u : (kind == 1u ? 20u : 24u);

    if (loader) {
      const rsrc_t r_x = mk_rsrc(gx, X_BYTES), r_w = mk_rsrc(gw, W_BYTES);
      const u32 vx = (lw * 8u + (lane >> 3)) * 1024u + (((lane & 7u) ^ (((lw & 1u) * 4u + (lane >> 4)) & 7u)) << 4);
      const u32 vw = (lw * 4u + ((lane >> 2) & 3u)) * 8192u + (32u * (lane >> 4) + 8u * (lane & 3u)) * 2u;
      const u32 vs = lw * 1024u + lane * 16u;
      auto issue_own = [&](u32 s) {
        LDSP char* dst = lds + (s & 3u) * STAGE + lw * 1024u;
        const u32 sx = grow * 1024u + s * 128u, sw = s * 64u * 8192u + gcol * 2u;
#pragma unroll
        for (u32 i = 0; i < 4; ++i) {
          __builtin_amdgcn_raw_ptr_buffer_load_lds(r_x, dst + i * 4096u, 16, vx, sx + i * 32768u, 0, 0);
          __builtin_amdgcn_raw_ptr_buffer_load_lds(r_w, dst + 16384u + i * 4096u, 16, vw, sw + i * 131072u, 0, 0);
        }
      };
      auto issue_rem = [&](u32 s) {
        LDSP char* dst = lds + (s & 3u) * STAGE + lw * 1024u;
        u32 sg = s >> 2; if (sg >= skip) ++sg;
        const u32 sx = OFF_XS + (sg * 16u + rt) * CH + (s & 3u) * 16384u, sw = OFF_WS + (sg * 16u + ct) * CH + (s & 3u) * 16384u;
#pragma unroll
        for (u32 i = 0; i < 4; ++i) {
          __builtin_amdgcn_raw_ptr_buffer_load_lds(r_me, dst + i * 4096u, 16, vs, sx + i * 4096u, 0, 0);
          __builtin_amdgcn_raw_ptr_buffer_load_lds(r_me, dst + 16384u + i * 4096u, 16, vs, sw + i * 4096u, 0, 0);
        }
      };
#pragma unroll 1
      for (u32 s = 0; s < 4; ++s) issue_own(s);
      WAIT_VM(24); BAR();
#pragma unroll 1
      for (u32 s = 0; s < 4; ++s) { WAIT_VM(16); BAR(); issue_own(s + 4u); }
      WAIT_VM(16); BAR(); WAIT_VM(8); BAR(); WAIT_VM(0); BAR(); BAR();
      if (nrem) {
        if (wid == 4u) {
          STAMP(round * 4 + 1);
          const u32 a = (K->posw[rt >> 2] >> ((rt & 3u) * 8u)) & 31u, b = (K->posw[4u + (ct >> 2)] >> ((ct & 3u) * 8u)) & 31u;
          const u32 pos = a > b ? a : b;
          const u32 sg = lane >> 3;
          wave_wait(me + F_LINK + ((sg + 2u) * 64u + pos) * 128u + (lane & 7u) * 4u, sg < 6u && sg != skip, call, tmo);
          STAMP(round * 4 + 2);
        }
        BAR();
#pragma unroll 1
        for (u32 s = 0; s < 4; ++s) issue_rem(s);
        WAIT_VM(24); BAR();
#pragma unroll 1
        for (u32 s = 0; s + 4u < nrem; ++s) { WAIT_VM(16); BAR(); issue_rem(s + 4u); }
        WAIT_VM(16); BAR(); WAIT_VM(8); BAR(); WAIT_VM(0); BAR(); BAR();
      }
      if (kind < 2u) {
        BAR();
        if (wid == 4u) {
          if (lane == 0) {
            if (kind == 0u) st_sys(K->ws[q ^ 1u] + F_LINK + (64u + rt * 16u + ct) * 128u, call);
            else st_agt(me + F_YRDY + (rt & 3u) * 128u + (((rt - 4u) >> 2) * 16u + ct) * 4u, call);
          }
          STAMP(3);
        }
      } else {
        const u32 nsrc = kind - 1u;
        for (u32 i = 0; i < nsrc; ++i) {
          if (wid == 4u) {
            if (kind == 2u) wave_wait(me + F_LINK + (64u + ga * 16u + ct) * 128u, lane == 0, call, tmo);
            else {
              const u32 dd = (q ^ (((q >> 2) ^ 1u) * 4u + (ga >> 1) * 2u + i)) & 7u;
              wave_wait(me + F_LINK + (dd * 64u + 32u + (ga & 1u) * 16u + ct) * 128u + (lane & 3u) * 4u, lane < 4u, call, tmo);
            }
          }
          BAR();
        }
        if (wid == 4u) { STAMP(7); if (lane == 0) ctrl[16] = call; WAIT_LGKM0(); }
        BAR();
        const u32 callw = rfl(ctrl[16]);
        if (wid == 4u) {
          u32 m = ctrl[4 + (lane & 3u)];
          m = max(m, (u32)__shfl_xor((int)m, 1)); m = max(m, (u32)__shfl_xor((int)m, 2));
          if (lane < 8u) __hip_atomic_store((GLBP u64*)(K->ws[lane] + F_AMAX + (q * 128u + fin) * 8u), ((u64)callw << 32) | m, __ATOMIC_RELAXED, __HIP_MEMORY_SCOPE_SYSTEM);
          STAMP(8);
        }
        u32 m = 0;
        {
          const u64 t0 = __builtin_amdgcn_s_memrealtime();
          const u32 off = F_AMAX + (lw * 64u + lane) * 32u;
          for (;;) {
            const u32x4 g0 = __builtin_amdgcn_raw_buffer_load_b128(r_me, off, 0, 17);
            const u32x4 g1 = __builtin_amdgcn_raw_buffer_load_b128(r_me, off + 16u, 0, 17);
            const bool ok = g0.y == callw && g0.w == callw && g1.y == callw && g1.w == callw;
            m = max(max(g0.x, g0.z), max(g1.x, g1.z));
            if (__all(ok)) break;
            if (rfl(ld_agt(tmo)) == callw) break;
            if (__builtin_amdgcn_s_memrealtime() - t0 > DEADLINE) { st_agt(tmo, callw); break; }
            __builtin_amdgcn_s_sleep(2);
          }
        }
#pragma unroll
        for (int o = 32; o >= 1; o >>= 1) m = max(m, (u32)__shfl_xor((int)m, o));
        const u32 bad = rfl(ld_agt(tmo)) == callw ? 1u : 0u;
        if (lane == 0) { ctrl[8 + lw] = m; ctrl[12 + lw] = bad; }
        WAIT_LGKM0();
        BAR();
        if (wid == 4u) STAMP(9);
      }
    } else {
      const u32 wn = wid & 1u, wm = wid >> 1;
      const u32 hh = lane >> 5, r31 = lane & 31u;
      const u32 z = (lane >> 4) & 1u, tq = (lane >> 2) & 3u, tp = lane & 3u;
      const u32 aw = 16384u + hh * 2048u + wn * 512u + tq * 64u + (16u * (tp & 1u) + 8u * z + 4u * (tp >> 1)) * 2u;
      const u32 axb = (wm * 64u + r31) * 128u, sw7 = (r31 >> 1) & 7u;
      u32 ax[4];
#pragma unroll
      for (u32 s = 0; s < 4; ++s) ax[s] = axb + (((2u * s + hh) ^ sw7) << 4);
      f32x16 acc[2][2] = {};
      s16x8 fa[2][2], fb[2][2];
      auto rd = [&](int buf, u32 slot_off, int ks) {
#pragma unroll
        for (int f = 0; f < 2; ++f) {
          const s16x4 lo = __builtin_amdgcn_ds_read_tr16_b64_v4i16((LDSP s16x4*)(lds + slot_off + aw + ks * 4096 + f * 256));
          const s16x4 hi = __builtin_amdgcn_ds_read_tr16_b64_v4i16((LDSP s16x4*)(lds + slot_off + aw + ks * 4096 + f * 256 + 1024));
          fa[buf][f] = __builtin_shufflevector(lo, hi, 0, 1, 2, 3, 4, 5, 6, 7);
          fb[buf][f] = *(LDSP s16x8*)(lds + slot_off + ax[ks] + f * 4096);
        }
      };
      auto mm = [&](int buf) {
#pragma unroll
        for (int fn = 0; fn < 2; ++fn)
#pragma unroll
          for (int fm = 0; fm < 2; ++fm)
            acc[fn][fm] = __builtin_amdgcn_mfma_f32_32x32x16_bf16(__builtin_bit_cast(bf16x8, fa[buf][fn]), __builtin_bit_cast(bf16x8, fb[buf][fm]), acc[fn][fm], 0, 0, 0);
      };
#pragma unroll 1
      for (int batch = 0; batch < 2; ++batch) {
        const u32 nst = batch == 0 ? 8u : nrem;
        if (nst == 0) break;
        if (batch == 1) BAR();
        BAR();
        asm volatile("" ::: "memory");
        rd(0, 0, 0);
#pragma unroll 1
        for (u32 s = 0; s < nst; ++s) {
          const u32 so = (s & 3u) * STAGE, sn = ((s + 1u) & 3u) * STAGE;
          rd(1, so, 1); SCHED(); mm(0); SCHED();
          rd(0, so, 2); SCHED(); mm(1); SCHED();
          rd(1, so, 3); SCHED(); mm(0); SCHED();
          WAIT_LGKM0(); BAR(); asm volatile("" ::: "memory");
          rd(0, sn, 0);
          SCHED(); mm(1); SCHED();
        }
      }
      const u32 vy = wid * 8192u + lane * 16u;
      if (kind < 2u) {
        const rsrc_t r_o = mk_rsrc(K->ws[kind == 0u ? q ^ 1u : q], WS_USED);
        const u32 so = kind == 0u ? OFF_YT + (rt * 16u + ct) * YT : OFF_YOUT + ((rt & 3u) * 32u + ((rt - 4u) >> 2) * 16u + ct) * YT;
#pragma unroll
        for (int f = 0; f < 4; ++f)
#pragma unroll
          for (int hf = 0; hf < 2; ++hf) {
            u32x4 v;
#pragma unroll
            for (int e = 0; e < 4; ++e) {
              const f32x2 p = {acc[f >> 1][f & 1][hf * 8 + e * 2], acc[f >> 1][f & 1][hf * 8 + e * 2 + 1]};
              v[e] = __builtin_bit_cast(u32, __builtin_convertvector(p, bf16x2));
            }
            if (kind == 0u) asm volatile("s_waitcnt vmcnt(%0)" ::"i"(WIN_T - 1) : "memory");
            __builtin_amdgcn_raw_buffer_store_b128(v, r_o, vy + so + (f * 2 + hf) * 1024, 0, 17);
          }
        WAIT_VM(0);
        BAR();
      } else {
        const u32 nsrc = kind - 1u;
        for (u32 i = 0; i < nsrc; ++i) {
          BAR();
          asm volatile("" ::: "memory");
          const u32 dd = (q ^ (((q >> 2) ^ 1u) * 4u + (ga >> 1) * 2u + i)) & 3u;
          const u32 so = kind == 2u ? OFF_YT + (ga * 16u + ct) * YT : OFF_YX + (dd * 32u + (ga & 1u) * 16u + ct) * YT;
          u32x4 v[8];
#pragma unroll
          for (int j = 0; j < 8; ++j) v[j] = __builtin_amdgcn_raw_buffer_load_b128(r_me, vy + j * 1024, so, 0);
#pragma unroll
          for (int j = 0; j < 8; ++j)
#pragma unroll
            for (int e = 0; e < 4; ++e) {
              acc[j >> 2][(j >> 1) & 1][(j & 1) * 8 + e * 2] += __builtin_bit_cast(float, v[j][e] << 16);
              acc[j >> 2][(j >> 1) & 1][(j & 1) * 8 + e * 2 + 1] += __builtin_bit_cast(float, v[j][e] & 0xFFFF0000u);
            }
        }
        float mx = 0.f;
#pragma unroll
        for (int f = 0; f < 4; ++f)
#pragma unroll
          for (int e = 0; e < 16; ++e) mx = fmaxf(mx, acc[f >> 1][f & 1][e]);
        u32 m = __builtin_bit_cast(u32, mx);
#pragma unroll
        for (int o = 32; o >= 1; o >>= 1) m = max(m, (u32)__shfl_xor((int)m, o));
        if (lane == 0) ctrl[4 + wid] = m;
        WAIT_LGKM0();
        BAR();
        BAR();
        const u32 am = max(max(ctrl[8], ctrl[9]), max(ctrl[10], ctrl[11]));
        const u32 bad = ctrl[12] | ctrl[13] | ctrl[14] | ctrl[15];
        const float amax = __builtin_bit_cast(float, am);
        const float scale = amax / 127.0f;
        const float inv = amax > 0.f ? 1.0f / scale : 0.f;
        const rsrc_t r_out = mk_rsrc(K->out, OUT_BYTES);
        const u32 vo = (ga * 128u + wm * 64u + r31) * 4096u + gcol + wn * 64u + hh * 16u;
#pragma unroll
        for (int f = 0; f < 4; ++f) {
          u32x4 v;
#pragma unroll
          for (int e = 0; e < 4; ++e) {
            u32 pk = 0;
#pragma unroll
            for (int b = 0; b < 4; ++b) {
              const float y = fminf(__builtin_rintf(fmaxf(acc[f >> 1][f & 1][e * 4 + b], 0.f) * inv), 127.f);
              pk |= (u32)y << (8 * b);
            }
            v[e] = bad ? 0x80808080u : pk;
          }
          __builtin_amdgcn_raw_buffer_store_b128(v, r_out, vo + (f & 1) * 32 * 4096 + (f >> 1) * 32, 0, 0);
        }
      }
    }
  }
}

extern "C" void kernel_launch(int rank, int nranks, ncclComm_t comm, hipStream_t stream,
                              void* const* d_in, const int* in_sizes, int n_in,
                              void* d_out, int out_size, const int* in_dtypes, int out_dtype,
                              void* const* d_out_all, void* const* d_ws_all, size_t ws_bytes) {
  (void)comm; (void)in_dtypes; (void)out_dtype; (void)d_out_all;
  if (nranks != NR || (unsigned)rank >= (unsigned)NR || n_in < 2 || ws_bytes < WS_USED ||
      (size_t)in_sizes[0] * 2 != X_BYTES || (size_t)in_sizes[1] * 2 != W_BYTES || (u32)out_size != OUT_BYTES) {
    fprintf(stderr, "kernel_launch: unexpected problem shape\n");
    return;
  }
  Params P;
  P.x = (const char*)d_in[0];
  P.w = (const char*)d_in[1];
  P.out = (char*)d_out;
  for (int p = 0; p < NR; ++p) P.ws[p] = (char*)d_ws_all[p];
  P.rank = (u32)rank;
  unsigned char seq[32], pos[32], yo[4][2][16];
  int n = 0;
  for (int i = 0; i < 12; ++i) { seq[n++] = (unsigned char)i; seq[n++] = (unsigned char)(16 + i); }
  for (int i = 12; i < 16; ++i) seq[n++] = (unsigned char)(16 + i);
  for (int i = 12; i < 16; ++i) seq[n++] = (unsigned char)i;
  for (int i = 0; i < 32; ++i) pos[seq[i]] = (unsigned char)i;
  for (int l = 0; l < 4; ++l) {
    int m = 0;
    for (int ps = 0; ps < 32; ++ps)
      for (int t = 0; t < 32; ++t) {
        const int a = pos[4 + l + 4 * (t >> 4)], b = pos[16 + (t & 15)];
        if ((a > b ? a : b) == ps) { yo[l][m & 1][m >> 1] = (unsigned char)t; ++m; }
      }
  }
  memcpy(P.seqw, seq, 32); memcpy(P.posw, pos, 32); memcpy(P.yw, yo, 128);
  hipError_t e = hipFuncSetAttribute((const void*)fused_kernel, hipFuncAttributeMaxDynamicSharedMemorySize, (int)LDS_BYTES);
  if (e != hipSuccess) { fprintf(stderr, "hipFuncSetAttribute: %s\n", hipGetErrorString(e)); return; }
  fused_kernel<<<dim3(256), dim3(512), LDS_BYTES, stream>>>(P);
  e = hipGetLastError();
  if (e != hipSuccess) fprintf(stderr, "launch: %s\n", hipGetErrorString(e));
}
```

```cpp
#include <hip/hip_runtime.h>
#include <rccl/rccl.h>
#include <cstdio>
#include <cstring>

typedef unsigned int u32;
typedef unsigned long long u64;
typedef __attribute__((ext_vector_type(4))) unsigned int u32x4;
typedef __attribute__((ext_vector_type(2))) unsigned int u32x2;
typedef __attribute__((ext_vector_type(4))) short s16x4;
typedef __attribute__((ext_vector_type(8))) short s16x8;
typedef __attribute__((ext_vector_type(8))) __bf16 bf16x8;
typedef __attribute__((ext_vector_type(2))) __bf16 bf16x2;
typedef __attribute__((ext_vector_type(2))) float f32x2;
typedef __attribute__((ext_vector_type(16))) float f32x16;
#define LDSP __attribute__((address_space(3)))
#define GLBP __attribute__((address_space(1)))
#define CSTP __attribute__((address_space(4)))
typedef __amdgpu_buffer_rsrc_t rsrc_t;

namespace {

constexpr int NR = 8;
constexpr u32 X_BYTES = 4096u * 512u * 2u;
constexpr u32 W_BYTES = 512u * 4096u * 2u;
constexpr u32 OUT_BYTES = 512u * 4096u;
constexpr u32 CH = 65536;
constexpr u32 YT = 32768;
constexpr u32 F_CALL = 0;
constexpr u32 F_TMO = 16384;
constexpr u32 F_TICKET = F_TMO + 1024;
constexpr u32 F_UNIT = F_TICKET + 1024;
constexpr u32 F_AMAX = 20480;
constexpr u32 F_YRDY = 28672;
constexpr u32 F_LINK = 32768;
constexpr u32 F_DBG = F_LINK + 8 * 64 * 128;
constexpr u32 OFF_XS = 1u << 20;
constexpr u32 OFF_WS = OFF_XS + 6 * 16 * CH;
constexpr u32 OFF_YOUT = OFF_WS + 6 * 16 * CH;
constexpr u32 OFF_YIN = OFF_YOUT + 192 * YT;
constexpr u32 WS_USED = OFF_YIN + 192 * YT;
constexpr u32 OOB = 0x80000000u;

constexpr int NSLOT = 4;
constexpr u32 STAGE = 32768;
constexpr u32 L_CTRL = NSLOT * STAGE;
constexpr u32 LDS_BYTES = L_CTRL + 1024;
constexpr u64 DEADLINE = 50000000ull;
constexpr int NPUSH = 7;
constexpr int NGEMM = 32 - NPUSH;
#define NSTO 2u
#define NLPS 3u
constexpr u32 NLD = NSTO * NLPS;
static_assert(NSTO + NLD <= 8 && NLPS <= 4, "pusher wave roles");
#define RAUX 17
#define SEQ_ID 3
#define LATE_TICKET 1
#define WIN 3
#define LEAD_A 5

struct Params {
  const char* x;
  const char* w;
  char* out;
  u32 rank, pad;
  char* peer[NR];
  u32 pad2[8];
  u32 seqw[8];
  u32 posw[8];
  u32 tab[7][16];
};

__device__ __forceinline__ rsrc_t mk_rsrc(const void* p, u32 bytes) {
  return __builtin_amdgcn_make_buffer_rsrc((void*)p, 0, bytes, 0x00020000);
}
__device__ __forceinline__ u32 rfl(u32 v) { return __builtin_amdgcn_readfirstlane(v); }
__device__ __forceinline__ u32 ld_sys(const char* p) { return __hip_atomic_load((const GLBP u32*)p, __ATOMIC_RELAXED, __HIP_MEMORY_SCOPE_SYSTEM); }
__device__ __forceinline__ u32 ld_agt(const char* p) { return __hip_atomic_load((const GLBP u32*)p, __ATOMIC_RELAXED, __HIP_MEMORY_SCOPE_AGENT); }
__device__ __forceinline__ void st_sys(char* p, u32 v) { __hip_atomic_store((GLBP u32*)p, v, __ATOMIC_RELAXED, __HIP_MEMORY_SCOPE_SYSTEM); }
__device__ __forceinline__ void st_agt(char* p, u32 v) { __hip_atomic_store((GLBP u32*)p, v, __ATOMIC_RELAXED, __HIP_MEMORY_SCOPE_AGENT); }
#define WAIT_VM(n) asm volatile("s_waitcnt vmcnt(" #n ")" ::: "memory")
#define WAIT_LGKM0() asm volatile("s_waitcnt lgkmcnt(0)" ::: "memory")
#define BAR() __builtin_amdgcn_s_barrier()
#define SCHED() __builtin_amdgcn_sched_barrier(0)

__device__ __forceinline__ u32 row_base(u32 q, u32 r) {
  if (r < 4) return 512u * (q ^ 1u) + 128u * r;
  if (r < 12) return 512u * (q ^ (4u + (r & 3u))) + 256u * ((q >> 1) & 1u) + 128u * ((r - 4u) >> 2);
  return 512u * q + 128u * (r - 12u);
}
__device__ __forceinline__ u32 k_half(u32 q, u32 p) { return (q ^ ((q >> 1) & p)) & 1u; }

template <int NWORD>
__device__ __forceinline__ void wave_wait(rsrc_t r, u32 off, bool want, u32 lane, u32 call, char* tmo) {
  const u64 t0 = __builtin_amdgcn_s_memrealtime();
  const u32 o = lane == 63u ? F_TMO : want ? off : OOB;
  for (;;) {
    const u32x4 v = __builtin_amdgcn_raw_buffer_load_b128(r, o, 0, 17);
    bool ok = (int)(v.x - call) >= 0;
    if (NWORD == 4) ok = ok && (int)(v.y - call) >= 0 && (int)(v.z - call) >= 0 && (int)(v.w - call) >= 0;
    if (__all(ok || !want || lane == 63u)) break;
    if (__builtin_amdgcn_readlane(v.x, 63) == call) break;
    if (__builtin_amdgcn_s_memrealtime() - t0 > DEADLINE) { st_agt(tmo, call); break; }
    __builtin_amdgcn_s_sleep(2);
  }
  asm volatile("" ::: "memory");
}

#define STAMP(i) do {} while (0)

}

__global__ void __launch_bounds__(512, 2) fused_kernel(Params params_by_value) {
  (void)params_by_value;
  const CSTP Params* const K = (const CSTP Params*)__builtin_amdgcn_kernarg_segment_ptr();
  const char* const gx = K->x;
  const char* const gw = K->w;
  const u32 warm = K->tab[min(blockIdx.x >> 3, 6u)][0];
  char* const far = K->peer[((blockIdx.x >> 3) + 1u) & 7u];
  asm volatile("" :: "s"(warm), "s"(far));
  extern __shared__ __attribute__((aligned(16))) char smem[];
  LDSP char* lds = (LDSP char*)smem;
  volatile LDSP u32* ctrl = (volatile LDSP u32*)(lds + L_CTRL);
  const u32 tid = threadIdx.x, lane = tid & 63u, wid = rfl(tid >> 6);
  const u32 q = K->rank & 7u;
  char* const me = K->peer[0];
  char* const tmo = me + F_TMO;
  const u32 grp = blockIdx.x & 7u, li = blockIdx.x >> 3;
  char* const my_ctr = me + F_CALL + blockIdx.x * 64u;

  if (li < (u32)NPUSH) {
    const u32 d = li + 1u;
    const u32 n_in = d == 1u ? 0u : d >= 4u ? 30u : 32u;
    const u32 total = d >= 4u ? 30u * 16u + 32u * 8u : 512u;
    const rsrc_t r_me = mk_rsrc(me, WS_USED);
    const u32 vl = lane * 16u;
    const u32 v0 = lane == 0u ? 0u : OOB;
    volatile LDSP u32x4* const full = (volatile LDSP u32x4*)(lds + L_CTRL);
    volatile LDSP u32* const freed = (volatile LDSP u32*)(lds + L_CTRL + 256);
    const u32 callv = ld_agt(my_ctr);
    if (tid < 128u) ctrl[tid] = 0;
    WAIT_LGKM0();
    BAR();
    if (wid == 0) STAMP(0);
    if (wid >= NSTO && wid < NSTO + NLD) {
      const u32 j = wid - NSTO;
      const u32 dst_rank = (q ^ d) & 7u;
      const u32 h = k_half(dst_rank, q);
      const u32 vx0 = (lane >> 3) * 1024u + (((lane & 7u) ^ (lane >> 4)) << 4), vx1 = vx0 ^ 64u;
      const u32 vw = ((lane >> 2) & 3u) * 8192u + (32u * (lane >> 4) + 8u * (lane & 3u)) * 2u;
      LDSP char* const slot = lds + j * 4096u;
      u32 pend = (j / NSTO) * (NSTO * 8u) + (j % NSTO) * 8u + grp;
      u32 call = 0, n = 0;
      for (;; ++n) {
        if (n) {
          while (rfl(freed[j]) != n) __builtin_amdgcn_s_sleep(1);
          const u32 tkv = (u32)__builtin_amdgcn_raw_ptr_buffer_atomic_add_i32(1, r_me, v0 + F_UNIT + li * 128u, 0, 0);
          call = rfl(callv) + 1u;
          pend = rfl(tkv) - (call - 1u) * total + NLD * 8u;
        }
        const bool valid = pend < total;
        const bool in = pend < n_in * 16u;
        const u32 r = pend - n_in * 16u;
        const u32 it = (in ? pend >> 4 : n_in + (r >> 3)) & 63u, sub = in ? pend & 15u : r & 7u;
        const u32 c = (K->tab[li][it >> 2] >> ((it & 3u) * 8u)) & 63u;
        u32 so, dl, st;
        if (!in) {
          const u32 ti = d == 1u ? c : d * 32u - 96u + c;
          so = OFF_YOUT + ti * YT + sub * 4096u; dl = OFF_YIN + ti * YT; st = 1024u;
          if (valid) {
            call = rfl(callv) + 1u;
            wave_wait<1>(r_me, F_YRDY + ti * 16u + (sub >> 1) * 4u, lane == 0u, lane, call, tmo);
          }
        } else if (c < 16u) {
          so = row_base(dst_rank, c) * 1024u + h * 512u + (sub >> 2) * 128u + (sub & 3u) * 32768u; dl = OFF_XS + ((d - 2u) * 16u + c) * CH; st = 8192u;
        } else {
          so = (h * 256u + sub * 16u) * 8192u + ((dst_rank >> 2) * 2048u + (c - 16u) * 128u) * 2u; dl = OFF_WS + ((d - 2u) * 16u + (c - 16u)) * CH; st = 32768u;
        }
        while (rfl(freed[j]) != n) __builtin_amdgcn_s_sleep(1);
        if (!valid) break;
        const rsrc_t rs = mk_rsrc((const void*)(!in ? (u64)me : c < 16u ? (u64)gx : (u64)gw), in ? X_BYTES : WS_USED);
        const u32 ve = !in ? vl : c < 16u ? vx0 : vw, vo = !in ? vl : c < 16u ? vx1 : vw;
#pragma unroll
        for (u32 i = 0; i < 4; ++i) __builtin_amdgcn_raw_ptr_buffer_load_lds(rs, slot + i * 1024u, 16, (i & 1u) ? vo : ve, so + i * st, 0, 16);
        WAIT_VM(0);
        if (lane == 0u) full[j] = u32x4{n + 1u, dl + sub * 4096u, F_LINK + (d * 64u + c) * 128u + sub * 4u, pend};
      }
      if (lane == 0u) full[j] = u32x4{n + 1u, OOB, 0u, ~0u};
    } else if (wid < NSTO) {
      const rsrc_t r_dst = mk_rsrc(far, WS_USED);
      const u32 js = (lane & 3u) % NLPS * NSTO + wid;
      u32 cnt = 0, prev = OOB >> 1;
      const u32 call = rfl(callv) + 1u;
      for (;;) {
        const u32x4 e = full[js];
        const u32 key = e.x != cnt ? e.w : ~0u;
        u32 best = 0, bk = __builtin_amdgcn_readlane(key, 0);
#pragma unroll
        for (u32 i = 1; i < NLPS; ++i) { const u32 k2 = __builtin_amdgcn_readlane(key, i); if (k2 < bk) { bk = k2; best = i; } }
        if (bk == ~0u) {
          if (prev != OOB >> 1) { WAIT_VM(0); __builtin_amdgcn_raw_buffer_store_b32(call, r_dst, v0 + prev, 0, 17); prev = OOB >> 1; }
          if (__all(e.x != cnt)) break;
          __builtin_amdgcn_s_sleep(1);
          continue;
        }
        const u32 j = best * NSTO + wid;
        u32x4 buf[4];
#pragma unroll
        for (u32 i = 0; i < 4; ++i) buf[i] = *(volatile LDSP u32x4*)(lds + j * 4096u + i * 1024u + vl);
        const u32 dl = __builtin_amdgcn_readlane(e.y, best), fl = __builtin_amdgcn_readlane(e.z, best);
        if (js == j) ++cnt;
        WAIT_LGKM0();
        if (lane == best) freed[j] = cnt;
#define ST_(i) __builtin_amdgcn_raw_buffer_store_b128(buf[i], r_dst, vl + dl + (i) * 1024u, 0, 17)
#define FL_() __builtin_amdgcn_raw_buffer_store_b32(call, r_dst, v0 + prev, 0, 17)
        WAIT_VM(3); ST_(0); WAIT_VM(2); ST_(1); WAIT_VM(2); FL_(); ST_(2); WAIT_VM(3); ST_(3);
        prev = fl;
      }
      STAMP(1 + wid);
    }
    BAR();
    if (tid == 0) st_agt(my_ctr, rfl(callv) + 1u);
    return;
  }

  const u32 kidx = li - NPUSH;
  const u32 ga = grp >> 1, gb = grp & 1u;
  const bool loader = wid >= 4u;
  const u32 lw = wid & 3u;
  u32 call = 0;
  if (wid == 4u) {
    call = rfl(ld_agt(my_ctr)) + 1u;
    if (lane == 0) ctrl[16] = call;
    WAIT_LGKM0();
    STAMP(0);
  }
  const rsrc_t r_me = mk_rsrc(me, WS_USED);

  u32 kind, rt, ct;
  u32 fin = 0;
  bool have = kidx < 24u;
  if (have) { rt = ga + 4u * (kidx >> 3); ct = 2u * (kidx & 7u) + gb; kind = rt < 4u ? 0u : 1u; }
  else { rt = 0; ct = 0; kind = 0; }

  for (int round = 0; round < 2; ++round) {
    if (round == 1) {
      if (wid == 4u) {
        u32 t = 0;
        if (lane == 0) t = __hip_atomic_fetch_add((GLBP u32*)(me + F_TICKET + grp * 128u), 1u, __ATOMIC_RELAXED, __HIP_MEMORY_SCOPE_AGENT);
        t = rfl(t) - (call - 1u) * (u32)NGEMM;
        if (lane == 0) { ctrl[0] = t; st_agt(my_ctr, call); }
        WAIT_LGKM0();
      }
      BAR();
      const u32 t = rfl(ctrl[0]);
      if (t >= 16u) return;
      have = true; fin = grp * 16u + t;
      kind = t < 8u ? 2u : 3u; rt = 12u + ga; ct = 2u * (t & 7u) + gb;
    }
    if (!have) continue;

    const u32 grow = row_base(q, rt);
    const u32 gcol = ((q >> 2) ^ (kind == 3u ? 1u : 0u)) * 2048u + ct * 128u;
    const u32 skip = kind == 1u ? 2u + (rt & 3u) : 99u;
    const u32 nrem = kind == 3u ? 0u : (kind == 1u ? 20u : 24u);

    if (loader) {
      const rsrc_t r_x = mk_rsrc(gx, X_BYTES), r_w = mk_rsrc(gw, W_BYTES);
      const u32 vx = (lw * 8u + (lane >> 3)) * 1024u + (((lane & 7u) ^ (((lw & 1u) * 4u + (lane >> 4)) & 7u)) << 4);
      const u32 vw = (lw * 4u + ((lane >> 2) & 3u)) * 8192u + (32u * (lane >> 4) + 8u * (lane & 3u)) * 2u;
      const u32 vs = lw * 1024u + lane * 16u;
      auto issue_own = [&](u32 s) {
        LDSP char* dst = lds + (s & 3u) * STAGE + lw * 1024u;
        const u32 sx = grow * 1024u + s * 128u, sw = s * 64u * 8192u + gcol * 2u;
#pragma unroll
        for (u32 i = 0; i < 4; ++i) {
          __builtin_amdgcn_raw_ptr_buffer_load_lds(r_x, dst + i * 4096u, 16, vx, sx + i * 32768u, 0, 0);
          __builtin_amdgcn_raw_ptr_buffer_load_lds(r_w, dst + 16384u + i * 4096u, 16, vw, sw + i * 131072u, 0, 0);
        }
      };
      auto issue_rem = [&](u32 s) {
        LDSP char* dst = lds + (s & 3u) * STAGE + lw * 1024u;
        u32 sg = s >> 2; if (sg >= skip) ++sg;
        const u32 sx = OFF_XS + (sg * 16u + rt) * CH + (s & 3u) * 16384u, sw = OFF_WS + (sg * 16u + ct) * CH + (s & 3u) * 16384u;
#pragma unroll
        for (u32 i = 0; i < 4; ++i) {
          __builtin_amdgcn_raw_ptr_buffer_load_lds(r_me, dst + i * 4096u, 16, vs, sx + i * 4096u, 0, RAUX & 31);
          __builtin_amdgcn_raw_ptr_buffer_load_lds(r_me, dst + 16384u + i * 4096u, 16, vs, sw + i * 4096u, 0, RAUX & 31);
        }
      };
      const u32 pa = (K->posw[rt >> 2] >> ((rt & 3u) * 8u)) & 31u, pb = (K->posw[4u + (ct >> 2)] >> ((ct & 3u) * 8u)) & 31u;
      const u32 pos = pa > pb ? pa : pb;
      if (nrem && pos >= (u32)LEAD_A) {
        if (wid == 4u) {
          const u32 pe = pos - (u32)LEAD_A;
          const u32 ce = (K->seqw[pe >> 2] >> ((pe & 3u) * 8u)) & 31u;
          wave_wait<1>(r_me, F_LINK + (2u * 64u + ce) * 128u, lane == 0, lane, call, tmo);
        }
      }
      BAR();
#pragma unroll 1
      for (u32 s = 0; s < 4; ++s) issue_own(s);
      WAIT_VM(24); BAR();
#pragma unroll 1
      for (u32 s = 0; s < 4; ++s) { WAIT_VM(16); BAR(); issue_own(s + 4u); }
      WAIT_VM(16); BAR(); WAIT_VM(8); BAR(); WAIT_VM(0); BAR(); BAR();
      if (nrem) {
        if (wid == 4u) {
          STAMP(round * 4 + 1);
          const u32 sg = lane >> 3;
          wave_wait<4>(r_me, F_LINK + ((sg + 2u) * 64u + ((lane & 4u) ? 16u + ct : rt)) * 128u + (lane & 3u) * 16u, sg < 6u && sg != skip, lane, call, tmo);
          STAMP(round * 4 + 2);
        }
        BAR();
#pragma unroll 1
        for (u32 s = 0; s < 4; ++s) issue_rem(s);
        WAIT_VM(24); BAR();
#pragma unroll 1
        for (u32 s = 0; s + 4u < nrem; ++s) { WAIT_VM(16); BAR(); issue_rem(s + 4u); }
        WAIT_VM(16); BAR(); WAIT_VM(8); BAR(); WAIT_VM(0); BAR(); BAR();
      }
      if (kind >= 2u) {
        const u32 nsrc = kind - 1u;
        for (u32 i = 0; i < nsrc; ++i) {
          if (wid == 4u) {
            if (kind == 2u) wave_wait<4>(r_me, F_LINK + (64u + ga * 16u + ct) * 128u + (lane & 1u) * 16u, lane < 2u, lane, call, tmo);
            else {
              const u32 dd = (q ^ (((q >> 2) ^ 1u) * 4u + (ga >> 1) * 2u + i)) & 7u;
              wave_wait<4>(r_me, F_LINK + (dd * 64u + 32u + (ga & 1u) * 16u + ct) * 128u + (lane & 1u) * 16u, lane < 2u, lane, call, tmo);
            }
          }
          BAR();
        }
        if (wid == 4u) STAMP(7);
        BAR();
        const u32 callw = rfl(ctrl[16]);
        if (wid == 4u) {
          u32 m = ctrl[4 + (lane & 3u)];
          m = max(m, (u32)__shfl_xor((int)m, 1)); m = max(m, (u32)__shfl_xor((int)m, 2));
          if (lane < 8u) __hip_atomic_store((GLBP u64*)(K->peer[lane] + F_AMAX + (q * 128u + fin) * 8u), ((u64)callw << 32) | m, __ATOMIC_RELAXED, __HIP_MEMORY_SCOPE_SYSTEM);
          STAMP(8);
        }
        u32 m = 0;
        {
          const u64 t0 = __builtin_amdgcn_s_memrealtime();
          const u32 off = F_AMAX + (lw * 64u + lane) * 32u;
          for (;;) {
            const u32x4 g0 = __builtin_amdgcn_raw_buffer_load_b128(r_me, off, 0, 17);
            const u32x4 g1 = __builtin_amdgcn_raw_buffer_load_b128(r_me, off + 16u, 0, 17);
            const u32 tv = ld_agt(tmo);
            const bool ok = g0.y == callw && g0.w == callw && g1.y == callw && g1.w == callw;
            m = max(max(g0.x, g0.z), max(g1.x, g1.z));
            if (__all(ok)) break;
            if (rfl(tv) == callw) break;
            if (__builtin_amdgcn_s_memrealtime() - t0 > DEADLINE) { st_agt(tmo, callw); break; }
            __builtin_amdgcn_s_sleep(2);
          }
        }
#pragma unroll
        for (int o = 32; o >= 1; o >>= 1) m = max(m, (u32)__shfl_xor((int)m, o));
        const u32 bad = rfl(ld_agt(tmo)) == callw ? 1u : 0u;
        if (lane == 0) { ctrl[8 + lw] = m; ctrl[12 + lw] = bad; }
        WAIT_LGKM0();
        BAR();
        if (wid == 4u) STAMP(9);
      }
    } else {
      const u32 wn = wid & 1u, wm = wid >> 1;
      const u32 hh = lane >> 5, r31 = lane & 31u;
      const u32 z = (lane >> 4) & 1u, tq = (lane >> 2) & 3u, tp = lane & 3u;
      const u32 aw = 16384u + hh * 2048u + wn * 512u + tq * 64u + (16u * (tp & 1u) + 8u * z + 4u * (tp >> 1)) * 2u;
      const u32 axb = (wm * 64u + r31) * 128u, sw7 = (r31 >> 1) & 7u;
      u32 ax[4];
#pragma unroll
      for (u32 s = 0; s < 4; ++s) ax[s] = axb + (((2u * s + hh) ^ sw7) << 4);
      f32x16 acc[2][2] = {};
      s16x8 fa[2][2], fb[2][2];
      auto rd = [&](int buf, u32 slot_off, int ks) {
#pragma unroll
        for (int f = 0; f < 2; ++f) {
          const s16x4 lo = __builtin_amdgcn_ds_read_tr16_b64_v4i16((LDSP s16x4*)(lds + slot_off + aw + ks * 4096 + f * 256));
          const s16x4 hi = __builtin_amdgcn_ds_read_tr16_b64_v4i16((LDSP s16x4*)(lds + slot_off + aw + ks * 4096 + f * 256 + 1024));
          fa[buf][f] = __builtin_shufflevector(lo, hi, 0, 1, 2, 3, 4, 5, 6, 7);
          fb[buf][f] = *(LDSP s16x8*)(lds + slot_off + ax[ks] + f * 4096);
        }
      };
      auto mm = [&](int buf) {
#pragma unroll
        for (int fn = 0; fn < 2; ++fn)
#pragma unroll
          for (int fm = 0; fm < 2; ++fm)
            acc[fn][fm] = __builtin_amdgcn_mfma_f32_32x32x16_bf16(__builtin_bit_cast(bf16x8, fa[buf][fn]), __builtin_bit_cast(bf16x8, fb[buf][fm]), acc[fn][fm], 0, 0, 0);
      };
#pragma unroll 1
      for (int batch = 0; batch < 2; ++batch) {
        const u32 nst = batch == 0 ? 8u : nrem;
        if (nst == 0) break;
        BAR();
        BAR();
        asm volatile("" ::: "memory");
        rd(0, 0, 0);
#pragma unroll 1
        for (u32 s = 0; s < nst; ++s) {
          const u32 so = (s & 3u) * STAGE, sn = ((s + 1u) & 3u) * STAGE;
          rd(1, so, 1); SCHED(); mm(0); SCHED();
          rd(0, so, 2); SCHED(); mm(1); SCHED();
          rd(1, so, 3); SCHED(); mm(0); SCHED();
          WAIT_LGKM0(); BAR(); asm volatile("" ::: "memory");
          rd(0, sn, 0);
          SCHED(); mm(1); SCHED();
        }
      }
      const u32 vy = wid * 8192u + lane * 16u;
      if (kind < 2u) {
        const u32 ti = kind == 0u ? rt * 16u + ct : 64u + (rt & 3u) * 32u + ((rt - 4u) >> 2) * 16u + ct;
        const u32 so = OFF_YOUT + ti * YT;
        const u32 callc = ctrl[16];
#pragma unroll
        for (int f = 0; f < 4; ++f)
#pragma unroll
          for (int hf = 0; hf < 2; ++hf) {
            u32x4 v;
#pragma unroll
            for (int e = 0; e < 4; ++e) {
              const f32x2 p = {acc[f >> 1][f & 1][hf * 8 + e * 2], acc[f >> 1][f & 1][hf * 8 + e * 2 + 1]};
              v[e] = __builtin_bit_cast(u32, __builtin_convertvector(p, bf16x2));
            }
            __builtin_amdgcn_raw_buffer_store_b128(v, r_me, vy + so + (f * 2 + hf) * 1024, 0, 16);
          }
        WAIT_VM(0);
        if (lane == 0) st_agt(me + F_YRDY + ti * 16u + wid * 4u, callc);
        if (wid == 0) STAMP(3);
      } else {
        const u32 nsrc = kind - 1u;
        for (u32 i = 0; i < nsrc; ++i) {
          BAR();
          asm volatile("" ::: "memory");
          const u32 dd = (q ^ (((q >> 2) ^ 1u) * 4u + (ga >> 1) * 2u + i)) & 3u;
          const u32 so = OFF_YIN + (kind == 2u ? ga * 16u + ct : 64u + dd * 32u + (ga & 1u) * 16u + ct) * YT;
          u32x4 v[8];
#pragma unroll
          for (int j = 0; j < 8; ++j) v[j] = __builtin_amdgcn_raw_buffer_load_b128(r_me, vy + j * 1024, so, RAUX & 31);
#pragma unroll
          for (int j = 0; j < 8; ++j)
#pragma unroll
            for (int e = 0; e < 4; ++e) {
              acc[j >> 2][(j >> 1) & 1][(j & 1) * 8 + e * 2] += __builtin_bit_cast(float, v[j][e] << 16);
              acc[j >> 2][(j >> 1) & 1][(j & 1) * 8 + e * 2 + 1] += __builtin_bit_cast(float, v[j][e] & 0xFFFF0000u);
            }
        }
        float mx = 0.f;
#pragma unroll
        for (int f = 0; f < 4; ++f)
#pragma unroll
          for (int e = 0; e < 16; ++e) mx = fmaxf(mx, acc[f >> 1][f & 1][e]);
        u32 m = __builtin_bit_cast(u32, mx);
#pragma unroll
        for (int o = 32; o >= 1; o >>= 1) m = max(m, (u32)__shfl_xor((int)m, o));
        if (lane == 0) ctrl[4 + wid] = m;
        WAIT_LGKM0();
        BAR();
        BAR();
        const u32 am = max(max(ctrl[8], ctrl[9]), max(ctrl[10], ctrl[11]));
        const u32 bad = ctrl[12] | ctrl[13] | ctrl[14] | ctrl[15];
        const float amax = __builtin_bit_cast(float, am);
        const float scale = amax / 127.0f;
        const float inv = amax > 0.f ? 1.0f / scale : 0.f;
        const rsrc_t r_out = mk_rsrc(K->out, OUT_BYTES);
        const u32 vo = (ga * 128u + wm * 64u + r31) * 4096u + gcol + wn * 64u + hh * 16u;
#pragma unroll
        for (int f = 0; f < 4; ++f) {
          u32x4 v;
#pragma unroll
          for (int e = 0; e < 4; ++e) {
            u32 pk = 0;
#pragma unroll
            for (int b = 0; b < 4; ++b) {
              const float y = fminf(__builtin_rintf(fmaxf(acc[f >> 1][f & 1][e * 4 + b], 0.f) * inv), 127.f);
              pk |= (u32)y << (8 * b);
            }
            v[e] = bad ? 0x80808080u : pk;
          }
          __builtin_amdgcn_raw_buffer_store_b128(v, r_out, vo + (f & 1) * 32 * 4096 + (f >> 1) * 32, 0, 0);
        }
      }
    }
  }
}

extern "C" void kernel_launch(int rank, int nranks, ncclComm_t comm, hipStream_t stream,
                              void* const* d_in, const int* in_sizes, int n_in,
                              void* d_out, int out_size, const int* in_dtypes, int out_dtype,
                              void* const* d_out_all, void* const* d_ws_all, size_t ws_bytes) {
  (void)comm; (void)in_dtypes; (void)out_dtype; (void)d_out_all;
  if (nranks != NR || (unsigned)rank >= (unsigned)NR || n_in < 2 || ws_bytes < WS_USED ||
      (size_t)in_sizes[0] * 2 != X_BYTES || (size_t)in_sizes[1] * 2 != W_BYTES || (u32)out_size != OUT_BYTES) {
    fprintf(stderr, "kernel_launch: unexpected problem shape\n");
    return;
  }
  Params P;
  P.x = (const char*)d_in[0];
  P.w = (const char*)d_in[1];
  P.out = (char*)d_out;
  for (int p = 0; p < NR; ++p) P.peer[p] = (char*)d_ws_all[rank ^ p];
  P.rank = (u32)rank; P.pad = 0;
  unsigned char seq[32], pos[32], tab[7][64];
  int n = 0;
  {
    static const char* const orders[] = {
        "xwxwxwxwxwxwxwxwxwxwxwxwwwwwxxxx",
        "xwxwxwxwwwxwwxwwxwwxwxwxwxwxxxxx",
        "xwxwxwxwwwwxwwxwwxwwxwxwxwxxxxxx",
        "xwxwxwxwwxwwxwwxwxwxwxwxwxwwxxxx",
    };
    int nx = 0, nw = 0;
    for (const char* p = orders[SEQ_ID]; *p; ++p) seq[n++] = (unsigned char)(*p == 'x' ? nx++ : 16 + nw++);
  }
  for (int i = 0; i < 32; ++i) pos[seq[i]] = (unsigned char)i;
  memset(tab, 0, sizeof tab);
  for (int d = 1; d < 8; ++d) {
    int m = 0;
    for (int i = 0; i < 32 && d >= 2; ++i) {
      const int c = seq[i];
      if (!(d >= 4 && c >= 4 && c < 12 && (c & 3) == d - 4)) tab[d - 1][m++] = (unsigned char)c;
    }
    const int nt = d == 1 ? 64 : d >= 4 ? 32 : 0;
    for (int ps = 0; ps < 32; ++ps)
      for (int t = 0; t < nt; ++t) {
        const int a = pos[d == 1 ? t >> 4 : d + 4 * (t >> 4)], b = pos[16 + (t & 15)];
        if ((a > b ? a : b) == ps) tab[d - 1][m++] = (unsigned char)(d == 1 ? t : 32 + t);
      }
  }
  memcpy(P.seqw, seq, 32); memcpy(P.posw, pos, 32); memcpy(P.tab, tab, sizeof tab);
  hipError_t e = hipFuncSetAttribute((const void*)fused_kernel, hipFuncAttributeMaxDynamicSharedMemorySize, (int)LDS_BYTES);
  if (e != hipSuccess) { fprintf(stderr, "hipFuncSetAttribute: %s\n", hipGetErrorString(e)); return; }
  fused_kernel<<<dim3(256), dim3(512), LDS_BYTES, stream>>>(P);
  e = hipGetLastError();
  if (e != hipSuccess) fprintf(stderr, "launch: %s\n", hipGetErrorString(e));
}
```

```cpp
#include <hip/hip_runtime.h>
#include <rccl/rccl.h>
#include <cstdio>
#include <cstring>
#include <algorithm>

typedef unsigned int u32;
typedef unsigned long long u64;
typedef __attribute__((ext_vector_type(4))) unsigned int u32x4;
typedef __attribute__((ext_vector_type(2))) unsigned int u32x2;
typedef __attribute__((ext_vector_type(4))) short s16x4;
typedef __attribute__((ext_vector_type(8))) short s16x8;
typedef __attribute__((ext_vector_type(8))) __bf16 bf16x8;
typedef __attribute__((ext_vector_type(2))) __bf16 bf16x2;
typedef __attribute__((ext_vector_type(2))) float f32x2;
typedef __attribute__((ext_vector_type(16))) float f32x16;
#define LDSP __attribute__((address_space(3)))
#define GLBP __attribute__((address_space(1)))
#define CSTP __attribute__((address_space(4)))
typedef __amdgpu_buffer_rsrc_t rsrc_t;

namespace {

constexpr int NR = 8;
constexpr u32 X_BYTES = 4096u * 512u * 2u;
constexpr u32 W_BYTES = 512u * 4096u * 2u;
constexpr u32 OUT_BYTES = 512u * 4096u;
constexpr u32 CH = 65536;
constexpr u32 YT = 32768;
constexpr u32 F_CALL = 0;
constexpr u32 F_TMO = 16384;
constexpr u32 F_TICKET = F_TMO + 1024;
constexpr u32 F_UNIT = F_TICKET + 1024;
constexpr u32 F_AMAX = 20480;
constexpr u32 F_YRDY = 28672;
constexpr u32 F_LINK = 32768;
constexpr u32 F_REL = F_LINK + 8 * 64 * 128;
constexpr u32 F_DBG = F_REL + 2 * 64 * 128;
constexpr u32 OFF_XS = 1u << 20;
constexpr u32 OFF_WS = OFF_XS + 6 * 16 * CH;
constexpr u32 OFF_YOUT = OFF_WS + 6 * 16 * CH;
constexpr u32 OFF_YIN = OFF_YOUT + 192 * YT;
constexpr u32 OFF_REL = OFF_YIN + 192 * YT;
constexpr u32 WS_USED = OFF_REL + 128 * YT;
constexpr u32 OOB = 0x80000000u;

constexpr int NSLOT = 4;
constexpr u32 STAGE = 32768;
constexpr u32 L_CTRL = NSLOT * STAGE;
constexpr u32 L_SNAP = L_CTRL + 1024;
constexpr u32 LDS_BYTES = L_SNAP + 1024;
constexpr u64 DEADLINE = 50000000ull;
constexpr int NPUSH = 7;
constexpr int NGEMM = 32 - NPUSH;
#define NSTO 2u
#define NLPS 3u
constexpr u32 NLD = NSTO * NLPS;
static_assert(NSTO + NLD <= 8 && NLPS <= 4, "pusher wave roles");
#define RAUX 17
#define LATE_TICKET 1
#define WIN 3
#define OUT_AUX 0
#define ROT 1
#define BACKOFF 12
#define SPEC_MISSING 12
#define BACKOFF2 36
#define BSLEEP 10
#define GLAY 0
#define TAILN 0
#define SCHED_ID 3
#define LEAD_US 5.0
#define DIRECT_N 12
#define MINSEG 2

enum : u32 { E_X = 0, E_W = 1, E_Y = 2, E_FW = 3, E_R1 = 4, E_R2 = 5 };
struct Params {
  const char* x;
  const char* w;
  char* out;
  u32 rank, pad;
  char* peer[NR];
  u32 nent[8];
  u32 pad2[16];
  u32 tab[7][48];
  u32 pace[64];
};

__device__ __forceinline__ rsrc_t mk_rsrc(const void* p, u32 bytes) {
  return __builtin_amdgcn_make_buffer_rsrc((void*)p, 0, bytes, 0x00020000);
}
__device__ __forceinline__ u32 rfl(u32 v) { return __builtin_amdgcn_readfirstlane(v); }
__device__ __forceinline__ u32 ld_sys(const char* p) { return __hip_atomic_load((const GLBP u32*)p, __ATOMIC_RELAXED, __HIP_MEMORY_SCOPE_SYSTEM); }
__device__ __forceinline__ u32 ld_agt(const char* p) { return __hip_atomic_load((const GLBP u32*)p, __ATOMIC_RELAXED, __HIP_MEMORY_SCOPE_AGENT); }
__device__ __forceinline__ void st_sys(char* p, u32 v) { __hip_atomic_store((GLBP u32*)p, v, __ATOMIC_RELAXED, __HIP_MEMORY_SCOPE_SYSTEM); }
__device__ __forceinline__ void st_agt(char* p, u32 v) { __hip_atomic_store((GLBP u32*)p, v, __ATOMIC_RELAXED, __HIP_MEMORY_SCOPE_AGENT); }
#define WAIT_VM(n) asm volatile("s_waitcnt vmcnt(" #n ")" ::: "memory")
#define WAIT_LGKM0() asm volatile("s_waitcnt lgkmcnt(0)" ::: "memory")
#define BAR() __builtin_amdgcn_s_barrier()
#define SCHED() __builtin_amdgcn_sched_barrier(0)

__device__ __forceinline__ u32 row_base(u32 q, u32 r) {
  if (r < 4) return 512u * (q ^ 1u) + 128u * r;
  if (r < 12) return 512u * (q ^ (4u + (r & 3u))) + 256u * ((q >> 1) & 1u) + 128u * ((r - 4u) >> 2);
  return 512u * q + 128u * (r - 12u);
}
__device__ __forceinline__ u32 k_half(u32 q, u32 p) { return (q ^ ((q >> 1) & p)) & 1u; }

__device__ __forceinline__ u64 lds_rd64(u32 addr) {
  u32x2 v;
  asm volatile("ds_read_b64 %0, %1\n\ts_waitcnt lgkmcnt(0)" : "=&v"(v) : "v"(addr) : "memory");
  return ((u64)rfl(v.y) << 32) | rfl(v.x);
}
__device__ __forceinline__ void lds_wr64(u32 addr, u64 x) {
  const u32x2 v = {(u32)x, (u32)(x >> 32)};
  asm volatile("ds_write_b64 %0, %1\n\ts_waitcnt lgkmcnt(0)" :: "v"(addr), "v"(v) : "memory");
}
__device__ __forceinline__ u32x4 lds_rd128(u32 addr) {
  u32x4 v;
  asm volatile("ds_read_b128 %0, %1\n\ts_waitcnt lgkmcnt(0)" : "=&v"(v) : "v"(addr) : "memory");
  return v;
}
__device__ __forceinline__ u32 seg_mask(u64 b, u32 lane) { return (u32)__ballot(lane < 6u && ((b >> ((lane & 7u) * 8u)) & 255u) == 255u); }
__device__ __forceinline__ bool all_ge(u32x4 v, u32 c) { return (int)((v.x - c) | (v.y - c) | (v.z - c) | (v.w - c)) >= 0; }

__device__ __forceinline__ u32 wave_max(u32 v) {
  v = max(v, (u32)__builtin_amdgcn_update_dpp(0, (int)v, 0x111, 0xf, 0xf, false));
  v = max(v, (u32)__builtin_amdgcn_update_dpp(0, (int)v, 0x112, 0xf, 0xf, false));
  v = max(v, (u32)__builtin_amdgcn_update_dpp(0, (int)v, 0x114, 0xf, 0xf, false));
  v = max(v, (u32)__builtin_amdgcn_update_dpp(0, (int)v, 0x118, 0xf, 0xf, false));
  v = max(v, (u32)__builtin_amdgcn_update_dpp(0, (int)v, 0x142, 0xa, 0xf, false));
  v = max(v, (u32)__builtin_amdgcn_update_dpp(0, (int)v, 0x143, 0xc, 0xf, false));
  return __builtin_amdgcn_readlane(v, 63);
}

template <int NWORD>
__device__ __forceinline__ void wave_wait(rsrc_t r, u32 off, bool want, u32 lane, u32 call, char* tmo) {
  const u64 t0 = __builtin_amdgcn_s_memrealtime();
  const u32 o = lane == 63u ? F_TMO : want ? off : OOB;
  for (;;) {
    const u32x4 v = __builtin_amdgcn_raw_buffer_load_b128(r, o, 0, 17);
    bool ok = (int)(v.x - call) >= 0;
    if (NWORD == 4) ok = ok && (int)(v.y - call) >= 0 && (int)(v.z - call) >= 0 && (int)(v.w - call) >= 0;
    if (__all(ok || !want || lane == 63u)) break;
    if (__builtin_amdgcn_readlane(v.x, 63) == call) break;
    if (__builtin_amdgcn_s_memrealtime() - t0 > DEADLINE) { st_agt(tmo, call); break; }
    __builtin_amdgcn_s_sleep(2);
  }
  asm volatile("" ::: "memory");
}

#define STAMP(i) do {} while (0)

}

__global__ void __launch_bounds__(512, 2) fused_kernel(Params params_by_value) {
  (void)params_by_value;
  const CSTP Params* const K = (const CSTP Params*)__builtin_amdgcn_kernarg_segment_ptr();
  const char* const gx = K->x;
  const char* const gw = K->w;
  const u32 warm = K->tab[min(blockIdx.x >> 3, 6u)][0];
  char* const far = K->peer[((blockIdx.x >> 3) + 1u) & 7u];
  const u32 nent = K->nent[(blockIdx.x >> 3) & 7u];
  asm volatile("" :: "s"(warm), "s"(far), "s"(nent));
  extern __shared__ __attribute__((aligned(16))) char smem[];
  LDSP char* lds = (LDSP char*)smem;
  volatile LDSP u32* ctrl = (volatile LDSP u32*)(lds + L_CTRL);
  const u32 tid = threadIdx.x, lane = tid & 63u, wid = rfl(tid >> 6);
  const u32 q = K->rank & 7u;
  char* const me = K->peer[0];
  char* const tmo = me + F_TMO;
  const u32 grp = blockIdx.x & 7u, li = blockIdx.x >> 3;
  char* const my_ctr = me + F_CALL + blockIdx.x * 64u;

  if (li < (u32)NPUSH) {
    const u32 d = li + 1u;
    const u32 total = min(nent, 96u) * 8u;
    const rsrc_t r_me = mk_rsrc(me, WS_USED);
    const u32 vl = lane * 16u;
    const u32 v0 = lane == 0u ? 0u : OOB;
    volatile LDSP u32x4* const full = (volatile LDSP u32x4*)(lds + L_CTRL);
    volatile LDSP u32* const freed = (volatile LDSP u32*)(lds + L_CTRL + 256);
    const u32 callv = ld_agt(my_ctr);
    if (tid < 128u) ctrl[tid] = 0;
    WAIT_LGKM0();
    const bool is_ld = wid >= NSTO && wid < NSTO + NLD;
    if (!is_ld || d == 1u) BAR();
    if (wid == 0) STAMP(0);
    if (is_ld) {
      const u32 j = wid - NSTO;
      const u32 dst_rank = (q ^ d) & 7u;
      const u32 h = k_half(dst_rank, q);
      const u32 vx0 = (lane >> 3) * 1024u + (((lane & 7u) ^ (lane >> 4)) << 4), vx1 = vx0 ^ 64u;
      const u32 vw = ((lane >> 2) & 3u) * 8192u + (32u * (lane >> 4) + 8u * (lane & 3u)) * 2u;
      LDSP char* const slot = lds + j * 4096u;
      u32 pend = (j / NSTO) * (NSTO * 8u) + (j % NSTO) * 8u + grp;
      u32 call = 0, n = 0;
      for (;; ++n) {
        if (n) {
          while (rfl(freed[j]) != n) __builtin_amdgcn_s_sleep(1);
          const u32 tkv = (u32)__builtin_amdgcn_raw_ptr_buffer_atomic_add_i32(1, r_me, v0 + F_UNIT + li * 128u, 0, 0);
          call = rfl(callv) + 1u;
          pend = rfl(tkv) - (call - 1u) * total + NLD * 8u;
        }
        const bool valid = pend < total;
        const u32 it = min(pend >> 3, 95u), sub = pend & 7u;
        const u32 ent = K->tab[li][it >> 1] >> ((it & 1u) * 16u);
        const u32 kind = (ent >> 8) & 7u, pl = ent & 255u;
        const bool in = kind <= E_W;
        u32 so, dl, fl, st = 1024u, gate = 0;
        if (in) {
          const u32 c = pl >> 1, u = (pl & 1u) * 8u + sub;
          fl = F_LINK + (d * 64u + kind * 16u + c) * 128u + u * 4u;
          if (kind == E_X) { so = row_base(dst_rank, c) * 1024u + h * 512u + (u >> 2) * 128u + (u & 3u) * 32768u; dl = OFF_XS; st = 8192u; }
          else { so = (h * 256u + u * 16u) * 8192u + ((dst_rank >> 2) * 2048u + c * 128u) * 2u; dl = OFF_WS; st = 32768u; }
          dl += ((d - 2u) * 16u + c) * CH + u * 4096u;
        } else if (kind == E_FW) {
          const u32 c = pl >> 1, u = ((q >> 1) & 1u) * 8u + sub;
          const u32 dq = q ^ (((q >> 2) ^ 1u) * 4u + (pl & 1u) * 2u + (d & 1u)), dr = dq ^ d;
          so = OFF_WS + ((dq - 2u) * 16u + c) * CH + u * 4096u; gate = F_LINK + (dq * 64u + 16u + c) * 128u + u * 4u;
          dl = OFF_WS + ((dr - 2u) * 16u + c) * CH + u * 4096u; fl = F_LINK + (dr * 64u + 16u + c) * 128u + u * 4u;
        } else {
          const u32 rin = ((d ^ 1u) & 1u) * 64u + pl, rout = (d & 1u) * 64u + pl;
          if (kind == E_R2) { so = OFF_REL + rin * YT; gate = F_REL + rin * 128u + sub * 4u; }
          else { so = OFF_YOUT + pl * YT; gate = F_YRDY + pl * 16u + (sub >> 1) * 4u; }
          if (kind == E_R1) { dl = OFF_REL + rout * YT; fl = F_REL + rout * 128u; }
          else { dl = OFF_YIN + pl * YT; fl = F_LINK + (pl < 64u ? 64u + pl : d * 64u + 32u + (pl & 31u)) * 128u; }
          so += sub * 4096u; dl += sub * 4096u; fl += sub * 4u;
        }
        if (valid && !in) {
          call = rfl(callv) + 1u;
          wave_wait<1>(r_me, gate, lane == 0u, lane, call, tmo);
        }
        if (!valid) break;
        const rsrc_t rs = mk_rsrc((const void*)(!in ? (u64)me : kind == E_X ? (u64)gx : (u64)gw), in ? X_BYTES : WS_USED);
        const u32 ve = !in ? vl : kind == E_X ? vx0 : vw, vo = !in ? vl : kind == E_X ? vx1 : vw;
#pragma unroll
        for (u32 i = 0; i < 4; ++i) __builtin_amdgcn_raw_ptr_buffer_load_lds(rs, slot + i * 1024u, 16, (i & 1u) ? vo : ve, so + i * st, 0, 17);
        if (n == 0u && d != 1u) BAR();
        WAIT_VM(0);
        if (lane == 0u) full[j] = u32x4{n + 1u, dl, fl, pend};
      }
      if (lane == 0u) full[j] = u32x4{n + 1u, OOB, 0u, ~0u};
    } else if (wid < NSTO) {
      const rsrc_t r_dst = mk_rsrc(far, WS_USED);
      const u32 js = (lane & 3u) % NLPS * NSTO + wid;
      u32 cnt = 0, prev = OOB >> 1;
      const u32 call = rfl(callv) + 1u;
      for (;;) {
        const u32x4 e = full[js];
        const u32 key = e.x != cnt ? e.w : ~0u;
        u32 best = 0, bk = __builtin_amdgcn_readlane(key, 0);
#pragma unroll
        for (u32 i = 1; i < NLPS; ++i) { const u32 k2 = __builtin_amdgcn_readlane(key, i); if (k2 < bk) { bk = k2; best = i; } }
        if (bk == ~0u) {
          if (prev != OOB >> 1) { WAIT_VM(0); __builtin_amdgcn_raw_buffer_store_b32(call, r_dst, v0 + prev, 0, 17); prev = OOB >> 1; }
          if (__all(e.x != cnt)) break;
          __builtin_amdgcn_s_sleep(1);
          continue;
        }
        const u32 j = best * NSTO + wid;
        u32x4 buf[4];
#pragma unroll
        for (u32 i = 0; i < 4; ++i) buf[i] = *(volatile LDSP u32x4*)(lds + j * 4096u + i * 1024u + vl);
        const u32 dl = __builtin_amdgcn_readlane(e.y, best), fl = __builtin_amdgcn_readlane(e.z, best);
        if (js == j) ++cnt;
        WAIT_LGKM0();
        if (lane == best) freed[j] = cnt;
#define ST_(i) __builtin_amdgcn_raw_buffer_store_b128(buf[i], r_dst, vl + dl + (i) * 1024u, 0, 17)
#define FL_() __builtin_amdgcn_raw_buffer_store_b32(call, r_dst, v0 + prev, 0, 17)
        if (bk + (u32)TAILN >= total) {
          ST_(0); ST_(1); ST_(2); ST_(3); WAIT_VM(4); FL_();
        } else {
        WAIT_VM(3); ST_(0); WAIT_VM(2); ST_(1); WAIT_VM(2); FL_(); ST_(2); WAIT_VM(3); ST_(3);
        }
        prev = fl;
      }
      STAMP(1 + wid);
    }
    BAR();
    if (tid == 0) st_agt(my_ctr, rfl(callv) + 1u);
    return;
  }

  const u32 kidx = li - NPUSH;
  const u32 ga = grp >> 1, gb = grp & 1u;
  const bool loader = wid >= 4u;
  const u32 lw = wid & 3u;
  u32 call = 0;
  if (wid == 4u) {
    call = rfl(ld_agt(my_ctr)) + 1u;
    if (lane == 0) ctrl[16] = call;
    WAIT_LGKM0();
    STAMP(0);
  }
  const rsrc_t r_me = mk_rsrc(me, WS_USED);

  u32 kind, rt, ct;
  u32 fin = 0;
  bool have = kidx < 24u;
  if (have) { rt = ga + 4u * (kidx >> 3); ct = 2u * (kidx & 7u) + gb; kind = rt < 4u ? 0u : 1u; }
  else { rt = 0; ct = 0; kind = 0; }

  for (int round = 0; round < 2; ++round) {
    if (round == 1) {
      if (wid == 4u) {
        u32 t = 0;
        if (lane == 0) t = __hip_atomic_fetch_add((GLBP u32*)(me + F_TICKET + grp * 128u), 1u, __ATOMIC_RELAXED, __HIP_MEMORY_SCOPE_AGENT);
        t = rfl(t) - (call - 1u) * (u32)NGEMM;
        if (lane == 0) { ctrl[0] = t; st_agt(my_ctr, call); }
        if (lane >= 1u && lane < 16u) ctrl[lane] = 0;
        WAIT_LGKM0();
      }
      BAR();
      const u32 t = rfl(ctrl[0]);
      if (t >= 16u) return;
      have = true; fin = grp * 16u + t;
      kind = t < 8u ? 2u : 3u; rt = 12u + ga; ct = 2u * (t & 7u) + gb;
    }
    if (!have) continue;

    const u32 grow = row_base(q, rt);
    const u32 gcol = ((q >> 2) ^ (kind == 3u ? 1u : 0u)) * 2048u + ct * 128u;
    const u32 skip = kind == 1u ? 2u + (rt & 3u) : 99u;
    const u32 remote = kind == 3u ? 0u : kind == 1u ? 63u & ~(1u << skip) : 63u;
    const u32 pvo = ((remote >> (lane >> 3)) & 1u) ? F_LINK + (((lane >> 3) + 2u) * 64u + ((lane & 4u) ? 16u + ct : rt)) * 128u + (lane & 3u) * 16u : OOB;
    const u32 lbase = (u32)(size_t)lds + L_CTRL;
    const u32 pace = (K->pace[rt * 4u + (ct >> 2)] >> ((ct & 3u) * 8u)) & 255u;

    if (loader) {
      const u32 vx = (lw * 8u + (lane >> 3)) * 1024u + (((lane & 7u) ^ (((lw & 1u) * 4u + (lane >> 4)) & 7u)) << 4);
      const u32 vw = (lw * 4u + ((lane >> 2) & 3u)) * 8192u + (32u * (lane >> 4) + 8u * (lane & 3u)) * 2u;
      const u32 vs = lw * 1024u + lane * 16u;
      const u32 rot = ROT ? (2u * (rt >> 2) + (ct >> 1)) % 6u : 0u;
      if (wid == 4u) {
        if (remote && !(pace & 32u))
          wave_wait<1>(r_me, F_LINK + (2u * 64u + (pace & 31u)) * 128u, lane == 0, lane, call, tmo);
        lds_wr64(lbase + 96u, 0);
        lds_wr64(lbase + 104u, 0);
      }
      BAR();
      u32 rem = remote, rdy = 0, snap = 0, sx = 0, sw = 0;
      auto pick = [&]() -> bool {
        const u32 av = rdy & rem;
        if (!av) return false;
        u32 n = (u32)__builtin_ctz(((av >> rot) | (av << (6u - rot))) & 63u) + rot;
        if (n >= 6u) n -= 6u;
        rem &= ~(1u << n);
        sx = OFF_XS + (n * 16u + rt) * CH; sw = OFF_WS + (n * 16u + ct) * CH;
        return true;
      };
      auto choose = [&]() -> bool {
        if (!rem) return false;
        rdy |= (u32)lds_rd64(lbase + 96u);
        return pick();
      };
      auto issue = [&](u32 slot) {
        u32 dofs = slot * STAGE + lw * 1024u;
        asm volatile("" : "+s"(dofs));
        LDSP char* dst = lds + dofs;
#pragma unroll
        for (u32 i = 0; i < 4; ++i) {
          __builtin_amdgcn_raw_ptr_buffer_load_lds(r_me, dst + i * 4096u, 16, vs, sx + i * 4096u, 0, RAUX & 31);
          __builtin_amdgcn_raw_ptr_buffer_load_lds(r_me, dst + 16384u + i * 4096u, 16, vs, sw + i * 4096u, 0, RAUX & 31);
        }
        sx += 16384u; sw += 16384u;
      };
      auto shoot = [&]() {
        if (wid == 4u && rem && !(rem & rdy)) { __builtin_amdgcn_raw_ptr_buffer_load_lds(r_me, lds + L_SNAP, 16, pvo, 0, 0, 17); snap = 1; }
      };
      auto publish = [&]() {
        if (snap) { lds_wr64(lbase + 96u, seg_mask(__ballot(all_ge(lds_rd128(lbase + 1024u + lane * 16u), call)), lane)); snap = 0; }
      };
      {
        const rsrc_t r_x = mk_rsrc(gx, X_BYTES), r_w = mk_rsrc(gw, W_BYTES);
#pragma unroll 1
        for (u32 s8 = 0; s8 < 8; ++s8) {
          if (s8 == 4u) { WAIT_VM(24); BAR(); }
          if (s8 >= 4u) { WAIT_VM(16); BAR(); }
          if (s8 == 5u) shoot();
          LDSP char* dst = lds + (s8 & 3u) * STAGE + lw * 1024u;
          const u32 ox = grow * 1024u + s8 * 128u, ow = s8 * 64u * 8192u + gcol * 2u;
#pragma unroll
          for (u32 i = 0; i < 4; ++i) {
            __builtin_amdgcn_raw_ptr_buffer_load_lds(r_x, dst + i * 4096u, 16, vx, ox + i * 32768u, 0, 0);
            __builtin_amdgcn_raw_ptr_buffer_load_lds(r_w, dst + 16384u + i * 4096u, 16, vw, ow + i * 131072u, 0, 0);
          }
        }
      }
      for (;;) {
        for (;;) {
          const bool pre = pick();
          WAIT_VM(16); publish(); BAR();
          if (pre || choose()) {
            issue(0); WAIT_VM(16); BAR();
            shoot(); issue(1); WAIT_VM(16); BAR();
          } else {
            shoot(); WAIT_VM(8); BAR();
            WAIT_VM(0); publish(); BAR();
            if (!choose()) break;
            shoot();
#pragma unroll 1
            for (u32 i = 0; i < 2; ++i) issue(i);
          }
          issue(2); WAIT_VM(16); BAR();
          issue(3);
        }
        if (wid == 4u) lds_wr64(lbase + 104u, rem ? 1u : 2u);
        BAR();
        if (!rem) break;
        if (wid == 4u) {
          if (rem == remote) STAMP(round * 4 + 1);
          const u64 t0 = __builtin_amdgcn_s_memrealtime();
          const u32 want = min((u32)MINSEG, (u32)__builtin_popcount(rem));
          u64 bal;
          for (;;) {
            const u32x4 v = __builtin_amdgcn_raw_buffer_load_b128(r_me, lane == 63u ? F_TMO : pvo, 0, 17);
            bal = __ballot(all_ge(v, call));
            if ((u32)__builtin_popcount(seg_mask(bal, lane) & rem) >= want) break;
            bal = ~0ull;
            if (__builtin_amdgcn_readlane(v.x, 63) == call) break;
            if (__builtin_amdgcn_s_memrealtime() - t0 > DEADLINE) { st_agt(tmo, call); break; }
            __builtin_amdgcn_s_sleep(2);
          }
          lds_wr64(lbase + 96u, seg_mask(bal, lane));
          if (rem == remote) STAMP(round * 4 + 2);
        }
        BAR();
        if (wid == 4u) lds_wr64(lbase + 104u, 0);
        choose();
#pragma unroll 1
        for (u32 i = 0; i < 4; ++i) issue(i);
        WAIT_VM(24); BAR();
      }
      if (kind >= 2u) {
        const u32 callw = rfl(ctrl[16]);
        const u64 t0 = __builtin_amdgcn_s_memrealtime();
        u32 o0 = F_AMAX + (lw * 64u + lane) * 16u, o1 = o0 + 4096u, m = 0, bad = 0;
        for (;;) {
          const u32x4 g0 = __builtin_amdgcn_raw_buffer_load_b128(r_me, o0, 0, 17);
          const u32x4 g1 = __builtin_amdgcn_raw_buffer_load_b128(r_me, o1, 0, 17);
          const u32 tv = __builtin_amdgcn_raw_buffer_load_b32(r_me, lane == 0u ? F_TMO : OOB, 0, 17);
          if (g0.y == callw && g0.w == callw) { m = max(m, max(g0.x, g0.z)); o0 = OOB; }
          if (g1.y == callw && g1.w == callw) { m = max(m, max(g1.x, g1.z)); o1 = OOB; }
          if (rfl(tv) == callw) { bad = 1; break; }
          const u32 missing = (u32)__builtin_popcountll(__ballot((o0 & o1) != OOB));
          if (missing == 0u) break;
          if (__builtin_amdgcn_s_memrealtime() - t0 > DEADLINE) { st_agt(tmo, callw); bad = 1; break; }
          if (missing > (u32)BACKOFF2) __builtin_amdgcn_s_sleep(40);
          else if (missing > (u32)BACKOFF) __builtin_amdgcn_s_sleep(BSLEEP);
          else {
            if (missing <= (u32)SPEC_MISSING) {
              const u32 pm = wave_max(m);
              if (lane == 0) *(volatile LDSP u32x2*)(lds + L_CTRL + 32 + lw * 8) = u32x2{pm, 1u};
            }
            __builtin_amdgcn_s_sleep(1);
          }
        }
        m = wave_max(m);
        if (lane == 0) *(volatile LDSP u32x2*)(lds + L_CTRL + 32 + lw * 8) = u32x2{m, 2u + bad};
        if (wid == 4u) STAMP(9);
      }
    } else {
      const u32 wn = wid & 1u, wm = wid >> 1;
      const u32 hh = lane >> 5, r31 = lane & 31u;
      const u32 z = (lane >> 4) & 1u, tq = (lane >> 2) & 3u, tp = lane & 3u;
      const u32 aw = 16384u + hh * 2048u + wn * 512u + tq * 64u + (16u * (tp & 1u) + 8u * z + 4u * (tp >> 1)) * 2u;
      const u32 axb = (wm * 64u + r31) * 128u, sw7 = (r31 >> 1) & 7u;
      u32 ax[4];
#pragma unroll
      for (u32 s = 0; s < 4; ++s) ax[s] = axb + (((2u * s + hh) ^ sw7) << 4);
      f32x16 acc[2][2] = {};
      s16x8 fa[2][2] = {}, fb[2][2] = {};
      s16x4 alo = {}, ahi = {};
      auto tr = [&](u32 off) { return __builtin_amdgcn_ds_read_tr16_b64_v4i16((LDSP s16x4*)(lds + off)); };
      auto rd0 = [&](int buf, u32 so, int ks) {
        const s16x4 lo = tr(so + aw + ks * 4096), hi = tr(so + aw + ks * 4096 + 1024);
        fa[buf][0] = __builtin_shufflevector(lo, hi, 0, 1, 2, 3, 4, 5, 6, 7);
        fb[buf][0] = *(LDSP s16x8*)(lds + so + ax[ks]);
      };
      auto rd1 = [&](int buf, u32 so, int ks) { fb[buf][1] = *(LDSP s16x8*)(lds + so + ax[ks] + 4096); alo = tr(so + aw + ks * 4096 + 256); };
      auto rd2 = [&](int buf, u32 so, int ks) { ahi = tr(so + aw + ks * 4096 + 256 + 1024); fa[buf][1] = __builtin_shufflevector(alo, ahi, 0, 1, 2, 3, 4, 5, 6, 7); };
      auto mm = [&](int buf, int fn, int fm) {
        acc[fn][fm] = __builtin_amdgcn_mfma_f32_32x32x16_bf16(__builtin_bit_cast(bf16x8, fa[buf][fn]), __builtin_bit_cast(bf16x8, fb[buf][fm]), acc[fn][fm], 0, 0, 0);
        SCHED();
      };
      u32 statv = 0;
      auto kstep = [&](int bc, u32 so, int ks, int sync) {
        mm(bc, 0, 0);
        if (sync) { WAIT_LGKM0(); BAR(); asm volatile("" ::: "memory"); }
        if (sync == 2) statv = ctrl[26];
        rd0(bc ^ 1, so, ks);
        SCHED();
        mm(bc, 0, 1);
        rd1(bc ^ 1, so, ks);
        SCHED();
        mm(bc, 1, 0);
        rd2(bc ^ 1, so, ks);
        SCHED();
        mm(bc, 1, 1);
      };
      BAR();
      __builtin_amdgcn_s_waitcnt(0x0F70);
      for (;;) {
        BAR();
        asm volatile("" ::: "memory");
        rd0(0, 0, 0); rd1(0, 0, 0); rd2(0, 0, 0); SCHED();
        u32 status;
        do {
#pragma unroll
          for (int st = 0; st < 4; ++st) {
            kstep(0, st * STAGE, 1, 0);
            kstep(1, st * STAGE, 2, 0);
            kstep(0, st * STAGE, 3, 0);
            kstep(1, ((st + 1) & 3) * STAGE, 0, st == 3 ? 2 : 1);
          }
          status = rfl(statv);
        } while (status == 0u);
        if (status == 2u) break;
        BAR();
      }
      const u32 vy = wid * 8192u + lane * 16u;
      if (kind < 2u) {
        const u32 ti = kind == 0u ? rt * 16u + ct : 64u + (rt & 3u) * 32u + ((rt - 4u) >> 2) * 16u + ct;
        const u32 callc = ctrl[16];
        const bool direct = kind == 0u && (pace >> 6) != 0u;
        char* const ybase = K->peer[direct ? 1u : 0u];
        const rsrc_t r_o = mk_rsrc(ybase, WS_USED);
        const u32 so = (direct ? OFF_YIN : OFF_YOUT) + ti * YT;
#pragma unroll
        for (int f = 0; f < 4; ++f)
#pragma unroll
          for (int hf = 0; hf < 2; ++hf) {
            u32x4 v;
#pragma unroll
            for (int e = 0; e < 4; ++e) {
              const f32x2 p = {acc[f >> 1][f & 1][hf * 8 + e * 2], acc[f >> 1][f & 1][hf * 8 + e * 2 + 1]};
              v[e] = __builtin_bit_cast(u32, __builtin_convertvector(p, bf16x2));
            }
            if (direct && (pace >> 6) == 2u) WAIT_VM(2);
            __builtin_amdgcn_raw_buffer_store_b128(v, r_o, vy + so + (f * 2 + hf) * 1024, 0, 17);
          }
        WAIT_VM(0);
        if (lane < (direct ? 2u : 1u)) st_sys(ybase + (direct ? F_LINK + (64u + ti) * 128u + wid * 8u + lane * 4u : F_YRDY + ti * 16u + wid * 4u), callc);
        if (wid == 0) STAMP(3);
      } else {
        const u32 callc = rfl(ctrl[16]);
        char* const gdst = K->peer[lane & 7u] + F_AMAX + (q * 128u + fin) * 8u;
        const u32 d0 = (q ^ (((q >> 2) ^ 1u) * 4u + (ga >> 1) * 2u)) & 7u, d1 = d0 ^ 1u;
        const u32 ty = (ga & 1u) * 16u + ct;
        const u32 so0 = OFF_YIN + (kind == 2u ? ga * 16u + ct : d0 * 32u - 64u + ty) * YT, so1 = OFF_YIN + (d1 * 32u - 64u + ty) * YT;
        const u32 fo0 = F_LINK + (kind == 2u ? 64u + ga * 16u + ct : d0 * 64u + 32u + ty) * 128u, fo1 = F_LINK + (d1 * 64u + 32u + ty) * 128u;
        const u32 po = lane == 63u ? F_TMO : lane < 4u ? ((lane & 2u) ? fo1 : fo0) + wid * 8u + (lane & 1u) * 4u : OOB;
        const u64 t0 = __builtin_amdgcn_s_memrealtime();
        auto poll = [&](u32 todo) {
          u32 go;
          for (;;) {
            const u32 pv = __builtin_amdgcn_raw_buffer_load_b32(r_me, po, 0, 17);
            const u32 okm = (u32)__ballot((int)(pv - callc) >= 0);
            go = (((okm & 3u) == 3u ? 1u : 0u) | ((okm & 12u) == 12u ? 2u : 0u)) & todo;
            if (go) break;
            go = todo;
            if (__builtin_amdgcn_readlane(pv, 63) == callc) break;
            if (__builtin_amdgcn_s_memrealtime() - t0 > DEADLINE) { st_agt(tmo, callc); break; }
            __builtin_amdgcn_s_sleep(1);
          }
          asm volatile("" ::: "memory");
          return go;
        };
        auto fetch = [&](u32x4 (&v)[8], u32 so) {
#pragma unroll
          for (int j = 0; j < 8; ++j) v[j] = __builtin_amdgcn_raw_buffer_load_b128(r_me, vy + j * 1024, so, RAUX & 31);
        };
        auto add = [&](const u32x4 (&v)[8]) {
#pragma unroll
          for (int j = 0; j < 8; ++j)
#pragma unroll
            for (int e = 0; e < 4; ++e) {
              acc[j >> 2][(j >> 1) & 1][(j & 1) * 8 + e * 2] += __builtin_bit_cast(float, v[j][e] << 16);
              acc[j >> 2][(j >> 1) & 1][(j & 1) * 8 + e * 2 + 1] += __builtin_bit_cast(float, v[j][e] & 0xFFFF0000u);
            }
        };
        u32x4 va[8], vb[8];
        const u32 go = poll(kind == 2u ? 1u : 3u);
        fetch(va, (go & 1u) ? so0 : so1);
        fetch(vb, go == 3u ? so1 : OOB);
        add(va);
        if (kind == 3u && go != 3u) { poll(go ^ 3u); fetch(vb, (go & 1u) ? so1 : so0); }
        if (kind == 3u) add(vb);
        if (wid == 0) STAMP(7);
        float mx = 0.f;
#pragma unroll
        for (int f = 0; f < 4; ++f)
#pragma unroll
          for (int e = 0; e < 16; ++e) mx = fmaxf(mx, acc[f >> 1][f & 1][e]);
        const u32 wm_ = wave_max(__builtin_bit_cast(u32, mx));
        u32 old = 0;
        if (lane == 0) {
          ctrl[4 + wid] = wm_;
          old = __hip_atomic_fetch_add((LDSP u32*)(lds + L_CTRL) + 1, 1u, __ATOMIC_RELAXED, __HIP_MEMORY_SCOPE_WORKGROUP);
        }
        if (rfl(old) == 3u) {
          const u32x4 m4 = *(volatile LDSP u32x4*)(lds + L_CTRL + 16);
          const u32 m = max(max(m4.x, m4.y), max(m4.z, m4.w));
          if (lane < 8u) __hip_atomic_store((GLBP u64*)gdst, ((u64)callc << 32) | m, __ATOMIC_RELAXED, __HIP_MEMORY_SCOPE_SYSTEM);
          STAMP(8);
        }
        const rsrc_t r_out = mk_rsrc(K->out, OUT_BYTES);
        const u32 vo = (ga * 128u + wm * 64u + r31) * 4096u + gcol + wn * 64u + hh * 16u;
        u32 used = ~0u;
        for (;;) {
          const u32x4 a4 = *(volatile LDSP u32x4*)(lds + L_CTRL + 32), b4 = *(volatile LDSP u32x4*)(lds + L_CTRL + 48);
          const u32 lo = rfl(min(min(a4.y, a4.w), min(b4.y, b4.w))), hi = rfl(max(max(a4.y, a4.w), max(b4.y, b4.w)));
          if (lo == 0u || (lo == 1u && used != ~0u)) { __builtin_amdgcn_s_sleep(1); continue; }
          const u32 am = rfl(max(max(max(a4.x, a4.z), max(b4.x, b4.z)), wm_));
          const bool bad = hi == 3u;
          if (am != used || bad) {
            if (used != ~0u) WAIT_VM(0);
            used = am;
            const float amax = __builtin_bit_cast(float, am);
            const float scale = amax / 127.0f;
            const float inv = amax > 0.f ? 1.0f / scale : 0.f;
#pragma unroll
            for (int f = 0; f < 4; ++f) {
              u32x4 v;
#pragma unroll
              for (int e = 0; e < 4; ++e) {
                u32 pk = 0;
#pragma unroll
                for (int b = 0; b < 4; ++b) {
                  const float y = __builtin_amdgcn_fmed3f(__builtin_rintf(acc[f >> 1][f & 1][e * 4 + b] * inv), 0.f, 127.f);
                  pk = __builtin_amdgcn_cvt_pk_u8_f32(y, b, pk);
                }
                v[e] = bad ? 0x80808080u : pk;
              }
              __builtin_amdgcn_raw_buffer_store_b128(v, r_out, vo + (f & 1) * 32 * 4096 + (f >> 1) * 32, 0, OUT_AUX);
            }
          }
          if (lo >= 2u) break;
        }
        if (wid == 0) STAMP(10);
      }
    }
  }
}

namespace {
struct Sched { const char* cross; const char* sib; };
const Sched kSched[] = {
    {"T0 w0 T1 w1 T2 w2 T3 w3 w4 X0 w5 X1 w6 X2 w7 X3 w8 X4 w9 X5 w10 X6 w11 X7 w12 w13 w14 w15 O0 O1 O2 O3",
     "T0 w0 T1 w1 T2 w2 T3 w3 w4 X0 w5 X1 w6 X2 w7 X3 w8 X4 w9 X5 w10 X6 w11 X7 w12 w13 w14 w15 O0 O1 O2 O3 a12 a13 b12 b13"},
    {"T0 w0 T1 w1 T2 w2 T3 w3 w4 X0 w5 X1 w6 X2 w7 X3 w8 X4 w9 X5 w10 X6 w11 X7 w12 w13 w14 w15 O0 O1 O2 O3",
     "T0 w0 T1 w1 T2 w2 T3 w3 w4 X0 w5 X1 w6 X2 w7 X3 w8 X4 w9 X5 w10 X6 f10 w11 X7 f11 w12 w13 w14 w15 O0 O1 O2 O3 a12 a13 b12 b13"},
    {"T0 w0 T1 w1 T2 w2 T3 w3 w4 X0 w5 X1 w6 X2 w7 X3 X4 w8 w12 X5 w9 w13 X6 w15 X7 w10 w11 w14 O0 O1 O2 O3",
     "T0 w0 T1 w1 T2 w2 T3 w3 w4 X0 w5 X1 w6 X2 w7 X3 X4 w8 X5 w9 X6 X7 w10 f12 w11 f13 f15 w12 w13 w14 w15 O0 O1 O2 O3 a12 a13 b12 b13"},
    {"T0 w0 T1 w1 T2 w2 T3 w3 w4 X0 w5 X1 w6 X2 w7 X3 X4 w8 w11 w12 X5 X6 w9 w14 w15 X7 w10 w13 O0 O1 O2 O3",
     "T0 w0 T1 w1 T2 w2 T3 w3 w4 X0 w5 X1 w6 X2 w7 X3 X4 w8 X5 X6 w9 X7 w10 f11 f12 f14 f15 w11 w12 w13 w14 w15 O0 O1 O2 O3 a12 a14 b12 b14"},
    {"w9 w3 w0 T2 T0 T1 X5 w1 T3 w2 w10 w4 w5 w7 X4 X0 w6 X6 w8 X3 w11 X2 X7 X1 O1 w15 w12 w13 w14 O0 O3 O2",
     "T1 w0 X1 w2 T0 T3 w1 T2 w4 w3 X6 w5 X2 X3 X4 f9 w6 w7 X5 X7 w8 w9 X0 f11 w10 f12 f15 w11 w12 w13 w14 w15 O0 O1 O2 O3 a13 a14 b13 b14"},
    {"w0 T1 T2 w1 X7 T0 w3 T3 w2 w4 w6 w5 X4 X0 X1 X2 w7 w9 w10 X6 w8 X5 X3 w11 w12 w15 w13 O2 w14 O0 O3 O1",
     "w2 T3 w0 T0 w1 T1 T2 X1 w3 w5 X3 X6 X0 w4 f5 w6 X7 w7 X2 w8 X5 f10 X4 w9 w10 f13 w11 f15 w12 w13 w14 w15 O0 O1 O2 O3 a12 a14 b12 b14"},
    {"T0 w1 w3 T2 w0 T1 w8 T3 w2 w4 w5 X2 w7 w15 w6 X5 X4 w11 X3 X0 w14 X6 w9 X1 X7 w12 w10 O2 O1 w13 O0 O3",
     "T0 X5 T1 w4 w2 w6 w0 w1 w3 w5 T2 X7 T3 X3 X4 X2 w9 X6 w10 w7 X0 X1 w8 f11 f12 f15 f14 w11 w12 w13 w14 w15 O0 O1 O2 O3 a12 a14 b12 b14"},
};

int parse_order(const char* p, int* v) {
  int n = 0;
  while (*p && n < 48) {
    const char c = *p++;
    int k = 0;
    while (*p >= '0' && *p <= '9') k = k * 10 + (*p++ - '0');
    while (*p == ' ') ++p;
    v[n++] = (c == 'T' ? 0 : c == 'X' ? 4 : c == 'O' ? 12 : c == 'w' ? 16 : c == 'f' ? 32 : c == 'a' ? 48 : 64) + k;
  }
  return n;
}

void build_tables(Params& P, int rank) {
  int cs[48], ss[48];
  const int nc = parse_order(kSched[SCHED_ID].cross, cs), ns = parse_order(kSched[SCHED_ID].sib, ss);
  u32 split = 0, relay = 0;
  for (int i = 0; i < ns; ++i) { if ((ss[i] >> 4) == 2) split |= 1u << (ss[i] & 15); if ((ss[i] >> 4) == 3) relay |= 1u << (ss[i] & 15); }
  const double C = 1.125;
  double ct[4][32] = {}, sb[48] = {}, first[16][16], done[16][16];
  for (int l = 0; l < 4; ++l) {
    double t = 0;
    for (int i = 0; i < nc; ++i) {
      const int it = cs[i];
      if (it >= 4 && it < 12 && (it & 3) == l) continue;
      ct[l][it] = t += (it >= 16 && ((split >> (it - 16)) & 1u)) ? C / 2 : C;
    }
  }
  double t = 0;
  for (int i = 0; i < ns; ++i) {
    const int it = ss[i];
    if ((it >> 4) == 2) for (int l = 0; l < 4; ++l) t = std::max(t, ct[l][it - 16] + 2.2);
    t += C;
    if (it < 48) sb[it] = t;
  }
  for (int r = 0; r < 16; ++r)
    for (int c = 0; c < 16; ++c) {
      double sg[6]; int n = 0;
      sg[n++] = std::max(sb[r], sb[16 + c]); sg[n++] = sg[0];
      for (int l = 0; l < 4; ++l)
        if (!(r >= 4 && r < 12 && (r & 3) == l)) sg[n++] = std::max(std::max(ct[l][r], ct[l][16 + c]), sb[32 + c]);
      std::sort(sg, sg + n);
      double e = first[r][c] = sg[1];
      for (int i = 0; i < n; ++i) e = std::max(e, sg[i] + (i >= 2 ? 0.9 : 0.0)) + 1.25;
      done[r][c] = e + 1e-3 * (c * 16 + r);
    }
  unsigned char pace[16][16];
  for (int r = 0; r < 16; ++r)
    for (int c = 0; c < 16; ++c) {
      int code = 32;
      for (int i = 0; i < ns; ++i) if (ss[i] < 32 && sb[ss[i]] - C <= first[r][c] - LEAD_US) code = ss[i];
      int earlier = 0;
      for (int k = 0; k < 64 && r < 4; ++k) earlier += !((relay >> (k & 15)) & 1u) && done[k >> 4][k & 15] < done[r][c];
      const int direct = r < 4 && !((relay >> c) & 1u) && earlier < DIRECT_N ? (earlier < 6 ? 1 : 2) : 0;
      pace[r][c] = (unsigned char)(code | direct << 6);
    }
  memcpy(P.pace, pace, sizeof pace);
  unsigned short tab[7][96] = {};
  memset(P.nent, 0, sizeof P.nent);
  memset(P.pad2, 0, sizeof P.pad2);
  P.pad = 0;
  for (int d = 1; d < 8; ++d) {
    int m = 0;
    auto put = [&](u32 kind, int pl) { if (m < 96) tab[d - 1][m++] = (unsigned short)(kind << 8 | (u32)pl); };
    const int tdst = ((rank ^ d) >> 1) & 1;
    for (int i = 0; i < (d >= 4 ? nc : d >= 2 ? ns : 0); ++i) {
      const int it = d >= 4 ? cs[i] : ss[i], k = it & 15;
      if (it < 32) {
        if (d >= 4 && it >= 4 && it < 12 && (it & 3) == d - 4) continue;
        for (int hf = 0; hf < 2; ++hf)
          if (!(d >= 4 && it >= 16 && ((split >> k) & 1u)) || hf == tdst) put(it < 16 ? E_X : E_W, k * 2 + hf);
      } else if (it < 48) { put(E_FW, k * 2); put(E_FW, k * 2 + 1); }
      else for (int r = (d ^ (it >= 64)) & 1; r < 4; r += 2) put(it >= 64 ? E_R2 : E_R1, r * 16 + k);
    }
    const int nt = d == 1 ? 64 : d >= 4 ? 32 : 0;
    int ord[64], no = 0;
    auto row = [&](int tl) { return d == 1 ? tl >> 4 : d + 4 * (tl >> 4); };
    for (int tl = 0; tl < nt; ++tl)
      if (d != 1 || !(((relay >> (tl & 15)) & 1u) || (pace[tl >> 4][tl & 15] >> 6))) ord[no++] = tl;
    std::sort(ord, ord + no, [&](int x, int y) { return done[row(x)][x & 15] < done[row(y)][y & 15]; });
    for (int i = 0; i < no; ++i) put(E_Y, d == 1 ? ord[i] : d * 32 - 64 + ord[i]);
    P.nent[d - 1] = (u32)m;
  }
  memcpy(P.tab, tab, sizeof tab);
}
}

extern "C" void kernel_launch(int rank, int nranks, ncclComm_t comm, hipStream_t stream,
                              void* const* d_in, const int* in_sizes, int n_in,
                              void* d_out, int out_size, const int* in_dtypes, int out_dtype,
                              void* const* d_out_all, void* const* d_ws_all, size_t ws_bytes) {
  (void)comm; (void)in_dtypes; (void)out_dtype; (void)d_out_all;
  if (nranks != NR || (unsigned)rank >= (unsigned)NR || n_in < 2 || ws_bytes < WS_USED ||
      (size_t)in_sizes[0] * 2 != X_BYTES || (size_t)in_sizes[1] * 2 != W_BYTES || (u32)out_size != OUT_BYTES) {
    fprintf(stderr, "kernel_launch: unexpected problem shape\n");
    return;
  }
  Params P;
  P.x = (const char*)d_in[0];
  P.w = (const char*)d_in[1];
  P.out = (char*)d_out;
  for (int p = 0; p < NR; ++p) P.peer[p] = (char*)d_ws_all[rank ^ p];
  P.rank = (u32)rank;
  build_tables(P, rank);
  hipError_t e = hipFuncSetAttribute((const void*)fused_kernel, hipFuncAttributeMaxDynamicSharedMemorySize, (int)LDS_BYTES);
  if (e != hipSuccess) { fprintf(stderr, "hipFuncSetAttribute: %s\n", hipGetErrorString(e)); return; }
  fused_kernel<<<dim3(256), dim3(512), LDS_BYTES, stream>>>(P);
  e = hipGetLastError();
  if (e != hipSuccess) fprintf(stderr, "launch: %s\n", hipGetErrorString(e));
}
```

```cpp
#include <hip/hip_runtime.h>
#include <rccl/rccl.h>
#include <cstdio>
#include <cstring>
#include <algorithm>

typedef unsigned int u32;
typedef unsigned long long u64;
typedef __attribute__((ext_vector_type(4))) unsigned int u32x4;
typedef __attribute__((ext_vector_type(2))) unsigned int u32x2;
typedef __attribute__((ext_vector_type(4))) short s16x4;
typedef __attribute__((ext_vector_type(8))) short s16x8;
typedef __attribute__((ext_vector_type(8))) __bf16 bf16x8;
typedef __attribute__((ext_vector_type(2))) __bf16 bf16x2;
typedef __attribute__((ext_vector_type(2))) float f32x2;
typedef __attribute__((ext_vector_type(16))) float f32x16;
#define LDSP __attribute__((address_space(3)))
#define GLBP __attribute__((address_space(1)))
#define CSTP __attribute__((address_space(4)))
typedef __amdgpu_buffer_rsrc_t rsrc_t;

namespace {

constexpr int NR = 8;
constexpr u32 X_BYTES = 4096u * 512u * 2u;
constexpr u32 W_BYTES = 512u * 4096u * 2u;
constexpr u32 OUT_BYTES = 512u * 4096u;
constexpr u32 CH = 65536;
constexpr u32 YT = 32768;
constexpr u32 F_CALL = 0;
constexpr u32 F_TMO = 16384;
constexpr u32 F_TICKET = F_TMO + 1024;
constexpr u32 F_UNIT = F_TICKET + 1024;
constexpr u32 F_AMAX = 20480;
constexpr u32 F_YRDY = 28672;
constexpr u32 F_LINK = 32768;
constexpr u32 F_REL = F_LINK + 8 * 64 * 128;
constexpr u32 F_DBG = F_REL + 2 * 64 * 128;
constexpr u32 OFF_XS = 1u << 20;
constexpr u32 OFF_WS = OFF_XS + 6 * 16 * CH;
constexpr u32 OFF_YOUT = OFF_WS + 6 * 16 * CH;
constexpr u32 OFF_YIN = OFF_YOUT + 192 * YT;
constexpr u32 OFF_REL = OFF_YIN + 192 * YT;
constexpr u32 WS_USED = OFF_REL + 128 * YT;
constexpr u32 OOB = 0x80000000u;

constexpr int NSLOT = 4;
constexpr u32 STAGE = 32768;
constexpr u32 L_CTRL = NSLOT * STAGE;
constexpr u32 L_SNAP = L_CTRL + 1024;
constexpr u32 LDS_BYTES = L_SNAP + 1024;
constexpr u64 DEADLINE = 50000000ull;
constexpr int NPUSH = 7;
constexpr int NGEMM = 32 - NPUSH;
#define NSTO 2u
#define NLD 3u
constexpr u32 NPWG = 16u / NSTO;
static_assert(NSTO + NLD <= 8 && (NSTO == 2 || NSTO == 4), "pusher wave roles");
#define RAUX 17
#define OUT_AUX 16
#define ROT 1
#define BACKOFF 12
#define SPEC_MISSING 36
#define BACKOFF2 36
#define BSLEEP 10
#define GLAY 0
#define SCHED_ID 7
#define LEAD_US 5.0
#define DIRECT_N 12
#define RA 2
#define OUT_LDS 1
#define FSLEEP 1
#define DIRECT_LAST 0
#define STAGGER 4
#define TLB_WARM 0
#define W4MASK 0u
#define W2MASK 0u
#define MINSEG 2

enum : u32 { E_X = 0, E_W = 1, E_Y = 2, E_FW = 3, E_R1 = 4, E_R2 = 5 };
struct Params {
  const char* x;
  const char* w;
  char* out;
  u32 rank, pad;
  char* peer[NR];
  u32 nent[8];
  u32 pad2[16];
  u32 tab[7][48];
  u32 pace[64];
};

__device__ __forceinline__ rsrc_t mk_rsrc(const void* p, u32 bytes) {
  return __builtin_amdgcn_make_buffer_rsrc((void*)p, 0, bytes, 0x00020000);
}
__device__ __forceinline__ u32 rfl(u32 v) { return __builtin_amdgcn_readfirstlane(v); }
__device__ __forceinline__ u32 ld_sys(const char* p) { return __hip_atomic_load((const GLBP u32*)p, __ATOMIC_RELAXED, __HIP_MEMORY_SCOPE_SYSTEM); }
__device__ __forceinline__ u32 ld_agt(const char* p) { return __hip_atomic_load((const GLBP u32*)p, __ATOMIC_RELAXED, __HIP_MEMORY_SCOPE_AGENT); }
__device__ __forceinline__ void st_sys(char* p, u32 v) { __hip_atomic_store((GLBP u32*)p, v, __ATOMIC_RELAXED, __HIP_MEMORY_SCOPE_SYSTEM); }
__device__ __forceinline__ void st_agt(char* p, u32 v) { __hip_atomic_store((GLBP u32*)p, v, __ATOMIC_RELAXED, __HIP_MEMORY_SCOPE_AGENT); }
#define WAIT_VM(n) asm volatile("s_waitcnt vmcnt(" #n ")" ::: "memory")
#define WAIT_LGKM0() asm volatile("s_waitcnt lgkmcnt(0)" ::: "memory")
#define BAR() __builtin_amdgcn_s_barrier()
#define SCHED() __builtin_amdgcn_sched_barrier(0)

__device__ __forceinline__ u32 row_base(u32 q, u32 r) {
  if (r < 4) return 512u * (q ^ 1u) + 128u * r;
  if (r < 12) return 512u * (q ^ (4u + (r & 3u))) + 256u * ((q >> 1) & 1u) + 128u * ((r - 4u) >> 2);
  return 512u * q + 128u * (r - 12u);
}
__device__ __forceinline__ u32 k_half(u32 q, u32 p) { return (q ^ ((q >> 1) & p)) & 1u; }

__device__ __forceinline__ u64 lds_rd64(u32 addr) {
  u32x2 v;
  asm volatile("ds_read_b64 %0, %1\n\ts_waitcnt lgkmcnt(0)" : "=&v"(v) : "v"(addr) : "memory");
  return ((u64)rfl(v.y) << 32) | rfl(v.x);
}
__device__ __forceinline__ void lds_wr64(u32 addr, u64 x) {
  const u32x2 v = {(u32)x, (u32)(x >> 32)};
  asm volatile("ds_write_b64 %0, %1\n\ts_waitcnt lgkmcnt(0)" :: "v"(addr), "v"(v) : "memory");
}
__device__ __forceinline__ u32x4 lds_rd128(u32 addr) {
  u32x4 v;
  asm volatile("ds_read_b128 %0, %1\n\ts_waitcnt lgkmcnt(0)" : "=&v"(v) : "v"(addr) : "memory");
  return v;
}
__device__ __forceinline__ u32 seg_mask(u64 b, u32 lane) { return (u32)__ballot(lane < 6u && ((b >> ((lane & 7u) * 8u)) & 255u) == 255u); }
__device__ __forceinline__ bool all_ge(u32x4 v, u32 c) { return (int)((v.x - c) | (v.y - c) | (v.z - c) | (v.w - c)) >= 0; }

__device__ __forceinline__ u32 wave_max(u32 v) {
  v = max(v, (u32)__builtin_amdgcn_update_dpp(0, (int)v, 0x111, 0xf, 0xf, false));
  v = max(v, (u32)__builtin_amdgcn_update_dpp(0, (int)v, 0x112, 0xf, 0xf, false));
  v = max(v, (u32)__builtin_amdgcn_update_dpp(0, (int)v, 0x114, 0xf, 0xf, false));
  v = max(v, (u32)__builtin_amdgcn_update_dpp(0, (int)v, 0x118, 0xf, 0xf, false));
  v = max(v, (u32)__builtin_amdgcn_update_dpp(0, (int)v, 0x142, 0xa, 0xf, false));
  v = max(v, (u32)__builtin_amdgcn_update_dpp(0, (int)v, 0x143, 0xc, 0xf, false));
  return __builtin_amdgcn_readlane(v, 63);
}

template <int NWORD>
__device__ __forceinline__ void wave_wait(rsrc_t r, u32 off, bool want, u32 lane, u32 call, char* tmo) {
  const u64 t0 = __builtin_amdgcn_s_memrealtime();
  const u32 o = lane == 63u ? F_TMO : want ? off : OOB;
  for (;;) {
    const u32x4 v = __builtin_amdgcn_raw_buffer_load_b128(r, o, 0, 17);
    bool ok = (int)(v.x - call) >= 0;
    if (NWORD == 4) ok = ok && (int)(v.y - call) >= 0 && (int)(v.z - call) >= 0 && (int)(v.w - call) >= 0;
    if (__all(ok || !want || lane == 63u)) break;
    if (__builtin_amdgcn_readlane(v.x, 63) == call) break;
    if (__builtin_amdgcn_s_memrealtime() - t0 > DEADLINE) { st_agt(tmo, call); break; }
    __builtin_amdgcn_s_sleep(2);
  }
  asm volatile("" ::: "memory");
}

#define JIT() do {} while (0)
#define STAMP(i) do {} while (0)

}

__global__ void __launch_bounds__(512, 2) fused_kernel(Params params_by_value) {
  (void)params_by_value;
  const CSTP Params* const K = (const CSTP Params*)__builtin_amdgcn_kernarg_segment_ptr();
  const char* const gx = K->x;
  const char* const gw = K->w;
  const u32 warm = K->tab[min(blockIdx.x >> 3, 6u)][0];
  char* const far = K->peer[((blockIdx.x >> 3) + 1u) & 7u];
  const u32 nent = K->nent[(blockIdx.x >> 3) & 7u];
  asm volatile("" :: "s"(warm), "s"(far), "s"(nent));
  extern __shared__ __attribute__((aligned(16))) char smem[];
  LDSP char* lds = (LDSP char*)smem;
  volatile LDSP u32* ctrl = (volatile LDSP u32*)(lds + L_CTRL);
  const u32 tid = threadIdx.x, lane = tid & 63u, wid = rfl(tid >> 6);
  const u32 q = K->rank & 7u;
  char* const me = K->peer[0];
  char* const tmo = me + F_TMO;
  const u32 grp = blockIdx.x & 7u, li = blockIdx.x >> 3;
  char* const my_ctr = me + F_CALL + blockIdx.x * 64u;

  if (li < (u32)NPUSH) {
    if (NPWG == 4u && ((grp ^ li) & 1u)) return;
    const u32 d = li + 1u;
    const u32 total = min(nent, 96u) * (8u / NSTO);
    const rsrc_t r_me = mk_rsrc(me, WS_USED);
    const u32 vl = lane * 16u;
    volatile LDSP u32x4* const full = (volatile LDSP u32x4*)(lds + L_CTRL);
    volatile LDSP u32* const freed = (volatile LDSP u32*)(lds + L_CTRL + 128);
    LDSP u32* const acked = (LDSP u32*)(lds + L_CTRL + 256);
    const u32 callv = ld_agt(my_ctr);
    if (tid < 128u) ctrl[tid] = 0;
    WAIT_LGKM0();
    const bool is_ld = wid >= NSTO && wid < NSTO + NLD;
    if (!is_ld || d == 1u) BAR();
    if (wid == 0) STAMP(0);
    if (is_ld) {
      const u32 j = wid - NSTO;
      const u32 dst_rank = (q ^ d) & 7u;
      const u32 h = k_half(dst_rank, q);
      const u32 vx0 = (lane >> 3) * 1024u + (((lane & 7u) ^ (lane >> 4)) << 4), vx1 = vx0 ^ 64u;
      const u32 vw = ((lane >> 2) & 3u) * 8192u + (32u * (lane >> 4) + 8u * (lane & 3u)) * 2u;
      LDSP char* const slot = lds + j * (NSTO * 4096u);
      u32 pend = j * NPWG + grp / (8u / NPWG);
      u32 call = 0, n = 0;
      for (;; ++n) {
        if (n) {
          while (!__all(freed[j * 4u + lane % NSTO] == n)) __builtin_amdgcn_s_sleep(1);
          const u32 tkv = (u32)__builtin_amdgcn_raw_ptr_buffer_atomic_add_i32(1, r_me, lane == 0u ? F_UNIT + li * 128u : OOB, 0, 0);
          call = rfl(callv) + 1u;
          pend = rfl(tkv) - (call - 1u) * total + NLD * NPWG;
        }
        JIT();
        if (pend >= total) break;
        const u32 it = pend / (8u / NSTO), sub = pend % (8u / NSTO) * NSTO;
        const u32 ent = K->tab[li][it >> 1] >> ((it & 1u) * 16u);
        const u32 kind = (ent >> 8) & 7u, pl = ent & 255u;
        const bool in = kind <= E_W;
        u32 so, dl, fl, st = 1024u, gate = 0, ngate = NSTO;
        if (in) {
          const u32 c = pl >> 1, u = (pl & 1u) * 8u + sub;
          fl = F_LINK + (d * 64u + kind * 16u + c) * 128u + u * 4u;
          if (kind == E_X) { so = row_base(dst_rank, c) * 1024u + h * 512u + (u >> 2) * 128u + (u & 3u) * 32768u; dl = OFF_XS; st = 8192u; }
          else { so = (h * 256u + u * 16u) * 8192u + ((dst_rank >> 2) * 2048u + c * 128u) * 2u; dl = OFF_WS; st = 32768u; }
          dl += ((d - 2u) * 16u + c) * CH + u * 4096u;
        } else if (kind == E_FW) {
          const u32 c = pl >> 1, u = ((q >> 1) & 1u) * 8u + sub;
          const u32 dq = q ^ (((q >> 2) ^ 1u) * 4u + (pl & 1u) * 2u + (d & 1u)), dr = dq ^ d;
          so = OFF_WS + ((dq - 2u) * 16u + c) * CH + u * 4096u; gate = F_LINK + (dq * 64u + 16u + c) * 128u + u * 4u;
          dl = OFF_WS + ((dr - 2u) * 16u + c) * CH + u * 4096u; fl = F_LINK + (dr * 64u + 16u + c) * 128u + u * 4u;
        } else {
          const u32 rin = ((d ^ 1u) & 1u) * 64u + pl, rout = (d & 1u) * 64u + pl;
          if (kind == E_R2) { so = OFF_REL + rin * YT; gate = F_REL + rin * 128u + sub * 4u; }
          else { so = OFF_YOUT + pl * YT; gate = F_YRDY + pl * 16u + (sub >> 1) * 4u; ngate = NSTO / 2u; }
          if (kind == E_R1) { dl = OFF_REL + rout * YT; fl = F_REL + rout * 128u; }
          else { dl = OFF_YIN + pl * YT; fl = F_LINK + (pl < 64u ? 64u + pl : d * 64u + 32u + (pl & 31u)) * 128u; }
          so += sub * 4096u; dl += sub * 4096u; fl += sub * 4u;
        }
        if (!in) {
          call = rfl(callv) + 1u;
          wave_wait<1>(r_me, gate + lane * 4u, lane < ngate, lane, call, tmo);
        }
        const rsrc_t rs = mk_rsrc((const void*)(!in ? (u64)me : kind == E_X ? (u64)gx : (u64)gw), in ? X_BYTES : WS_USED);
        const u32 ve = !in ? vl : kind == E_X ? vx0 : vw, vo = !in ? vl : kind == E_X ? vx1 : vw;
#pragma unroll
        for (u32 i = 0; i < NSTO * 4u; ++i) __builtin_amdgcn_raw_ptr_buffer_load_lds(rs, slot + i * 1024u, 16, (i & 1u) ? vo : ve, so + i * st, 0, 17);
        if (n == 0u && d != 1u) BAR();
        WAIT_VM(0);
        JIT();
        if (lane == 0u) full[j] = u32x4{n + 1u, dl, fl, pend};
      }
      if (lane == 0u) full[j] = u32x4{n + 1u, OOB, 0u, ~0u};
    } else if (wid < NSTO) {
      const rsrc_t r_dst = mk_rsrc(far, WS_USED);
      const u32 js = (lane & 7u) % NLD;
      const u32 vf = lane < NSTO ? lane * 4u : OOB;
      u32 cnt = 0, prev = 0, prevc = 0;
      const u32 call = rfl(callv) + 1u;
      auto flag = [&]() {
        u32 old = 0;
        if (prev && lane == 0u) old = __hip_atomic_fetch_add(acked + prevc, 1u, __ATOMIC_RELAXED, __HIP_MEMORY_SCOPE_WORKGROUP);
        __builtin_amdgcn_raw_buffer_store_b32(call, r_dst, vf + (prev && rfl(old) % NSTO == NSTO - 1u ? prev : OOB >> 1), 0, 17);
        prev = 0;
      };
      for (;;) {
        const u32x4 e = full[js];
        const u32 key = e.x != cnt ? e.w : ~0u;
        u32 j = 0, bk = __builtin_amdgcn_readlane(key, 0);
#pragma unroll
        for (u32 i = 1; i < NLD; ++i) { const u32 k2 = __builtin_amdgcn_readlane(key, i); if (k2 < bk) { bk = k2; j = i; } }
        if (bk == ~0u) {
          if (prev) { WAIT_VM(0); flag(); }
          if (__all(e.x != cnt)) break;
          __builtin_amdgcn_s_sleep(1);
          continue;
        }
        u32x4 buf[4];
#pragma unroll
        for (u32 i = 0; i < 4; ++i) buf[i] = *(volatile LDSP u32x4*)(lds + (j * NSTO + wid) * 4096u + i * 1024u + vl);
        const u32 dl = __builtin_amdgcn_readlane(e.y, j) + wid * 4096u, fl = __builtin_amdgcn_readlane(e.z, j);
        const u32 gen = __builtin_amdgcn_readlane(e.x, j);
        if (js == j) ++cnt;
        WAIT_LGKM0();
        if (lane == 0u) freed[j * 4u + wid] = gen;
        JIT();
#define ST_(i) __builtin_amdgcn_raw_buffer_store_b128(buf[i], r_dst, vl + dl + (i) * 1024u, 0, 17)
        if (((W4MASK >> (grp * 2u + wid)) & 1u)) { WAIT_VM(4); ST_(0); WAIT_VM(4); ST_(1); WAIT_VM(4); ST_(2); WAIT_VM(4); ST_(3); flag(); }
        else if (((W2MASK >> (grp * 2u + wid)) & 1u)) { WAIT_VM(1); ST_(0); WAIT_VM(1); ST_(1); flag(); WAIT_VM(2); ST_(2); WAIT_VM(2); ST_(3); }
        else { WAIT_VM(3); ST_(0); WAIT_VM(3); ST_(1); WAIT_VM(2); ST_(2); flag(); WAIT_VM(3); ST_(3); }
#undef ST_
        prev = fl; prevc = j * 4u + (gen & 3u);
      }
      STAMP(1 + wid);
    }
    BAR();
    if (tid == 0) st_agt(my_ctr, rfl(callv) + 1u);
    return;
  }

  const u32 kidx = li - NPUSH;
  const u32 ga = grp >> 1, gb = grp & 1u;
  const bool loader = wid >= 4u;
  const u32 lw = wid & 3u;
  u32 call = 0;
  if (wid == 4u) {
    call = rfl(ld_agt(my_ctr)) + 1u;
    if (lane == 0) ctrl[16] = call;
    WAIT_LGKM0();
    STAMP(0);
  }
  const rsrc_t r_me = mk_rsrc(me, WS_USED);

  u32 kind, rt, ct;
  u32 fin = 0;
  bool have = kidx < 24u;
  if (have) { rt = ga + 4u * (kidx >> 3); ct = 2u * (kidx & 7u) + gb; kind = rt < 4u ? 0u : 1u; }
  else { rt = 0; ct = 0; kind = 0; }

  for (int round = 0; round < 2; ++round) {
    if (round == 1) {
      if (wid == 4u) {
        u32 t = 0;
        if (lane == 0) t = __hip_atomic_fetch_add((GLBP u32*)(me + F_TICKET + grp * 128u), 1u, __ATOMIC_RELAXED, __HIP_MEMORY_SCOPE_AGENT);
        t = rfl(t) - (call - 1u) * (u32)NGEMM;
        if (lane == 0) { ctrl[0] = t; st_agt(my_ctr, call); }
        if (lane >= 1u && lane < 16u) ctrl[lane] = 0;
        WAIT_LGKM0();
      }
      BAR();
      const u32 t = rfl(ctrl[0]);
      if (t >= 16u) return;
      have = true; fin = grp * 16u + t;
      kind = t < 8u ? 2u : 3u; rt = 12u + ga; ct = 2u * (t & 7u) + gb;
    }
    if (!have) continue;

    const u32 grow = row_base(q, rt);
    const u32 gcol = ((q >> 2) ^ (kind == 3u ? 1u : 0u)) * 2048u + ct * 128u;
    const u32 skip = kind == 1u ? 2u + (rt & 3u) : 99u;
    const u32 remote = kind == 3u ? 0u : kind == 1u ? 63u & ~(1u << skip) : 63u;
    const u32 pvo = ((remote >> (lane >> 3)) & 1u) ? F_LINK + (((lane >> 3) + 2u) * 64u + ((lane & 4u) ? 16u + ct : rt)) * 128u + (lane & 3u) * 16u : OOB;
    const u32 lbase = (u32)(size_t)lds + L_CTRL;
    const u32 pace = (K->pace[rt * 4u + (ct >> 2)] >> ((ct & 3u) * 8u)) & 255u;

    if (loader) {
      const u32 vx = (lw * 8u + (lane >> 3)) * 1024u + (((lane & 7u) ^ (((lw & 1u) * 4u + (lane >> 4)) & 7u)) << 4);
      const u32 vw = (lw * 4u + ((lane >> 2) & 3u)) * 8192u + (32u * (lane >> 4) + 8u * (lane & 3u)) * 2u;
      const u32 vs = lw * 1024u + lane * 16u;
      const u32 rot = ROT ? (2u * (rt >> 2) + (ct >> 1)) % 6u : 0u;
      if (wid == 4u) {
        if (remote && !(pace & 32u))
          wave_wait<1>(r_me, F_LINK + (2u * 64u + (pace & 31u)) * 128u, lane == 0, lane, call, tmo);
        lds_wr64(lbase + 96u, 0);
        lds_wr64(lbase + 104u, 0);
      }
      BAR();
      u32 rem = remote, rdy = 0, snap = 0, sx = 0, sw = 0;
      auto pick = [&]() -> bool {
        const u32 av = rdy & rem;
        if (!av) return false;
        u32 n = (u32)__builtin_ctz(((av >> rot) | (av << (6u - rot))) & 63u) + rot;
        if (n >= 6u) n -= 6u;
        rem &= ~(1u << n);
        sx = OFF_XS + (n * 16u + rt) * CH; sw = OFF_WS + (n * 16u + ct) * CH;
        return true;
      };
      auto choose = [&]() -> bool {
        if (!rem) return false;
        rdy |= (u32)lds_rd64(lbase + 96u);
        return pick();
      };
      auto issue = [&](u32 slot) {
        u32 dofs = slot * STAGE + lw * 1024u;
        asm volatile("" : "+s"(dofs));
        LDSP char* dst = lds + dofs;
#pragma unroll
        for (u32 i = 0; i < 4; ++i) {
          __builtin_amdgcn_raw_ptr_buffer_load_lds(r_me, dst + i * 4096u, 16, vs, sx + i * 4096u, 0, RAUX & 31);
          __builtin_amdgcn_raw_ptr_buffer_load_lds(r_me, dst + 16384u + i * 4096u, 16, vs, sw + i * 4096u, 0, RAUX & 31);
        }
        sx += 16384u; sw += 16384u;
      };
      auto shoot = [&]() {
        if (wid == 4u && rem && !(rem & rdy)) { __builtin_amdgcn_raw_ptr_buffer_load_lds(r_me, lds + L_SNAP, 16, pvo, 0, 0, 17); snap = 1; }
      };
      auto publish = [&]() {
        if (snap) { lds_wr64(lbase + 96u, seg_mask(__ballot(all_ge(lds_rd128(lbase + 1024u + lane * 16u), call)), lane)); snap = 0; }
      };
      {
        const rsrc_t r_x = mk_rsrc(gx, X_BYTES), r_w = mk_rsrc(gw, W_BYTES);
#pragma unroll 1
        for (u32 s8 = 0; s8 < 8; ++s8) {
          if (s8 == 4u) { WAIT_VM(24); BAR(); }
          if (s8 >= 4u) { WAIT_VM(16); BAR(); }
          if (s8 == 5u) shoot();
          LDSP char* dst = lds + (s8 & 3u) * STAGE + lw * 1024u;
          const u32 ox = grow * 1024u + s8 * 128u, ow = s8 * 64u * 8192u + gcol * 2u;
#pragma unroll
          for (u32 i = 0; i < 4; ++i) {
            __builtin_amdgcn_raw_ptr_buffer_load_lds(r_x, dst + i * 4096u, 16, vx, ox + i * 32768u, 0, 0);
            __builtin_amdgcn_raw_ptr_buffer_load_lds(r_w, dst + 16384u + i * 4096u, 16, vw, ow + i * 131072u, 0, 0);
          }
        }
      }
      for (;;) {
        for (;;) {
          JIT();
          const bool pre = pick();
          WAIT_VM(16); publish(); BAR();
          if (pre || choose()) {
            issue(0); WAIT_VM(16); BAR();
            shoot(); issue(1); WAIT_VM(16); BAR();
          } else {
            shoot(); WAIT_VM(8); BAR();
            WAIT_VM(0); publish(); BAR();
            if (!choose()) break;
            shoot();
#pragma unroll 1
            for (u32 i = 0; i < 2; ++i) issue(i);
          }
          JIT();
          issue(2); WAIT_VM(16); BAR();
          issue(3);
        }
        if (wid == 4u) lds_wr64(lbase + 104u, rem ? 1u : 2u);
        BAR();
        if (!rem) break;
        if (wid == 4u) {
          if (rem == remote) STAMP(round * 4 + 1);
          const u64 t0 = __builtin_amdgcn_s_memrealtime();
          const u32 want = min((u32)MINSEG, (u32)__builtin_popcount(rem));
          u64 bal;
          for (;;) {
            const u32x4 v = __builtin_amdgcn_raw_buffer_load_b128(r_me, lane == 63u ? F_TMO : pvo, 0, 17);
            bal = __ballot(all_ge(v, call));
            if ((u32)__builtin_popcount(seg_mask(bal, lane) & rem) >= want) break;
            bal = ~0ull;
            if (__builtin_amdgcn_readlane(v.x, 63) == call) break;
            if (__builtin_amdgcn_s_memrealtime() - t0 > DEADLINE) { st_agt(tmo, call); break; }
            __builtin_amdgcn_s_sleep(2);
          }
          lds_wr64(lbase + 96u, seg_mask(bal, lane));
          if (rem == remote) STAMP(round * 4 + 2);
        }
        BAR();
        if (wid == 4u) lds_wr64(lbase + 104u, 0);
        choose();
#pragma unroll 1
        for (u32 i = 0; i < 4; ++i) issue(i);
        WAIT_VM(24); BAR();
      }
      if (kind >= 2u) {
        const u32 callw = rfl(ctrl[16]);
        const u64 t0 = __builtin_amdgcn_s_memrealtime();
        u32 o0 = F_AMAX + (lw * 64u + lane) * 16u, o1 = o0 + 4096u, m = 0, bad = 0, seen = 0;
        for (;;) {
          const u32x4 g0 = __builtin_amdgcn_raw_buffer_load_b128(r_me, o0, 0, 17);
          const u32x4 g1 = __builtin_amdgcn_raw_buffer_load_b128(r_me, o1, 0, 17);
          const u32 tv = __builtin_amdgcn_raw_buffer_load_b32(r_me, lane == 0u ? F_TMO : OOB, 0, 17);
          if (g0.y == callw && g0.w == callw) { m = max(m, max(g0.x, g0.z)); o0 = OOB; }
          if (g1.y == callw && g1.w == callw) { m = max(m, max(g1.x, g1.z)); o1 = OOB; }
          if (rfl(tv) == callw) { bad = 1; break; }
          const u32 missing = (u32)__builtin_popcountll(__ballot((o0 & o1) != OOB));
          if (missing == 0u) break;
          if (__builtin_amdgcn_s_memrealtime() - t0 > DEADLINE) { st_agt(tmo, callw); bad = 1; break; }
          if (missing <= (u32)SPEC_MISSING && missing != seen) {
            const u32 pm = wave_max(m);
            if (lane == 0) *(volatile LDSP u32x2*)(lds + L_CTRL + 32 + lw * 8) = u32x2{pm, 1u};
            seen = missing;
          }
          if (missing > (u32)BACKOFF2) __builtin_amdgcn_s_sleep(40);
          else if (missing > (u32)BACKOFF) __builtin_amdgcn_s_sleep(BSLEEP);
          else __builtin_amdgcn_s_sleep(FSLEEP);
        }
        m = wave_max(m);
        if (lane == 0) *(volatile LDSP u32x2*)(lds + L_CTRL + 32 + lw * 8) = u32x2{m, 2u + bad};
        if (wid == 4u) STAMP(9);
      }
    } else {
      const u32 wn = wid & 1u, wm = wid >> 1;
      const u32 hh = lane >> 5, r31 = lane & 31u;
      const u32 z = (lane >> 4) & 1u, tq = (lane >> 2) & 3u, tp = lane & 3u;
      const u32 aw = 16384u + hh * 2048u + wn * 512u + tq * 64u + (16u * (tp & 1u) + 8u * z + 4u * (tp >> 1)) * 2u;
      const u32 axb = (wm * 64u + r31) * 128u, sw7 = (r31 >> 1) & 7u;
      u32 ax[4];
#pragma unroll
      for (u32 s = 0; s < 4; ++s) ax[s] = axb + (((2u * s + hh) ^ sw7) << 4);
      f32x16 acc[2][2] = {};
      s16x8 fa[4][2] = {}, fb[4][2] = {};
      s16x4 alo = {}, ahi = {};
      auto tr = [&](u32 off) { return __builtin_amdgcn_ds_read_tr16_b64_v4i16((LDSP s16x4*)(lds + off)); };
      auto rd0 = [&](int buf, u32 so, int ks) {
        const s16x4 lo = tr(so + aw + ks * 4096), hi = tr(so + aw + ks * 4096 + 1024);
        fa[buf][0] = __builtin_shufflevector(lo, hi, 0, 1, 2, 3, 4, 5, 6, 7);
        fb[buf][0] = *(LDSP s16x8*)(lds + so + ax[ks]);
      };
      auto rd1 = [&](int buf, u32 so, int ks) { fb[buf][1] = *(LDSP s16x8*)(lds + so + ax[ks] + 4096); alo = tr(so + aw + ks * 4096 + 256); };
      auto rd2 = [&](int buf, u32 so, int ks) { ahi = tr(so + aw + ks * 4096 + 256 + 1024); fa[buf][1] = __builtin_shufflevector(alo, ahi, 0, 1, 2, 3, 4, 5, 6, 7); };
      auto mm = [&](int buf, int fn, int fm) {
        acc[fn][fm] = __builtin_amdgcn_mfma_f32_32x32x16_bf16(__builtin_bit_cast(bf16x8, fa[buf][fn]), __builtin_bit_cast(bf16x8, fb[buf][fm]), acc[fn][fm], 0, 0, 0);
        SCHED();
      };
      u32 statv = 0;
      auto kstep = [&](int kc, u32 so, int kn, int sync) {
        mm(kc, 0, 0);
        if (RA == 2 && sync) mm(kc, 0, 1);
        if (sync) { WAIT_LGKM0(); BAR(); asm volatile("" ::: "memory"); }
        if (sync == 2) statv = ctrl[26];
        rd0(kn, so, kn); SCHED();
        if (!(RA == 2 && sync)) { mm(kc, 0, 1); rd1(kn, so, kn); SCHED(); }
        mm(kc, 1, 0);
        if (RA == 2 && sync) rd1(kn, so, kn);
        rd2(kn, so, kn); SCHED();
        mm(kc, 1, 1);
      };
      BAR();
      __builtin_amdgcn_s_waitcnt(0x0F70);
      for (;;) {
        BAR();
        asm volatile("" ::: "memory");
#pragma unroll
        for (int k = 0; k < RA; ++k) { rd0(k, 0, k); rd1(k, 0, k); rd2(k, 0, k); SCHED(); }
        u32 status;
        do {
#pragma unroll
          for (int st = 0; st < 4; ++st)
#pragma unroll
            for (int k = 0; k < 4; ++k)
              kstep(k, (k + RA < 4 ? st : (st + 1) & 3) * STAGE, (k + RA) & 3, k + RA == 4 ? (st == 3 ? 2 : 1) : 0);
          status = rfl(statv);
          JIT();
        } while (status == 0u);
        if (status == 2u) break;
        BAR();
      }
      const u32 vy = wid * 8192u + lane * 16u;
      if (kind < 2u) {
        const u32 ti = kind == 0u ? rt * 16u + ct : 64u + (rt & 3u) * 32u + ((rt - 4u) >> 2) * 16u + ct;
        const u32 callc = ctrl[16];
        const bool direct = kind == 0u && (pace >> 6) != 0u;
        char* const ybase = K->peer[direct ? 1u : 0u];
        const rsrc_t r_o = mk_rsrc(ybase, WS_USED);
        const u32 so = (direct ? OFF_YIN : OFF_YOUT) + ti * YT;
#pragma unroll
        for (int f = 0; f < 4; ++f)
#pragma unroll
          for (int hf = 0; hf < 2; ++hf) {
            u32x4 v;
#pragma unroll
            for (int e = 0; e < 4; ++e) {
              const f32x2 p = {acc[f >> 1][f & 1][hf * 8 + e * 2], acc[f >> 1][f & 1][hf * 8 + e * 2 + 1]};
              v[e] = __builtin_bit_cast(u32, __builtin_convertvector(p, bf16x2));
            }
            if (direct && (pace >> 6) == 2u) WAIT_VM(2);
            __builtin_amdgcn_raw_buffer_store_b128(v, r_o, vy + so + (f * 2 + hf) * 1024, 0, 17);
          }
        WAIT_VM(0);
        if (lane < (direct ? 2u : 1u)) st_sys(ybase + (direct ? F_LINK + (64u + ti) * 128u + wid * 8u + lane * 4u : F_YRDY + ti * 16u + wid * 4u), callc);
        if (wid == 0) STAMP(3);
      } else {
        const u32 callc = rfl(ctrl[16]);
        char* const gdst = K->peer[lane & 7u] + F_AMAX + (q * 128u + fin) * 8u;
        const u32 d0 = (q ^ (((q >> 2) ^ 1u) * 4u + (ga >> 1) * 2u)) & 7u, d1 = d0 ^ 1u;
        const u32 ty = (ga & 1u) * 16u + ct;
        const u32 so0 = OFF_YIN + (kind == 2u ? ga * 16u + ct : d0 * 32u - 64u + ty) * YT, so1 = OFF_YIN + (d1 * 32u - 64u + ty) * YT;
        const u32 fo0 = F_LINK + (kind == 2u ? 64u + ga * 16u + ct : d0 * 64u + 32u + ty) * 128u, fo1 = F_LINK + (d1 * 64u + 32u + ty) * 128u;
        const u32 po = lane == 63u ? F_TMO : lane < 4u ? ((lane & 2u) ? fo1 : fo0) + wid * 8u + (lane & 1u) * 4u : OOB;
        const rsrc_t r_out = mk_rsrc(K->out, OUT_BYTES);
        const u32 vo = (ga * 128u + wm * 64u + (lane >> 2)) * 4096u + gcol + wn * 64u + (lane & 3u) * 16u;
        const u64 t0 = __builtin_amdgcn_s_memrealtime();
        auto poll = [&](u32 todo, u32 pv) {
          u32 go;
          for (;; pv = __builtin_amdgcn_raw_buffer_load_b32(r_me, po, 0, 17)) {
            const u32 okm = (u32)__ballot((int)(pv - callc) >= 0);
            go = (((okm & 3u) == 3u ? 1u : 0u) | ((okm & 12u) == 12u ? 2u : 0u)) & todo;
            if (go) break;
            go = todo;
            if (__builtin_amdgcn_readlane(pv, 63) == callc) break;
            if (__builtin_amdgcn_s_memrealtime() - t0 > DEADLINE) { st_agt(tmo, callc); break; }
            __builtin_amdgcn_s_sleep(1);
          }
          asm volatile("" ::: "memory");
          return go;
        };
        auto fetch = [&](u32x4 (&v)[8], u32 so) {
#pragma unroll
          for (int j = 0; j < 8; ++j) v[j] = __builtin_amdgcn_raw_buffer_load_b128(r_me, vy + j * 1024, so, RAUX & 31);
        };
        auto add = [&](const u32x4 (&v)[8]) {
#pragma unroll
          for (int j = 0; j < 8; ++j)
#pragma unroll
            for (int e = 0; e < 4; ++e) {
              acc[j >> 2][(j >> 1) & 1][(j & 1) * 8 + e * 2] += __builtin_bit_cast(float, v[j][e] << 16);
              acc[j >> 2][(j >> 1) & 1][(j & 1) * 8 + e * 2 + 1] += __builtin_bit_cast(float, v[j][e] & 0xFFFF0000u);
            }
        };
        u32x4 va[8], vb[8];
        JIT();
        const u32 go = poll(kind == 2u ? 1u : 3u, __builtin_amdgcn_raw_buffer_load_b32(r_me, po, 0, 17));
        fetch(va, (go & 1u) ? so0 : so1);
        fetch(vb, go == 3u ? so1 : OOB);
        const u32 pv2 = __builtin_amdgcn_raw_buffer_load_b32(r_me, po, 0, 17);
        add(va);
        if (kind == 3u && go != 3u) {
          poll(go ^ 3u, pv2);
          fetch(vb, (go & 1u) ? so1 : so0);
        }
        if (kind == 3u) add(vb);
        if (wid == 0) STAMP(7);
        float mx = 0.f;
#pragma unroll
        for (int f = 0; f < 4; ++f)
#pragma unroll
          for (int e = 0; e < 16; ++e) mx = fmaxf(mx, acc[f >> 1][f & 1][e]);
        const u32 wm_ = wave_max(__builtin_bit_cast(u32, mx));
        u32 old = 0;
        if (lane == 0) {
          ctrl[4 + wid] = wm_;
          old = __hip_atomic_fetch_add((LDSP u32*)(lds + L_CTRL) + 1, 1u, __ATOMIC_RELAXED, __HIP_MEMORY_SCOPE_WORKGROUP);
        }
        if (rfl(old) == 3u) {
          const u32x4 m4 = *(volatile LDSP u32x4*)(lds + L_CTRL + 16);
          const u32 m = max(max(m4.x, m4.y), max(m4.z, m4.w));
          if (lane < 8u) __hip_atomic_store((GLBP u64*)gdst, ((u64)callc << 32) | m, __ATOMIC_RELAXED, __HIP_MEMORY_SCOPE_SYSTEM);
          STAMP(8);
        }
        u32 used = ~0u;
        for (;;) {
          const u32x4 a4 = *(volatile LDSP u32x4*)(lds + L_CTRL + 32), b4 = *(volatile LDSP u32x4*)(lds + L_CTRL + 48);
          const u32 lo = rfl(min(min(a4.y, a4.w), min(b4.y, b4.w))), hi = rfl(max(max(a4.y, a4.w), max(b4.y, b4.w)));
          if (lo == 0u) { __builtin_amdgcn_s_sleep(2); continue; }
          const u32 am = rfl(max(max(max(a4.x, a4.z), max(b4.x, b4.z)), wm_));
          const bool bad = hi == 3u;
          if (am != used || bad) {
            if (used != ~0u) WAIT_VM(0);
            used = am;
            const float amax = __builtin_bit_cast(float, am);
            const float scale = amax / 127.0f;
            const float inv = amax > 0.f ? 1.0f / scale : 0.f;
#pragma unroll
            for (int f = 0; f < 4; ++f) {
              u32x4 v;
#pragma unroll
              for (int e = 0; e < 4; ++e) {
                u32 pk = 0;
#pragma unroll
                for (int b = 0; b < 4; ++b) {
                  const float y = __builtin_amdgcn_fmed3f(__builtin_rintf(acc[f >> 1][f & 1][e * 4 + b] * inv), 0.f, 127.f);
                  pk = __builtin_amdgcn_cvt_pk_u8_f32(y, b, pk);
                }
                v[e] = bad ? 0x80808080u : pk;
              }
              *(LDSP u32x4*)(lds + wid * 4096u + ((f & 1) * 32u + r31) * 64u + ((((f >> 1) * 2u + hh) ^ ((r31 >> 1) & 3u)) << 4)) = v;
            }
#pragma unroll
            for (u32 i = 0; i < 4; ++i) {
              const u32x4 v = *(LDSP u32x4*)(lds + wid * 4096u + (i * 16u + (lane >> 2)) * 64u + (((lane & 3u) ^ ((lane >> 3) & 3u)) << 4));
              __builtin_amdgcn_raw_buffer_store_b128(v, r_out, vo + i * 16u * 4096u, 0, OUT_AUX);
            }
          }
          if (lo >= 2u) break;
          __builtin_amdgcn_s_sleep(1);
        }
        if (wid == 0) STAMP(10);
      }
    }
  }
}

namespace {
struct Sched { const char* cross; const char* sib; };
const Sched kSched[] = {
    {"T0 w0 T1 w1 T2 w2 T3 w3 w4 X0 w5 X1 w6 X2 w7 X3 w8 X4 w9 X5 w10 X6 w11 X7 w12 w13 w14 w15 O0 O1 O2 O3",
     "T0 w0 T1 w1 T2 w2 T3 w3 w4 X0 w5 X1 w6 X2 w7 X3 w8 X4 w9 X5 w10 X6 w11 X7 w12 w13 w14 w15 O0 O1 O2 O3 a12 a13 b12 b13"},
    {"T0 w0 T1 w1 T2 w2 T3 w3 w4 X0 w5 X1 w6 X2 w7 X3 w8 X4 w9 X5 w10 X6 w11 X7 w12 w13 w14 w15 O0 O1 O2 O3",
     "T0 w0 T1 w1 T2 w2 T3 w3 w4 X0 w5 X1 w6 X2 w7 X3 w8 X4 w9 X5 w10 X6 f10 w11 X7 f11 w12 w13 w14 w15 O0 O1 O2 O3 a12 a13 b12 b13"},
    {"T0 w0 T1 w1 T2 w2 T3 w3 w4 X0 w5 X1 w6 X2 w7 X3 X4 w8 w12 X5 w9 w13 X6 w15 X7 w10 w11 w14 O0 O1 O2 O3",
     "T0 w0 T1 w1 T2 w2 T3 w3 w4 X0 w5 X1 w6 X2 w7 X3 X4 w8 X5 w9 X6 X7 w10 f12 w11 f13 f15 w12 w13 w14 w15 O0 O1 O2 O3 a12 a13 b12 b13"},
    {"T0 w0 T1 w1 T2 w2 T3 w3 w4 X0 w5 X1 w6 X2 w7 X3 X4 w8 w11 w12 X5 X6 w9 w14 w15 X7 w10 w13 O0 O1 O2 O3",
     "T0 w0 T1 w1 T2 w2 T3 w3 w4 X0 w5 X1 w6 X2 w7 X3 X4 w8 X5 X6 w9 X7 w10 f11 f12 f14 f15 w11 w12 w13 w14 w15 O0 O1 O2 O3 a12 a14 b12 b14"},
    {"w9 w3 w0 T2 T0 T1 X5 w1 T3 w2 w10 w4 w5 w7 X4 X0 w6 X6 w8 X3 w11 X2 X7 X1 O1 w15 w12 w13 w14 O0 O3 O2",
     "T1 w0 X1 w2 T0 T3 w1 T2 w4 w3 X6 w5 X2 X3 X4 f9 w6 w7 X5 X7 w8 w9 X0 f11 w10 f12 f15 w11 w12 w13 w14 w15 O0 O1 O2 O3 a13 a14 b13 b14"},
    {"w0 T1 T2 w1 X7 T0 w3 T3 w2 w4 w6 w5 X4 X0 X1 X2 w7 w9 w10 X6 w8 X5 X3 w11 w12 w15 w13 O2 w14 O0 O3 O1",
     "w2 T3 w0 T0 w1 T1 T2 X1 w3 w5 X3 X6 X0 w4 f5 w6 X7 w7 X2 w8 X5 f10 X4 w9 w10 f13 w11 f15 w12 w13 w14 w15 O0 O1 O2 O3 a12 a14 b12 b14"},
    {"T0 w1 w3 T2 w0 T1 w8 T3 w2 w4 w5 X2 w7 w15 w6 X5 X4 w11 X3 X0 w14 X6 w9 X1 X7 w12 w10 O2 O1 w13 O0 O3",
     "T0 X5 T1 w4 w2 w6 w0 w1 w3 w5 T2 X7 T3 X3 X4 X2 w9 X6 w10 w7 X0 X1 w8 f11 f12 f15 f14 w11 w12 w13 w14 w15 O0 O1 O2 O3 a12 a14 b12 b14"},
    {"T0 w0 T1 w1 T2 w2 T3 w3 w4 X0 w5 X1 w6 X2 X3 w7 X4 X5 w8 X6 w11 w12 X7 w9 w14 w15 w10 w13 O0 O1 O2 O3",
     "T0 w0 T1 w1 T2 w2 T3 w3 w4 X0 w5 X1 w6 X2 w7 X3 X4 w8 X5 X6 w9 X7 w10 f11 f12 f14 f15 w11 w12 w13 w14 w15 O0 O1 O2 O3 a12 a14 b12 b14"},
    {"T0 w0 T1 w1 T2 w2 T3 w3 w4 X0 w5 X1 w6 X2 w7 X3 X4 w8 X5 w11 w12 X6 X7 w14 w15 w9 w10 w13 O0 O1 O2 O3",
     "T0 w0 T1 w1 T2 w2 T3 w3 w4 X0 w5 X1 w6 X2 w7 X3 X4 w8 X5 X6 w9 X7 w10 f11 f12 f14 f15 w11 w12 w13 w14 w15 O0 O1 O2 O3 a12 a14 b12 b14"},
    {"T0 w0 T1 w1 T2 w2 T3 w3 w4 X0 w5 X1 w6 X2 X3 w7 X4 X5 w8 X6 X7 w11 w9 w12 w14 w10 w15 w13 O0 O1 O2 O3",
     "T0 w0 T1 w1 T2 w2 T3 w3 w4 X0 w5 X1 w6 X2 w7 X3 X4 w8 X5 X6 X7 w9 w10 f11 f12 w11 f14 f15 w12 w13 w14 w15 O0 O1 O2 O3 a12 a13 b12 b13"},
    {"T0 w0 T1 w1 T2 w2 T3 w3 w4 X0 w5 X1 w6 X2 X3 w7 X4 X5 w8 X6 w11 w12 X7 w9 w13 w14 w15 w10 O0 O1 O2 O3",
     "T0 w0 T1 w1 T2 w2 T3 w3 w4 X0 w5 X1 w6 X2 w7 X3 X4 w8 X5 X6 w9 X7 w10 f11 f12 f13 w11 f14 f15 w12 w13 w14 w15 O0 O1 O2 O3 a12 a13 b12 b13"},
};

int parse_order(const char* p, int* v) {
  int n = 0;
  while (*p && n < 48) {
    const char c = *p++;
    int k = 0;
    while (*p >= '0' && *p <= '9') k = k * 10 + (*p++ - '0');
    while (*p == ' ') ++p;
    v[n++] = (c == 'T' ? 0 : c == 'X' ? 4 : c == 'O' ? 12 : c == 'w' ? 16 : c == 'f' ? 32 : c == 'a' ? 48 : 64) + k;
  }
  return n;
}

void build_tables(Params& P, int rank) {
  int cs[48], ss[48];
  const int nc = parse_order(kSched[SCHED_ID].cross, cs), ns = parse_order(kSched[SCHED_ID].sib, ss);
  u32 split = 0, relay = 0;
  for (int i = 0; i < ns; ++i) { if ((ss[i] >> 4) == 2) split |= 1u << (ss[i] & 15); if ((ss[i] >> 4) == 3) relay |= 1u << (ss[i] & 15); }
  const double C = 1.125;
  double ct[4][32] = {}, sb[48] = {}, first[16][16], done[16][16];
  for (int l = 0; l < 4; ++l) {
    double t = 0;
    for (int i = 0; i < nc; ++i) {
      const int it = cs[i];
      if (it >= 4 && it < 12 && (it & 3) == l) continue;
      ct[l][it] = t += (it >= 16 && ((split >> (it - 16)) & 1u)) ? C / 2 : C;
    }
  }
  double t = 0;
  for (int i = 0; i < ns; ++i) {
    const int it = ss[i];
    if ((it >> 4) == 2) for (int l = 0; l < 4; ++l) t = std::max(t, ct[l][it - 16] + 2.2);
    t += C;
    if (it < 48) sb[it] = t;
  }
  for (int r = 0; r < 16; ++r)
    for (int c = 0; c < 16; ++c) {
      double sg[6]; int n = 0;
      sg[n++] = std::max(sb[r], sb[16 + c]); sg[n++] = sg[0];
      for (int l = 0; l < 4; ++l)
        if (!(r >= 4 && r < 12 && (r & 3) == l)) sg[n++] = std::max(std::max(ct[l][r], ct[l][16 + c]), sb[32 + c]);
      std::sort(sg, sg + n);
      double e = first[r][c] = sg[1];
      for (int i = 0; i < n; ++i) e = std::max(e, sg[i] + (i >= 2 ? 0.9 : 0.0)) + 1.25;
      done[r][c] = e + 1e-3 * (c * 16 + r);
    }
  unsigned char pace[16][16];
  for (int r = 0; r < 16; ++r)
    for (int c = 0; c < 16; ++c) {
      int code = 32;
      for (int i = 0; i < ns; ++i) if (ss[i] < 32 && sb[ss[i]] - C <= first[r][c] - LEAD_US) code = ss[i];
      int earlier = 0;
      for (int k = 0; k < 64 && r < 4; ++k) earlier += !((relay >> (k & 15)) & 1u) && done[k >> 4][k & 15] < done[r][c];
      const int nt = 64 - 4 * __builtin_popcount(relay);
      const int direct = r >= 4 || ((relay >> c) & 1u) ? 0 : earlier < DIRECT_N ? (earlier < 6 ? 1 : 2) : earlier >= nt - DIRECT_LAST ? 2 : 0;
      pace[r][c] = (unsigned char)(code | direct << 6);
    }
  memcpy(P.pace, pace, sizeof pace);
  unsigned short tab[7][96] = {};
  memset(P.nent, 0, sizeof P.nent);
  memset(P.pad2, 0, sizeof P.pad2);
  P.pad = 0;
  for (int d = 1; d < 8; ++d) {
    int m = 0;
    auto put = [&](u32 kind, int pl) { if (m < 96) tab[d - 1][m++] = (unsigned short)(kind << 8 | (u32)pl); };
    const int tdst = ((rank ^ d) >> 1) & 1;
    for (int i = 0; i < (d >= 4 ? nc : d >= 2 ? ns : 0); ++i) {
      const int it = d >= 4 ? cs[i] : ss[i], k = it & 15;
      if (it < 32) {
        if (d >= 4 && it >= 4 && it < 12 && (it & 3) == d - 4) continue;
        for (int hf = 0; hf < 2; ++hf)
          if (!(d >= 4 && it >= 16 && ((split >> k) & 1u)) || hf == tdst) put(it < 16 ? E_X : E_W, k * 2 + hf);
      } else if (it < 48) { put(E_FW, k * 2); put(E_FW, k * 2 + 1); }
      else for (int r = (d ^ (it >= 64)) & 1; r < 4; r += 2) put(it >= 64 ? E_R2 : E_R1, r * 16 + k);
    }
    const int nt = d == 1 ? 64 : d >= 4 ? 32 : 0;
    int ord[64], no = 0;
    auto row = [&](int tl) { return d == 1 ? tl >> 4 : d + 4 * (tl >> 4); };
    for (int tl = 0; tl < nt; ++tl)
      if (d != 1 || !(((relay >> (tl & 15)) & 1u) || (pace[tl >> 4][tl & 15] >> 6))) ord[no++] = tl;
    std::sort(ord, ord + no, [&](int x, int y) { return done[row(x)][x & 15] < done[row(y)][y & 15]; });
    if (d >= 4)
      std::stable_sort(ord + no - STAGGER, ord + no, [&](int x, int y) { return ((x >> 4) ^ (rank & 1)) < ((y >> 4) ^ (rank & 1)); });
    for (int i = 0; i < no; ++i) put(E_Y, d == 1 ? ord[i] : d * 32 - 64 + ord[i]);
    P.nent[d - 1] = (u32)m;
  }
  memcpy(P.tab, tab, sizeof tab);
}
}

extern "C" void kernel_launch(int rank, int nranks, ncclComm_t comm, hipStream_t stream,
                              void* const* d_in, const int* in_sizes, int n_in,
                              void* d_out, int out_size, const int* in_dtypes, int out_dtype,
                              void* const* d_out_all, void* const* d_ws_all, size_t ws_bytes) {
  (void)comm; (void)in_dtypes; (void)out_dtype; (void)d_out_all;
  if (nranks != NR || (unsigned)rank >= (unsigned)NR || n_in < 2 || ws_bytes < WS_USED ||
      (size_t)in_sizes[0] * 2 != X_BYTES || (size_t)in_sizes[1] * 2 != W_BYTES || (u32)out_size != OUT_BYTES) {
    fprintf(stderr, "kernel_launch: unexpected problem shape\n");
    return;
  }
  Params P;
  P.x = (const char*)d_in[0];
  P.w = (const char*)d_in[1];
  P.out = (char*)d_out;
  for (int p = 0; p < NR; ++p) P.peer[p] = (char*)d_ws_all[rank ^ p];
  P.rank = (u32)rank;
  build_tables(P, rank);
  hipError_t e = hipFuncSetAttribute((const void*)fused_kernel, hipFuncAttributeMaxDynamicSharedMemorySize, (int)LDS_BYTES);
  if (e != hipSuccess) { fprintf(stderr, "hipFuncSetAttribute: %s\n", hipGetErrorString(e)); return; }
  fused_kernel<<<dim3(256), dim3(512), LDS_BYTES, stream>>>(P);
  e = hipGetLastError();
  if (e != hipSuccess) fprintf(stderr, "launch: %s\n", hipGetErrorString(e));
}
```

```cpp
#include <hip/hip_runtime.h>
#include <rccl/rccl.h>
#include <algorithm>
#include <cstdio>
#include <cstring>

typedef unsigned int u32;
typedef unsigned long long u64;
typedef __attribute__((ext_vector_type(4))) unsigned int u32x4;
typedef __attribute__((ext_vector_type(2))) unsigned int u32x2;
typedef __attribute__((ext_vector_type(4))) short s16x4;
typedef __attribute__((ext_vector_type(8))) short s16x8;
typedef __attribute__((ext_vector_type(8))) __bf16 bf16x8;
typedef __attribute__((ext_vector_type(2))) __bf16 bf16x2;
typedef __attribute__((ext_vector_type(2))) float f32x2;
typedef __attribute__((ext_vector_type(16))) float f32x16;
#define LDSP __attribute__((address_space(3)))
#define GLBP __attribute__((address_space(1)))
#define CSTP __attribute__((address_space(4)))
typedef __amdgpu_buffer_rsrc_t rsrc_t;

namespace {

constexpr int NR = 8;
constexpr u32 X_BYTES = 4096u * 512u * 2u;
constexpr u32 W_BYTES = 512u * 4096u * 2u;
constexpr u32 OUT_BYTES = 512u * 4096u;
constexpr u32 CH = 65536;
constexpr u32 YT = 32768;
constexpr u32 F_CALL = 0;
constexpr u32 F_TMO = 16384;
constexpr u32 F_TICKET = F_TMO + 1024;
constexpr u32 F_UNIT = F_TICKET + 1024;
constexpr u32 F_AMAX = 20480;
constexpr u32 F_YRDY = 28672;
constexpr u32 F_LINK = 32768;
constexpr u32 F_REL = F_LINK + 8 * 64 * 128;
constexpr u32 OFF_XS = 1u << 20;
constexpr u32 OFF_WS = OFF_XS + 6 * 16 * CH;
constexpr u32 OFF_YOUT = OFF_WS + 6 * 16 * CH;
constexpr u32 OFF_YIN = OFF_YOUT + 192 * YT;
constexpr u32 OFF_REL = OFF_YIN + 192 * YT;
constexpr u32 WS_USED = OFF_REL + 128 * YT;
constexpr u32 OOB = 0x80000000u;
constexpr int SYS = 17;

constexpr u32 STAGE = 32768;
constexpr u32 L_CTRL = 4 * STAGE;
constexpr u32 L_SNAP = L_CTRL + 1024;
constexpr u32 LDS_BYTES = L_SNAP + 1024;
constexpr u64 DEADLINE = 50000000ull;
constexpr u32 NPUSH = 7;
constexpr u32 NGEMM = 32 - NPUSH;
constexpr u32 NLD = 3;

enum : u32 { E_X = 0, E_W = 1, E_Y = 2, E_FW = 3, E_R1 = 4, E_R2 = 5 };
struct Params {
  const char* x;
  const char* w;
  char* out;
  u32 rank, pad;
  char* peer[NR];
  u32 nent[8];
  u32 pad2[16];
  u32 tab[7][48];
  u32x4 first[8][8];
  u32 pace[64];
};

__device__ __forceinline__ rsrc_t mk_rsrc(const void* p, u32 bytes) {
  return __builtin_amdgcn_make_buffer_rsrc((void*)p, 0, bytes, 0x00020000);
}
__device__ __forceinline__ u32 rfl(u32 v) { return __builtin_amdgcn_readfirstlane(v); }
__device__ __forceinline__ u32 ld_agt(const char* p) { return __hip_atomic_load((const GLBP u32*)p, __ATOMIC_RELAXED, __HIP_MEMORY_SCOPE_AGENT); }
__device__ __forceinline__ void st_sys(char* p, u32 v) { __hip_atomic_store((GLBP u32*)p, v, __ATOMIC_RELAXED, __HIP_MEMORY_SCOPE_SYSTEM); }
__device__ __forceinline__ void st_agt(char* p, u32 v) { __hip_atomic_store((GLBP u32*)p, v, __ATOMIC_RELAXED, __HIP_MEMORY_SCOPE_AGENT); }
__device__ __forceinline__ u32 poll32(rsrc_t r, u32 off) { asm volatile("" ::: "memory"); return __builtin_amdgcn_raw_buffer_load_b32(r, off, 0, SYS); }
__device__ __forceinline__ u32x4 poll128(rsrc_t r, u32 off) { asm volatile("" ::: "memory"); return __builtin_amdgcn_raw_buffer_load_b128(r, off, 0, SYS); }
#define WAIT_VM(n) asm volatile("s_waitcnt vmcnt(" #n ")" ::: "memory")
#define WAIT_LGKM0() asm volatile("s_waitcnt lgkmcnt(0)" ::: "memory")
#define BAR() __builtin_amdgcn_s_barrier()
#define SCHED() __builtin_amdgcn_sched_barrier(0)

__host__ __device__ __forceinline__ u32 row_base(u32 q, u32 r) {
  if (r < 4) return 512u * (q ^ 1u) + 128u * r;
  if (r < 12) return 512u * (q ^ (4u + (r & 3u))) + 256u * ((q >> 1) & 1u) + 128u * ((r - 4u) >> 2);
  return 512u * q + 128u * (r - 12u);
}
__host__ __device__ __forceinline__ u32 k_half(u32 q, u32 p) { return (q ^ ((q >> 1) & p)) & 1u; }

__device__ __forceinline__ u32 lds_rd32(u32 addr) {
  u32 v;
  asm volatile("ds_read_b32 %0, %1\n\ts_waitcnt lgkmcnt(0)" : "=&v"(v) : "v"(addr) : "memory");
  return rfl(v);
}
__device__ __forceinline__ void lds_wr32(u32 addr, u32 v) {
  asm volatile("ds_write_b32 %0, %1\n\ts_waitcnt lgkmcnt(0)" :: "v"(addr), "v"(v) : "memory");
}
__device__ __forceinline__ u32x4 lds_rd128(u32 addr) {
  u32x4 v;
  asm volatile("ds_read_b128 %0, %1\n\ts_waitcnt lgkmcnt(0)" : "=&v"(v) : "v"(addr) : "memory");
  return v;
}
__device__ __forceinline__ u32 seg_mask(u32x4 v, u32 call, u32 lane) {
  const u64 b = __ballot((int)((v.x - call) | (v.y - call) | (v.z - call) | (v.w - call)) >= 0);
  return (u32)__ballot(lane < 6u && ((b >> ((lane & 7u) * 8u)) & 255u) == 255u);
}

__device__ __forceinline__ u32 wave_max(u32 v) {
  v = max(v, (u32)__builtin_amdgcn_update_dpp(0, (int)v, 0x111, 0xf, 0xf, false));
  v = max(v, (u32)__builtin_amdgcn_update_dpp(0, (int)v, 0x112, 0xf, 0xf, false));
  v = max(v, (u32)__builtin_amdgcn_update_dpp(0, (int)v, 0x114, 0xf, 0xf, false));
  v = max(v, (u32)__builtin_amdgcn_update_dpp(0, (int)v, 0x118, 0xf, 0xf, false));
  v = max(v, (u32)__builtin_amdgcn_update_dpp(0, (int)v, 0x142, 0xa, 0xf, false));
  v = max(v, (u32)__builtin_amdgcn_update_dpp(0, (int)v, 0x143, 0xc, 0xf, false));
  return __builtin_amdgcn_readlane(v, 63);
}

template <int SLP>
__device__ __forceinline__ void wave_wait(rsrc_t r, u32 off, u32 n, u32 lane, u32 call, char* tmo) {
  const u64 t0 = __builtin_amdgcn_s_memrealtime();
  const u32 o = lane == 63u ? F_TMO : lane < n ? off + lane * 4u : OOB;
  for (;;) {
    const u32 v = poll32(r, o);
    if (__all((int)(v - call) >= 0 || lane >= n)) break;
    if (__builtin_amdgcn_readlane(v, 63) == call) break;
    if (__builtin_amdgcn_s_memrealtime() - t0 > DEADLINE) { st_agt(tmo, call); break; }
    __builtin_amdgcn_s_sleep(SLP);
  }
  asm volatile("" ::: "memory");
}

}

__global__ void __launch_bounds__(512, 2) fused_kernel(Params params_by_value) {
  (void)params_by_value;
  const CSTP Params* const K = (const CSTP Params*)__builtin_amdgcn_kernarg_segment_ptr();
  const char* const gx = K->x;
  const char* const gw = K->w;
  extern __shared__ __attribute__((aligned(16))) char smem[];
  LDSP char* lds = (LDSP char*)smem;
  volatile LDSP u32* ctrl = (volatile LDSP u32*)(lds + L_CTRL);
  const u32 tid = threadIdx.x, lane = tid & 63u, wid = rfl(tid >> 6);
  const u32 grp = blockIdx.x & 7u, li = blockIdx.x >> 3;
  const u32 pend0 = (wid - 2u) * 8u + grp;
  const u32x4 fst = K->first[li & 7u][(pend0 >> 2) & 7u];
  char* const far = K->peer[(li + 1u) & 7u];
  const u32 nent = K->nent[li & 7u];
  asm volatile("" :: "s"(fst), "s"(far), "s"(nent));
  const u32 vx0 = (lane >> 3) * 1024u + (((lane & 7u) ^ (lane >> 4)) << 4), vx1 = vx0 ^ 64u;
  const u32 vwp = ((lane >> 2) & 3u) * 8192u + (32u * (lane >> 4) + 8u * (lane & 3u)) * 2u;
  const bool fast = li - 1u < 6u && wid - 2u < NLD;
  if (fast) {
    const u32 sub = (pend0 & 3u) * 2u, st = fst.w ? 32768u : 8192u;
    const u32 so = fst.x + (fst.w ? sub * 131072u : (sub >> 2) * 128u + (sub & 3u) * 32768u);
    const rsrc_t rs = mk_rsrc((const void*)(fst.w ? (u64)gw : (u64)gx), X_BYTES);
#pragma unroll
    for (u32 i = 0; i < 8; ++i)
      __builtin_amdgcn_raw_ptr_buffer_load_lds(rs, lds + (wid - 2u) * 8192u + i * 1024u, 16, fst.w ? vwp : (i & 1u) ? vx1 : vx0, so + i * st, 0, SYS);
  }
  const u32 q = K->rank & 7u;
  char* const me = K->peer[0];
  char* const tmo = me + F_TMO;
  char* const my_ctr = me + F_CALL + blockIdx.x * 64u;
  const rsrc_t r_me = mk_rsrc(me, WS_USED);

  if (li < NPUSH) {
    const u32 d = li + 1u;
    const u32 total = min(nent, 96u) * 4u;
    const u32 vl = lane * 16u;
    volatile LDSP u32x4* const full = (volatile LDSP u32x4*)(lds + L_CTRL);
    volatile LDSP u32* const freed = (volatile LDSP u32*)(lds + L_CTRL + 128);
    LDSP u32* const acked = (LDSP u32*)(lds + L_CTRL + 256);
    const u32 callv = ld_agt(my_ctr);
    if (tid < 128u) ctrl[tid] = 0;
    WAIT_LGKM0();
    BAR();
    if (wid - 2u < NLD) {
      const u32 j = wid - 2u;
      const u32 dst_rank = (q ^ d) & 7u;
      const u32 h = k_half(dst_rank, q);
      u32 pend = pend0, n = 0;
      if (fast) {
        WAIT_VM(0);
        if (lane == 0u) full[j] = u32x4{1u, fst.y + (pend0 & 3u) * 8192u, fst.z + (pend0 & 3u) * 8u, pend0};
        n = 1;
      }
      const u32 call = rfl(callv) + 1u;
      for (;; ++n) {
        if (n) {
          while (!__all(freed[j * 4u + (lane & 1u)] == n)) __builtin_amdgcn_s_sleep(1);
          const u32 tkv = (u32)__builtin_amdgcn_raw_ptr_buffer_atomic_add_i32(1, r_me, lane == 0u ? F_UNIT + li * 128u : OOB, 0, 0);
          pend = rfl(tkv) - (call - 1u) * total + NLD * 8u;
        }
        if (pend >= total) break;
        const u32 it = pend >> 2, sub = (pend & 3u) * 2u;
        const u32 ent = K->tab[li][it >> 1] >> ((it & 1u) * 16u);
        const u32 kind = (ent >> 8) & 7u, pl = ent & 255u;
        const bool in = kind <= E_W;
        u32 so, dl, fl, st = 1024u, gate = 0, ngate = 2;
        if (in) {
          const u32 c = pl >> 1, u = (pl & 1u) * 8u + sub;
          fl = F_LINK + (d * 64u + kind * 16u + c) * 128u + u * 4u;
          if (kind == E_X) { so = row_base(dst_rank, c) * 1024u + h * 512u + (u >> 2) * 128u + (u & 3u) * 32768u; dl = OFF_XS; st = 8192u; }
          else { so = (h * 256u + u * 16u) * 8192u + ((dst_rank >> 2) * 2048u + c * 128u) * 2u; dl = OFF_WS; st = 32768u; }
          dl += ((d - 2u) * 16u + c) * CH + u * 4096u;
        } else if (kind == E_FW) {
          const u32 c = pl >> 1, u = ((q >> 1) & 1u) * 8u + sub;
          const u32 dq = q ^ (((q >> 2) ^ 1u) * 4u + (pl & 1u) * 2u + (d & 1u)), dr = dq ^ d;
          so = OFF_WS + ((dq - 2u) * 16u + c) * CH + u * 4096u; gate = F_LINK + (dq * 64u + 16u + c) * 128u + u * 4u;
          dl = OFF_WS + ((dr - 2u) * 16u + c) * CH + u * 4096u; fl = F_LINK + (dr * 64u + 16u + c) * 128u + u * 4u;
        } else {
          const u32 rin = ((d ^ 1u) & 1u) * 64u + pl, rout = (d & 1u) * 64u + pl;
          if (kind == E_R2) { so = OFF_REL + rin * YT; gate = F_REL + rin * 128u + sub * 4u; }
          else { so = OFF_YOUT + pl * YT; gate = F_YRDY + pl * 16u + sub * 2u; ngate = 1; }
          if (kind == E_R1) { dl = OFF_REL + rout * YT; fl = F_REL + rout * 128u; }
          else { dl = OFF_YIN + pl * YT; fl = F_LINK + (pl < 64u ? 64u + pl : d * 64u + 32u + (pl & 31u)) * 128u; }
          so += sub * 4096u; dl += sub * 4096u; fl += sub * 4u;
        }
        if (!in) wave_wait<2>(r_me, gate, ngate, lane, call, tmo);
        const rsrc_t rs = mk_rsrc((const void*)(!in ? (u64)me : kind == E_X ? (u64)gx : (u64)gw), in ? X_BYTES : WS_USED);
        const u32 ve = !in ? vl : kind == E_X ? vx0 : vwp, vo = !in ? vl : kind == E_X ? vx1 : vwp;
#pragma unroll
        for (u32 i = 0; i < 8; ++i) __builtin_amdgcn_raw_ptr_buffer_load_lds(rs, lds + j * 8192u + i * 1024u, 16, (i & 1u) ? vo : ve, so + i * st, 0, SYS);
        WAIT_VM(0);
        if (lane == 0u) full[j] = u32x4{n + 1u, dl, fl, pend};
      }
      if (lane == 0u) full[j] = u32x4{n + 1u, OOB, 0u, ~0u};
    } else if (wid < 2u) {
      const rsrc_t r_dst = mk_rsrc(far, WS_USED);
      const u32 js = (lane & 7u) % NLD;
      const u32 vf = lane < 2u ? lane * 4u : OOB;
      u32 cnt = 0, prev = 0, prevc = 0;
      const u32 call = rfl(callv) + 1u;
      auto flag = [&]() {
        u32 old = 0;
        if (prev && lane == 0u) old = __hip_atomic_fetch_add(acked + prevc, 1u, __ATOMIC_RELAXED, __HIP_MEMORY_SCOPE_WORKGROUP);
        __builtin_amdgcn_raw_buffer_store_b32(call, r_dst, vf + (prev && (rfl(old) & 1u) ? prev : OOB >> 1), 0, SYS);
        prev = 0;
      };
      for (;;) {
        const u32x4 e = full[js];
        const u32 key = e.x != cnt ? e.w : ~0u;
        u32 j = 0, bk = __builtin_amdgcn_readlane(key, 0);
#pragma unroll
        for (u32 i = 1; i < NLD; ++i) { const u32 k2 = __builtin_amdgcn_readlane(key, i); if (k2 < bk) { bk = k2; j = i; } }
        if (bk == ~0u) {
          if (prev) { WAIT_VM(0); flag(); }
          if (__all(e.x != cnt)) break;
          __builtin_amdgcn_s_sleep(1);
          continue;
        }
        u32x4 buf[4];
#pragma unroll
        for (u32 i = 0; i < 4; ++i) buf[i] = *(volatile LDSP u32x4*)(lds + j * 8192u + wid * 1024u + i * 2048u + vl);
        const u32 dl = __builtin_amdgcn_readlane(e.y, j) + wid * 1024u, fl = __builtin_amdgcn_readlane(e.z, j);
        const u32 gen = __builtin_amdgcn_readlane(e.x, j);
        if (js == j) ++cnt;
        WAIT_LGKM0();
        if (lane == 0u) freed[j * 4u + wid] = gen;
#define ST_(i, gap) do { __builtin_amdgcn_raw_buffer_store_b128(buf[i], r_dst, vl + dl + (i) * 2048u, 0, SYS); __builtin_amdgcn_s_sleep(gap); } while (0)
        WAIT_VM(3); ST_(0, 4); WAIT_VM(3); ST_(1, 4); WAIT_VM(2); ST_(2, 2); flag(); WAIT_VM(3); ST_(3, 1);
#undef ST_
        prev = fl; prevc = j * 4u + (gen & 3u);
      }
    }
    BAR();
    if (tid == 0) st_agt(my_ctr, rfl(callv) + 1u);
    return;
  }

  const u32 kidx = li - NPUSH;
  const u32 ga = grp >> 1, gb = grp & 1u;
  const u32 lw = wid & 3u;
  u32 call = 0;
  if (wid == 4u) {
    call = rfl(ld_agt(my_ctr)) + 1u;
    if (lane == 0) ctrl[16] = call;
    WAIT_LGKM0();
  }

  u32 rt = (ga + 4u * (kidx >> 3)) & 15u, ct = 2u * (kidx & 7u) + gb, kind = rt < 4u ? 0u : 1u, fin = 0;
  for (int round = kidx < 24u ? 0 : 1; round < 2; ++round) {
    if (round == 1) {
      if (wid == 4u) {
        u32 t = 0;
        if (lane == 0) t = __hip_atomic_fetch_add((GLBP u32*)(me + F_TICKET + grp * 128u), 1u, __ATOMIC_RELAXED, __HIP_MEMORY_SCOPE_AGENT);
        t = rfl(t) - (call - 1u) * NGEMM;
        if (lane == 0) { ctrl[0] = t; st_agt(my_ctr, call); }
        if (lane >= 1u && lane < 16u) ctrl[lane] = 0;
        WAIT_LGKM0();
      }
      BAR();
      const u32 t = rfl(ctrl[0]);
      if (t >= 16u) return;
      fin = grp * 16u + t;
      kind = t < 8u ? 2u : 3u; rt = 12u + ga; ct = 2u * (t & 7u) + gb;
    }
    const u32 gcol = ((q >> 2) ^ (kind == 3u ? 1u : 0u)) * 2048u + ct * 128u;

    if (wid >= 4u) {
      const u32 remote = kind == 3u ? 0u : kind == 1u ? 63u & ~(4u << (rt & 3u)) : 63u;
      const u32 pvo = ((remote >> (lane >> 3)) & 1u) ? F_LINK + (((lane >> 3) + 2u) * 64u + ((lane & 4u) ? 16u + ct : rt)) * 128u + (lane & 3u) * 16u : OOB;
      const u32 lbase = (u32)(size_t)lds + L_CTRL;
      const u32 vs = lw * 1024u + lane * 16u;
      if (wid == 4u) {
        const u32 pace = (K->pace[rt * 4u + (ct >> 2)] >> ((ct & 3u) * 8u)) & 63u;
        if (remote && pace < 32u)
          wave_wait<16>(r_me, F_LINK + (2u * 64u + pace) * 128u, 1, lane, call, tmo);
        lds_wr32(lbase + 96u, 0);
        lds_wr32(lbase + 104u, 0);
      }
      BAR();
      u32 rem = remote, rdy = 0, snap = 0, sx = 0, sw = 0;
      auto pick = [&]() -> bool {
        const u32 av = rdy & rem;
        if (!av) return false;
        const u32 n = (u32)__builtin_ctz(av);
        rem &= ~(1u << n);
        sx = OFF_XS + (n * 16u + rt) * CH; sw = OFF_WS + (n * 16u + ct) * CH;
        return true;
      };
      auto choose = [&]() -> bool {
        if (!rem) return false;
        rdy |= lds_rd32(lbase + 96u);
        return pick();
      };
      auto issue = [&](u32 slot) {
        u32 dofs = slot * STAGE + lw * 1024u;
        asm volatile("" : "+s"(dofs));
        LDSP char* dst = lds + dofs;
#pragma unroll
        for (u32 i = 0; i < 4; ++i) {
          __builtin_amdgcn_raw_ptr_buffer_load_lds(r_me, dst + i * 4096u, 16, vs, sx + i * 4096u, 0, SYS);
          __builtin_amdgcn_raw_ptr_buffer_load_lds(r_me, dst + 16384u + i * 4096u, 16, vs, sw + i * 4096u, 0, SYS);
        }
        sx += 16384u; sw += 16384u;
      };
      auto shoot = [&]() {
        if (wid == 4u && rem && !(rem & rdy)) { __builtin_amdgcn_raw_ptr_buffer_load_lds(r_me, lds + L_SNAP, 16, pvo, 0, 0, SYS); snap = 1; }
      };
      auto publish = [&]() {
        if (snap) { lds_wr32(lbase + 96u, seg_mask(lds_rd128(lbase + 1024u + lane * 16u), call, lane)); snap = 0; }
      };
      {
        const rsrc_t r_x = mk_rsrc(gx, X_BYTES), r_w = mk_rsrc(gw, W_BYTES);
        const u32 vx = lw * 8192u + ((lw & 1u) ? vx1 : vx0), vw = lw * 32768u + vwp;
        const u32 grow = row_base(q, rt);
#pragma unroll 1
        for (u32 s8 = 0; s8 < 8; ++s8) {
          if (s8 == 4u) { WAIT_VM(24); BAR(); }
          if (s8 >= 4u) { WAIT_VM(16); BAR(); }
          if (s8 == 5u) shoot();
          LDSP char* dst = lds + (s8 & 3u) * STAGE + lw * 1024u;
          const u32 ox = grow * 1024u + s8 * 128u, ow = s8 * 64u * 8192u + gcol * 2u;
#pragma unroll
          for (u32 i = 0; i < 4; ++i) {
            __builtin_amdgcn_raw_ptr_buffer_load_lds(r_x, dst + i * 4096u, 16, vx, ox + i * 32768u, 0, 0);
            __builtin_amdgcn_raw_ptr_buffer_load_lds(r_w, dst + 16384u + i * 4096u, 16, vw, ow + i * 131072u, 0, 0);
          }
        }
      }
      for (;;) {
        for (;;) {
          const bool pre = pick();
          WAIT_VM(16); publish(); BAR();
          if (pre || choose()) {
            issue(0); WAIT_VM(16); BAR();
            shoot(); issue(1); WAIT_VM(16); BAR();
          } else {
            shoot(); WAIT_VM(8); BAR();
            WAIT_VM(0); publish(); BAR();
            if (!choose()) break;
            shoot();
#pragma unroll 1
            for (u32 i = 0; i < 2; ++i) issue(i);
          }
          issue(2); WAIT_VM(16); BAR();
          issue(3);
        }
        if (wid == 4u) lds_wr32(lbase + 104u, rem ? 1u : 2u);
        BAR();
        if (!rem) break;
        if (wid == 4u) {
          const u64 t0 = __builtin_amdgcn_s_memrealtime();
          u32 sm;
          for (;;) {
            const u32x4 v = poll128(r_me, lane == 63u ? F_TMO : pvo);
            sm = seg_mask(v, call, lane);
            if (sm & rem) break;
            sm = 63u;
            if (__builtin_amdgcn_readlane(v.x, 63) == call) break;
            if (__builtin_amdgcn_s_memrealtime() - t0 > DEADLINE) { st_agt(tmo, call); break; }
            __builtin_amdgcn_s_sleep(2);
          }
          lds_wr32(lbase + 96u, sm);
        }
        BAR();
        if (wid == 4u) lds_wr32(lbase + 104u, 0);
        choose();
#pragma unroll 1
        for (u32 i = 0; i < 4; ++i) issue(i);
        WAIT_VM(24); BAR();
      }
      if (kind >= 2u) {
        const u32 callw = rfl(ctrl[16]);
        const u64 t0 = __builtin_amdgcn_s_memrealtime();
        volatile LDSP u32x2* const res = (volatile LDSP u32x2*)(lds + L_CTRL + 32u + lw * 8u);
        u32 o0 = F_AMAX + (lw * 64u + lane) * 16u, o1 = o0 + 4096u, m = 0, bad = 0, seen = 0;
        for (;;) {
          const u32x4 g0 = poll128(r_me, o0), g1 = poll128(r_me, o1);
          const u32 tv = poll32(r_me, lane == 0u ? F_TMO : OOB);
          if (g0.y == callw && g0.w == callw) { m = max(m, max(g0.x, g0.z)); o0 = OOB; }
          if (g1.y == callw && g1.w == callw) { m = max(m, max(g1.x, g1.z)); o1 = OOB; }
          if (rfl(tv) == callw) { bad = 1; break; }
          const u32 missing = (u32)__builtin_popcountll(__ballot((o0 & o1) != OOB));
          if (missing == 0u) break;
          if (missing <= 36u && missing != seen) {
            const u32 pm = wave_max(m);
            if (lane == 0) *res = u32x2{pm, 1u};
            seen = missing;
          }
          if (__builtin_amdgcn_s_memrealtime() - t0 > DEADLINE) { st_agt(tmo, callw); bad = 1; break; }
          if (missing > 24u) __builtin_amdgcn_s_sleep(40);
          else if (missing > 8u) __builtin_amdgcn_s_sleep(10);
          else __builtin_amdgcn_s_sleep(1);
        }
        m = wave_max(m);
        if (lane == 0) *res = u32x2{m, 2u + bad};
      }
    } else {
      const u32 wn = wid & 1u, wm = wid >> 1;
      const u32 hh = lane >> 5, r31 = lane & 31u;
      const u32 z = (lane >> 4) & 1u, tq = (lane >> 2) & 3u, tp = lane & 3u;
      const u32 aw = 16384u + hh * 2048u + wn * 512u + tq * 64u + (16u * (tp & 1u) + 8u * z + 4u * (tp >> 1)) * 2u;
      const u32 axb = (wm * 64u + r31) * 128u, sw7 = (r31 >> 1) & 7u;
      u32 ax[4];
#pragma unroll
      for (u32 s = 0; s < 4; ++s) ax[s] = axb + (((2u * s + hh) ^ sw7) << 4);
      f32x16 acc[2][2] = {};
      s16x8 fa[4][2] = {}, fb[4][2] = {};
      s16x4 alo = {};
      auto tr = [&](u32 off) -> s16x4 { return __builtin_amdgcn_ds_read_tr16_b64_v4i16((LDSP s16x4*)(lds + off)); };
      auto rd0 = [&](u32 so, int ks) {
        const s16x4 lo = tr(so + aw + ks * 4096), hi = tr(so + aw + ks * 4096 + 1024);
        fa[ks][0] = __builtin_shufflevector(lo, hi, 0, 1, 2, 3, 4, 5, 6, 7);
        fb[ks][0] = *(LDSP s16x8*)(lds + so + ax[ks]);
      };
      auto rd1 = [&](u32 so, int ks) { fb[ks][1] = *(LDSP s16x8*)(lds + so + ax[ks] + 4096); alo = tr(so + aw + ks * 4096 + 256); };
      auto rd2 = [&](u32 so, int ks) { fa[ks][1] = __builtin_shufflevector(alo, tr(so + aw + ks * 4096 + 256 + 1024), 0, 1, 2, 3, 4, 5, 6, 7); };
      auto mm = [&](int ks, int fn, int fm) {
        acc[fn][fm] = __builtin_amdgcn_mfma_f32_32x32x16_bf16(__builtin_bit_cast(bf16x8, fa[ks][fn]), __builtin_bit_cast(bf16x8, fb[ks][fm]), acc[fn][fm], 0, 0, 0);
        SCHED();
      };
      u32 statv = 0;
      auto kstep = [&](int kc, u32 so, int kn, int sync) {
        mm(kc, 0, 0);
        if (sync) {
          mm(kc, 0, 1);
          WAIT_LGKM0(); BAR(); asm volatile("" ::: "memory");
          if (sync == 2) statv = ctrl[26];
        }
        rd0(so, kn); SCHED();
        if (!sync) { mm(kc, 0, 1); rd1(so, kn); SCHED(); }
        mm(kc, 1, 0);
        if (sync) rd1(so, kn);
        rd2(so, kn); SCHED();
        mm(kc, 1, 1);
      };
      BAR();
      __builtin_amdgcn_s_waitcnt(0x0F70);
      for (;;) {
        BAR();
        asm volatile("" ::: "memory");
#pragma unroll
        for (int k = 0; k < 2; ++k) { rd0(0, k); rd1(0, k); rd2(0, k); SCHED(); }
        u32 status;
        do {
#pragma unroll
          for (int st = 0; st < 4; ++st)
#pragma unroll
            for (int k = 0; k < 4; ++k)
              kstep(k, (k < 2 ? st : (st + 1) & 3) * STAGE, (k + 2) & 3, k == 2 ? (st == 3 ? 2 : 1) : 0);
          status = rfl(statv);
        } while (status == 0u);
        if (status == 2u) break;
        BAR();
      }
      const u32 vy = wid * 8192u + lane * 16u;
      const u32 callc = rfl(ctrl[16]);
      if (kind < 2u) {
        const u32 ti = kind == 0u ? rt * 16u + ct : 64u + (rt & 3u) * 32u + ((rt - 4u) >> 2) * 16u + ct;
#pragma unroll
        for (int f = 0; f < 4; ++f)
#pragma unroll
          for (int hf = 0; hf < 2; ++hf) {
            u32x4 v;
#pragma unroll
            for (int e = 0; e < 4; ++e) {
              const f32x2 p = {acc[f >> 1][f & 1][hf * 8 + e * 2], acc[f >> 1][f & 1][hf * 8 + e * 2 + 1]};
              v[e] = __builtin_bit_cast(u32, __builtin_convertvector(p, bf16x2));
            }
            __builtin_amdgcn_raw_buffer_store_b128(v, r_me, vy + OFF_YOUT + ti * YT + (f * 2 + hf) * 1024, 0, SYS);
          }
        WAIT_VM(0);
        if (lane == 0u) st_sys(me + F_YRDY + ti * 16u + wid * 4u, callc);
      } else {
        char* const gdst = K->peer[lane & 7u] + F_AMAX + (q * 128u + fin) * 8u;
        const u32 d0 = (q ^ (((q >> 2) ^ 1u) * 4u + (ga >> 1) * 2u)) & 7u, d1 = d0 ^ 1u;
        const u32 ty = (ga & 1u) * 16u + ct;
        const u32 so0 = OFF_YIN + (kind == 2u ? ga * 16u + ct : d0 * 32u - 64u + ty) * YT, so1 = OFF_YIN + (d1 * 32u - 64u + ty) * YT;
        const u32 fo0 = F_LINK + (kind == 2u ? 64u + ga * 16u + ct : d0 * 64u + 32u + ty) * 128u, fo1 = F_LINK + (d1 * 64u + 32u + ty) * 128u;
        const u32 po = lane == 63u ? F_TMO : lane < 16u ? ((lane & 8u) ? fo1 : fo0) + (lane & 7u) * 4u : OOB;
        const rsrc_t r_out = mk_rsrc(K->out, OUT_BYTES);
        const u32 vo = (ga * 128u + wm * 64u + (lane >> 2)) * 4096u + gcol + wn * 64u + (lane & 3u) * 16u;
        const u64 t0 = __builtin_amdgcn_s_memrealtime();
        auto poll = [&](u32 todo, u32 pv) {
          u32 go;
          for (;; pv = poll32(r_me, po)) {
            const u32 okm = (u32)__ballot((int)(pv - callc) >= 0), own = okm >> (wid * 2u);
            go = (((own & 3u) == 3u ? 1u : 0u) | ((own & 0x300u) == 0x300u ? 2u : 0u)) & todo;
            if (go) break;
            go = todo;
            if (__builtin_amdgcn_readlane(pv, 63) == callc) break;
            if (__builtin_amdgcn_s_memrealtime() - t0 > DEADLINE) { st_agt(tmo, callc); break; }
            if (okm & (((todo & 1u) ? 0xFFu : 0u) | ((todo & 2u) ? 0xFF00u : 0u))) __builtin_amdgcn_s_sleep(1);
            else __builtin_amdgcn_s_sleep(12);
          }
          asm volatile("" ::: "memory");
          return go;
        };
        auto fetch = [&](u32x4 (&v)[8], u32 so) {
#pragma unroll
          for (int j = 0; j < 8; ++j) v[j] = __builtin_amdgcn_raw_buffer_load_b128(r_me, vy + j * 1024, so, SYS);
        };
        auto add = [&](const u32x4 (&v)[8]) {
#pragma unroll
          for (int j = 0; j < 8; ++j)
#pragma unroll
            for (int e = 0; e < 4; ++e) {
              acc[j >> 2][(j >> 1) & 1][(j & 1) * 8 + e * 2] += __builtin_bit_cast(float, v[j][e] << 16);
              acc[j >> 2][(j >> 1) & 1][(j & 1) * 8 + e * 2 + 1] += __builtin_bit_cast(float, v[j][e] & 0xFFFF0000u);
            }
        };
        u32x4 va[8], vb[8];
        const u32 go = poll(kind == 2u ? 1u : 3u, poll32(r_me, po));
        fetch(va, (go & 1u) ? so0 : so1);
        fetch(vb, go == 3u ? so1 : OOB);
        const u32 pv2 = poll32(r_me, po);
        add(va);
        if (kind == 3u && go != 3u) { poll(go ^ 3u, pv2); fetch(vb, (go & 1u) ? so1 : so0); }
        if (kind == 3u) add(vb);
        float mx = 0.f;
#pragma unroll
        for (int f = 0; f < 4; ++f)
#pragma unroll
          for (int e = 0; e < 16; ++e) mx = fmaxf(mx, acc[f >> 1][f & 1][e]);
        const u32 wmax = wave_max(__builtin_bit_cast(u32, mx) & 0x7FFFFFFFu);
        u32 old = 0;
        if (lane == 0) {
          ctrl[4 + wid] = wmax;
          old = __hip_atomic_fetch_add((LDSP u32*)(lds + L_CTRL) + 1, 1u, __ATOMIC_RELAXED, __HIP_MEMORY_SCOPE_WORKGROUP);
        }
        if (rfl(old) == 3u) {
          const u32x4 m4 = *(volatile LDSP u32x4*)(lds + L_CTRL + 16);
          const u32 m = max(max(m4.x, m4.y), max(m4.z, m4.w));
          if (lane < 8u) __hip_atomic_store((GLBP u64*)gdst, ((u64)callc << 32) | m, __ATOMIC_RELAXED, __HIP_MEMORY_SCOPE_SYSTEM);
        }
        u32 used = ~0u;
        for (;;) {
          const u32x4 a4 = *(volatile LDSP u32x4*)(lds + L_CTRL + 32), b4 = *(volatile LDSP u32x4*)(lds + L_CTRL + 48);
          const u32 lo = rfl(min(min(a4.y, a4.w), min(b4.y, b4.w))), hi = rfl(max(max(a4.y, a4.w), max(b4.y, b4.w)));
          if (lo == 0u) { __builtin_amdgcn_s_sleep(2); continue; }
          const u32 am = rfl(max(max(max(a4.x, a4.z), max(b4.x, b4.z)), wmax));
          const bool bad = hi == 3u;
          if (am != used || bad) {
            if (used != ~0u) WAIT_VM(0);
            used = am;
            const float amax = __builtin_bit_cast(float, am);
            const float scale = amax / 127.0f;
            const float inv = amax > 0.f ? 1.0f / scale : 0.f;
#pragma unroll
            for (int f = 0; f < 4; ++f) {
              u32x4 v;
#pragma unroll
              for (int e = 0; e < 4; ++e) {
                u32 pk = 0;
#pragma unroll
                for (int b = 0; b < 4; ++b) {
                  const float y = __builtin_amdgcn_fmed3f(__builtin_rintf(acc[f >> 1][f & 1][e * 4 + b] * inv), 0.f, 127.f);
                  pk = __builtin_amdgcn_cvt_pk_u8_f32(y, b, pk);
                }
                v[e] = bad ? 0x80808080u : pk;
              }
              *(LDSP u32x4*)(lds + wid * 4096u + ((f & 1) * 32u + r31) * 64u + ((((f >> 1) * 2u + hh) ^ ((r31 >> 1) & 3u)) << 4)) = v;
            }
#pragma unroll
            for (u32 i = 0; i < 4; ++i) {
              const u32x4 v = *(LDSP u32x4*)(lds + wid * 4096u + (i * 16u + (lane >> 2)) * 64u + (((lane & 3u) ^ ((lane >> 3) & 3u)) << 4));
              __builtin_amdgcn_raw_buffer_store_b128(v, r_out, vo + i * 16u * 4096u, 0, 16);
            }
          }
          if (lo >= 2u) break;
          __builtin_amdgcn_s_sleep(1);
        }
      }
    }
  }
}

namespace {
const char* const kCross = "T0 w0 T1 w1 T2 w2 T3 w3 w4 X0 w5 X1 w6 X2 X3 w7 X4 X5 w8 X6 X7 w11 w9 w12 w14 w10 w15 w13 O0 O1 O2 O3";
const char* const kSib = "T0 w0 T1 w1 T2 w2 T3 w3 w4 X0 w5 X1 w6 X2 w7 X3 X4 w8 X5 X6 X7 w9 w10 f11 f12 w11 f14 f15 w12 w13 w14 w15 "
                         "O0 O1 O2 O3 a12 a13 b12 b13";

int parse_order(const char* p, int* v) {
  int n = 0;
  while (*p && n < 48) {
    const char c = *p++;
    int k = 0;
    while (*p >= '0' && *p <= '9') k = k * 10 + (*p++ - '0');
    while (*p == ' ') ++p;
    v[n++] = (c == 'T' ? 0 : c == 'X' ? 4 : c == 'O' ? 12 : c == 'w' ? 16 : c == 'f' ? 32 : c == 'a' ? 48 : 64) + k;
  }
  return n;
}

void build_tables(Params& P, int rank) {
  int cs[48], ss[48];
  const int nc = parse_order(kCross, cs), ns = parse_order(kSib, ss);
  u32 split = 0, relay = 0;
  for (int i = 0; i < ns; ++i) { if ((ss[i] >> 4) == 2) split |= 1u << (ss[i] & 15); if ((ss[i] >> 4) == 3) relay |= 1u << (ss[i] & 15); }
  const double C = 1.125, LEAD = 8.0;
  double ct[4][32] = {}, sb[48] = {}, done[16][16];
  for (int l = 0; l < 4; ++l) {
    double t = 0;
    for (int i = 0; i < nc; ++i) {
      const int it = cs[i];
      if (it >= 4 && it < 12 && (it & 3) == l) continue;
      ct[l][it] = t += (it >= 16 && ((split >> (it - 16)) & 1u)) ? C / 2 : C;
    }
  }
  double t = 0;
  for (int i = 0; i < ns; ++i) {
    const int it = ss[i];
    if ((it >> 4) == 2) for (int l = 0; l < 4; ++l) t = std::max(t, ct[l][it - 16] + 2.2);
    t += C;
    if (it < 48) sb[it] = t;
  }
  unsigned char pace[16][16];
  for (int r = 0; r < 16; ++r)
    for (int c = 0; c < 16; ++c) {
      double sg[6]; int n = 0;
      sg[n++] = std::max(sb[r], sb[16 + c]); sg[n++] = sg[0];
      for (int l = 0; l < 4; ++l)
        if (!(r >= 4 && r < 12 && (r & 3) == l)) sg[n++] = std::max(std::max(ct[l][r], ct[l][16 + c]), sb[32 + c]);
      std::sort(sg, sg + n);
      double e = sg[1];
      for (int i = 0; i < n; ++i) e = std::max(e, sg[i] + (i >= 2 ? 0.9 : 0.0)) + 1.25;
      done[r][c] = e + 1e-3 * (c * 16 + r);
      pace[r][c] = 32;
      for (int i = 0; i < ns; ++i) if (ss[i] < 32 && sb[ss[i]] - C <= sg[1] - LEAD) pace[r][c] = (unsigned char)ss[i];
    }
  memcpy(P.pace, pace, sizeof pace);
  unsigned short tab[7][96] = {};
  for (int d = 1; d < 8; ++d) {
    int m = 0;
    auto put = [&](u32 kind, int pl) { if (m < 96) tab[d - 1][m++] = (unsigned short)(kind << 8 | (u32)pl); };
    const int tdst = ((rank ^ d) >> 1) & 1;
    for (int i = 0; i < (d >= 4 ? nc : d >= 2 ? ns : 0); ++i) {
      const int it = d >= 4 ? cs[i] : ss[i], k = it & 15;
      if (it < 32) {
        if (d >= 4 && it >= 4 && it < 12 && (it & 3) == d - 4) continue;
        for (int hf = 0; hf < 2; ++hf)
          if (!(d >= 4 && it >= 16 && ((split >> k) & 1u)) || hf == tdst) put(it < 16 ? E_X : E_W, k * 2 + hf);
      } else if (it < 48) { put(E_FW, k * 2); put(E_FW, k * 2 + 1); }
      else for (int r = (d ^ (it >= 64)) & 1; r < 4; r += 2) put(it >= 64 ? E_R2 : E_R1, r * 16 + k);
    }
    const int nt = d == 1 ? 64 : d >= 4 ? 32 : 0;
    int ord[64], no = 0;
    auto key = [&](int tl) { return done[d == 1 ? tl >> 4 : d + 4 * (tl >> 4)][tl & 15]; };
    for (int tl = 0; tl < nt; ++tl)
      if (d != 1 || !((relay >> (tl & 15)) & 1u)) ord[no++] = tl;
    std::sort(ord, ord + no, [&](int x, int y) { return key(x) < key(y); });
    if (d >= 4)
      std::sort(ord + no - 4, ord + no, [&](int x, int y) { return key(x) + 1e3 * ((x >> 4) ^ (rank & 1)) < key(y) + 1e3 * ((y >> 4) ^ (rank & 1)); });
    for (int i = 0; i < no; ++i) put(E_Y, d == 1 ? ord[i] : d * 32 - 64 + ord[i]);
    P.nent[d - 1] = (u32)m;
    for (int i = 0; i < 8 && d >= 2; ++i) {
      const u32 kind = tab[d - 1][i] >> 8, c = (tab[d - 1][i] & 255u) >> 1, u = (tab[d - 1][i] & 1u) * 8u;
      const u32 dst = (u32)(rank ^ d), h = k_half(dst, (u32)rank);
      P.first[d - 1][i] = u32x4{kind == E_X ? row_base(dst, c) * 1024u + h * 512u + (u >> 2) * 128u : (h * 256u + u * 16u) * 8192u + ((dst >> 2) * 2048u + c * 128u) * 2u,
                                (kind == E_X ? OFF_XS : OFF_WS) + ((d - 2u) * 16u + c) * CH + u * 4096u,
                                F_LINK + (d * 64u + kind * 16u + c) * 128u + u * 4u, kind};
    }
  }
  memcpy(P.tab, tab, sizeof tab);
}
}

extern "C" void kernel_launch(int rank, int nranks, ncclComm_t comm, hipStream_t stream,
                              void* const* d_in, const int* in_sizes, int n_in,
                              void* d_out, int out_size, const int* in_dtypes, int out_dtype,
                              void* const* d_out_all, void* const* d_ws_all, size_t ws_bytes) {
  (void)comm; (void)in_dtypes; (void)out_dtype; (void)d_out_all;
  if (nranks != NR || (unsigned)rank >= (unsigned)NR || n_in < 2 || ws_bytes < WS_USED ||
      (size_t)in_sizes[0] * 2 != X_BYTES || (size_t)in_sizes[1] * 2 != W_BYTES || (u32)out_size != OUT_BYTES) {
    fprintf(stderr, "kernel_launch: unexpected problem shape\n");
    return;
  }
  Params P;
  memset(&P, 0, sizeof P);
  P.x = (const char*)d_in[0];
  P.w = (const char*)d_in[1];
  P.out = (char*)d_out;
  for (int p = 0; p < NR; ++p) P.peer[p] = (char*)d_ws_all[rank ^ p];
  P.rank = (u32)rank;
  build_tables(P, rank);
  hipError_t e = hipFuncSetAttribute((const void*)fused_kernel, hipFuncAttributeMaxDynamicSharedMemorySize, (int)LDS_BYTES);
  if (e != hipSuccess) { fprintf(stderr, "hipFuncSetAttribute: %s\n", hipGetErrorString(e)); return; }
  fused_kernel<<<dim3(256), dim3(512), LDS_BYTES, stream>>>(P);
  e = hipGetLastError();
  if (e != hipSuccess) fprintf(stderr, "launch: %s\n", hipGetErrorString(e));
}
```

```cpp
#include <hip/hip_runtime.h>
#include <rccl/rccl.h>
#include <algorithm>
#include <cstdio>
#include <cstring>

typedef unsigned int u32;
typedef unsigned long long u64;
typedef __attribute__((ext_vector_type(4))) unsigned int u32x4;
typedef __attribute__((ext_vector_type(2))) unsigned int u32x2;
typedef __attribute__((ext_vector_type(4))) short s16x4;
typedef __attribute__((ext_vector_type(8))) short s16x8;
typedef __attribute__((ext_vector_type(8))) __bf16 bf16x8;
typedef __attribute__((ext_vector_type(2))) __bf16 bf16x2;
typedef __attribute__((ext_vector_type(2))) float f32x2;
typedef __attribute__((ext_vector_type(16))) float f32x16;
#define LDSP __attribute__((address_space(3)))
#define GLBP __attribute__((address_space(1)))
#define CSTP __attribute__((address_space(4)))
typedef __amdgpu_buffer_rsrc_t rsrc_t;

namespace {

constexpr int NR = 8;
constexpr u32 X_BYTES = 4096u * 512u * 2u;
constexpr u32 W_BYTES = 512u * 4096u * 2u;
constexpr u32 OUT_BYTES = 512u * 4096u;
constexpr u32 CH = 65536;
constexpr u32 YT = 32768;
constexpr u32 F_CALL = 0;
constexpr u32 F_TMO = 16384;
constexpr u32 F_TICKET = F_TMO + 1024;
constexpr u32 F_UNIT = F_TICKET + 1024;
constexpr u32 F_AMAX = 20480;
constexpr u32 F_YRDY = 28672;
constexpr u32 F_LINK = 32768;
constexpr u32 F_REL = F_LINK + 8 * 64 * 128;
constexpr u32 OFF_XS = 1u << 20;
constexpr u32 OFF_WS = OFF_XS + 6 * 16 * CH;
constexpr u32 OFF_YOUT = OFF_WS + 6 * 16 * CH;
constexpr u32 OFF_YIN = OFF_YOUT + 192 * YT;
constexpr u32 OFF_REL = OFF_YIN + 192 * YT;
constexpr u32 WS_USED = OFF_REL + 128 * YT;
constexpr u32 OOB = 0x80000000u;
constexpr int SYS = 17;

constexpr u32 STAGE = 32768;
constexpr u32 L_CTRL = 4 * STAGE;
constexpr u32 L_SNAP = L_CTRL + 1024;
constexpr u32 LDS_BYTES = L_SNAP + 1024;
constexpr u64 DEADLINE = 50000000ull;
constexpr u32 NPUSH = 7;
constexpr u32 NGEMM = 32 - NPUSH;
constexpr u32 NLD = 3;

enum : u32 { E_X = 0, E_W = 1, E_Y = 2, E_FW = 3, E_R1 = 4, E_R2 = 5 };
struct Params {
  const char* x;
  const char* w;
  char* out;
  u32 rank, pad;
  char* peer[NR];
  u32 nent[8];
  u32 pad2[16];
  u32 tab[7][48];
  u32x4 first[8][8];
  u32x4 fso[64];
  u32 pace[64];
};

__device__ __forceinline__ rsrc_t mk_rsrc(const void* p, u32 bytes) {
  return __builtin_amdgcn_make_buffer_rsrc((void*)p, 0, bytes, 0x00020000);
}
__device__ __forceinline__ u32 rfl(u32 v) { return __builtin_amdgcn_readfirstlane(v); }
__device__ __forceinline__ u32 ld_agt(const char* p) { return __hip_atomic_load((const GLBP u32*)p, __ATOMIC_RELAXED, __HIP_MEMORY_SCOPE_AGENT); }
__device__ __forceinline__ void st_sys(char* p, u32 v) { __hip_atomic_store((GLBP u32*)p, v, __ATOMIC_RELAXED, __HIP_MEMORY_SCOPE_SYSTEM); }
__device__ __forceinline__ void st_agt(char* p, u32 v) { __hip_atomic_store((GLBP u32*)p, v, __ATOMIC_RELAXED, __HIP_MEMORY_SCOPE_AGENT); }
__device__ __forceinline__ u32 poll32(rsrc_t r, u32 off) { asm volatile("" ::: "memory"); return __builtin_amdgcn_raw_buffer_load_b32(r, off, 0, SYS); }
__device__ __forceinline__ u32x4 poll128(rsrc_t r, u32 off) { asm volatile("" ::: "memory"); return __builtin_amdgcn_raw_buffer_load_b128(r, off, 0, SYS); }
#define WAIT_VM(n) asm volatile("s_waitcnt vmcnt(" #n ")" ::: "memory")
#define WAIT_LGKM0() asm volatile("s_waitcnt lgkmcnt(0)" ::: "memory")
#define BAR() __builtin_amdgcn_s_barrier()
#define SCHED() __builtin_amdgcn_sched_barrier(0)

__host__ __device__ __forceinline__ u32 row_base(u32 q, u32 r) {
  if (r < 4) return 512u * (q ^ 1u) + 128u * r;
  if (r < 12) return 512u * (q ^ (4u + (r & 3u))) + 256u * ((q >> 1) & 1u) + 128u * ((r - 4u) >> 2);
  return 512u * q + 128u * (r - 12u);
}
__host__ __device__ __forceinline__ u32 k_half(u32 q, u32 p) { return (q ^ ((q >> 1) & p)) & 1u; }

__device__ __forceinline__ u32 lds_rd32(u32 addr) {
  u32 v;
  asm volatile("ds_read_b32 %0, %1\n\ts_waitcnt lgkmcnt(0)" : "=&v"(v) : "v"(addr) : "memory");
  return rfl(v);
}
__device__ __forceinline__ void lds_wr32(u32 addr, u32 v) {
  asm volatile("ds_write_b32 %0, %1\n\ts_waitcnt lgkmcnt(0)" :: "v"(addr), "v"(v) : "memory");
}
__device__ __forceinline__ u32x4 lds_rd128(u32 addr) {
  u32x4 v;
  asm volatile("ds_read_b128 %0, %1\n\ts_waitcnt lgkmcnt(0)" : "=&v"(v) : "v"(addr) : "memory");
  return v;
}
__device__ __forceinline__ u32 seg_mask(u32x4 v, u32 call, u32 lane) {
  const u64 b = __ballot((int)((v.x - call) | (v.y - call) | (v.z - call) | (v.w - call)) >= 0);
  return (u32)__ballot(lane < 6u && ((b >> ((lane & 7u) * 8u)) & 255u) == 255u);
}

__device__ __forceinline__ u32 wave_max(u32 v) {
  v = max(v, (u32)__builtin_amdgcn_update_dpp(0, (int)v, 0x111, 0xf, 0xf, false));
  v = max(v, (u32)__builtin_amdgcn_update_dpp(0, (int)v, 0x112, 0xf, 0xf, false));
  v = max(v, (u32)__builtin_amdgcn_update_dpp(0, (int)v, 0x114, 0xf, 0xf, false));
  v = max(v, (u32)__builtin_amdgcn_update_dpp(0, (int)v, 0x118, 0xf, 0xf, false));
  v = max(v, (u32)__builtin_amdgcn_update_dpp(0, (int)v, 0x142, 0xa, 0xf, false));
  v = max(v, (u32)__builtin_amdgcn_update_dpp(0, (int)v, 0x143, 0xc, 0xf, false));
  return __builtin_amdgcn_readlane(v, 63);
}

template <int SLP>
__device__ __forceinline__ void wave_wait(rsrc_t r, u32 off, u32 n, u32 lane, u32 call, char* tmo) {
  const u64 t0 = __builtin_amdgcn_s_memrealtime();
  const u32 o = lane == 63u ? F_TMO : lane < n ? off + lane * 4u : OOB;
  for (;;) {
    const u32 v = poll32(r, o);
    if (__all((int)(v - call) >= 0 || lane >= n)) break;
    if (__builtin_amdgcn_readlane(v, 63) == call) break;
    if (__builtin_amdgcn_s_memrealtime() - t0 > DEADLINE) { st_agt(tmo, call); break; }
    __builtin_amdgcn_s_sleep(SLP);
  }
  asm volatile("" ::: "memory");
}

}

extern "C" __global__ void __launch_bounds__(512, 2) fused_kernel(Params params_by_value) {
  (void)params_by_value;
  const CSTP Params* const K = (const CSTP Params*)__builtin_amdgcn_kernarg_segment_ptr();
  const u32 grp = blockIdx.x & 7u, li = blockIdx.x >> 3;
  const char* const gx = K->x;
  const char* const gw = K->w;
  const u32x4 fso = K->fso[blockIdx.x & 63u];
  char* const far = K->peer[(li + 1u) & 7u];
  SCHED();
  extern __shared__ __attribute__((aligned(16))) char smem[];
  LDSP char* lds = (LDSP char*)smem;
  LDSP char* const sink = lds + L_CTRL + 512u;
  {
    u64 pc; u32 code_bytes;
    asm volatile("s_getpc_b64 %0\nfused_kernel_here:\n\ts_mov_b32 %1, fused_kernel_mark-fused_kernel_here" : "=&s"(pc), "=&s"(code_bytes));
    __builtin_amdgcn_raw_ptr_buffer_load_lds(mk_rsrc((const void*)pc, li == 31u ? code_bytes : 0u), sink, 4, threadIdx.x * 128u, 0, 0, 0);
    SCHED();
  }
  volatile LDSP u32* ctrl = (volatile LDSP u32*)(lds + L_CTRL);
  const u32 tid = threadIdx.x, lane = tid & 63u, wid = rfl(tid >> 6);
  const u32 vx0 = (lane >> 3) * 1024u + (((lane & 7u) ^ (lane >> 4)) << 4), vx1 = vx0 ^ 64u;
  const u32 vwp = ((lane >> 2) & 3u) * 8192u + (32u * (lane >> 4) + 8u * (lane & 3u)) * 2u;
  const bool fast = li - 1u < 6u && wid < 2u + NLD;
  const u32 twv = tid ? (li & 7u) * 16u * CH + (tid - 1u) * (OFF_WS - OFF_XS) : 0u, twb = li >= 24u ? OFF_YOUT : 0u;
  const u32 vx01 = wid ? vx1 + 8192u : vx0, head = fast && wid < 2u;
  asm volatile("" :: "v"(twv), "s"(twb), "v"(vx01), "s"(head));
  SCHED();
  asm volatile("" :: "s"(gx), "s"(gw), "s"(fso), "s"(far));
  __builtin_amdgcn_raw_ptr_buffer_load_lds(mk_rsrc(far, twb), sink, 4, twv, 0, 0, SYS);
  if (head) {
    u32x4 b0[4];
#pragma unroll
    for (u32 i = 0; i < 4; ++i) b0[i] = __builtin_amdgcn_raw_buffer_load_b128(mk_rsrc(gx, X_BYTES), vx01, fso.x + i * 16384u, SYS);
    const u32 vd = OFF_XS + (li - 1u) * 16u * CH + grp * 8192u + tid * 16u;
#pragma unroll
    for (u32 i = 0; i < 4; ++i) {
      __builtin_amdgcn_raw_buffer_store_b128(b0[i], mk_rsrc(far, WS_USED), vd + i * 2048u, 0, SYS);
      if (i < 3) __builtin_amdgcn_s_sleep(2);
    }
  }
  if (fast && wid >= 2u) {
    const u32 f = wid == 2u ? fso.y : wid == 3u ? fso.z : fso.w;
    const bool isw = f & 1u;
    const rsrc_t rs = mk_rsrc((const void*)(isw ? (u64)gw : (u64)gx), X_BYTES);
    const u32 ve = isw ? vwp : vx0, vo = isw ? vwp + 32768u : vx1 + 8192u;
    u32 so = f & ~1u;
#pragma unroll 1
    for (u32 i = 0; i < 8192u; i += 2048u, so += isw ? 65536u : 16384u) {
      __builtin_amdgcn_raw_ptr_buffer_load_lds(rs, lds + (wid - 2u) * 8192u + i, 16, ve, so, 0, SYS);
      __builtin_amdgcn_raw_ptr_buffer_load_lds(rs, lds + (wid - 2u) * 8192u + i + 1024u, 16, vo, so, 0, SYS);
    }
    WAIT_VM(0);
    BAR();
  }
  if (wid == 7u && li == 31u)
    __builtin_amdgcn_raw_ptr_buffer_load_lds(mk_rsrc((const void*)K, sizeof(Params)), sink, 4, lane * 128u, 0, 0, 0);
  if (wid == 0u && li - 24u < 7u) {
    const u32 dd = li - 23u;
    __builtin_amdgcn_s_sleep(48);
    __builtin_amdgcn_raw_ptr_buffer_load_lds(mk_rsrc(far, WS_USED), sink, 4, dd >= 4u ? (lane == 0u ? OFF_YIN + (dd * 32u - 64u) * YT : OOB)
        : lane < 2u ? OFF_YIN + (lane << 20) : dd == 1u ? OOB : lane < 6u ? OFF_WS + lane * 16u * CH
        : lane < 8u ? OFF_REL + (dd & 1u) * 64u * YT + ((lane - 6u) << 20) : OOB, 0, 0, SYS);
    __builtin_amdgcn_raw_ptr_buffer_load_lds(mk_rsrc(K->peer[0], WS_USED), sink, 4, lane < 4u ? (dd * 4u - 3u + lane) << 20 : OOB, 0, 0, SYS);
  }
  const u32 q = K->rank & 7u;
  char* const me = K->peer[0];
  char* const tmo = me + F_TMO;
  char* const my_ctr = me + F_CALL + blockIdx.x * 64u;
  const rsrc_t r_me = mk_rsrc(me, WS_USED);

  if (li < NPUSH) {
    const u32 d = li + 1u;
    const u32 total = min(K->nent[li], 96u) * 4u;
    const u32 nstat = d == 1u ? NLD * 8u : NLD * 8u + 8u;
    const u32 vl = lane * 16u;
    volatile LDSP u32x4* const full = (volatile LDSP u32x4*)(lds + L_CTRL);
    volatile LDSP u32* const freed = (volatile LDSP u32*)(lds + L_CTRL + 128);
    LDSP u32* const acked = (LDSP u32*)(lds + L_CTRL + 256);
    auto get_call = [&]() { u32 c; while ((c = rfl(ctrl[127])) == 0u) __builtin_amdgcn_s_sleep(1); return c; };
    if (tid - 320u < 128u) ctrl[tid - 320u] = 0;
    if (wid == 5u && d != 1u) {
#pragma unroll
      for (u32 j = 0; j < NLD; ++j) {
        const u32 p0 = j * 8u + 8u + grp;
        const u32x4 fst = K->first[li][p0 >> 2];
        if (lane == 0u) full[j] = u32x4{1u, fst.y + (p0 & 3u) * 8192u, fst.z + (p0 & 3u) * 8u, p0};
      }
    }
    WAIT_LGKM0();
    if (!fast || wid < 2u) BAR();
    if (wid == 7u) {
      if (lane == 0u) { const u32 c = ld_agt(my_ctr) + 1u; ctrl[127] = c; st_agt(my_ctr, c); }
    } else if (wid - 2u < NLD) {
      const u32 j = wid - 2u;
      const u32 dst_rank = (q ^ d) & 7u;
      const u32 h = k_half(dst_rank, q);
      u32 pend = j * 8u + grp, n = fast ? 1u : 0u;
      const u32 call = get_call();
      for (;; ++n) {
        if (n) {
          while (!__all(freed[j * 4u + (lane & 1u)] == n)) __builtin_amdgcn_s_sleep(1);
          const u32 tkv = (u32)__builtin_amdgcn_raw_ptr_buffer_atomic_add_i32(1, r_me, lane == 0u ? F_UNIT + li * 128u : OOB, 0, 0);
          pend = rfl(tkv) - (call - 1u) * (total + NLD * 8u - nstat) + nstat;
        }
        if (pend >= total) break;
        const u32 it = pend >> 2, sub = (pend & 3u) * 2u;
        const u32 ent = K->tab[li][it >> 1] >> ((it & 1u) * 16u);
        const u32 kind = (ent >> 8) & 7u, pl = ent & 255u;
        const bool in = kind <= E_W;
        u32 so, dl, fl, st = 1024u, gate = 0, ngate = 2;
        if (in) {
          const u32 c = pl >> 1, u = (pl & 1u) * 8u + sub;
          fl = F_LINK + (d * 64u + kind * 16u + c) * 128u + u * 4u;
          if (kind == E_X) { so = row_base(dst_rank, c) * 1024u + h * 512u + (u >> 2) * 128u + (u & 3u) * 32768u; dl = OFF_XS; st = 8192u; }
          else { so = (h * 256u + u * 16u) * 8192u + ((dst_rank >> 2) * 2048u + c * 128u) * 2u; dl = OFF_WS; st = 32768u; }
          dl += ((d - 2u) * 16u + c) * CH + u * 4096u;
        } else if (kind == E_FW) {
          const u32 c = pl >> 1, u = ((q >> 1) & 1u) * 8u + sub;
          const u32 dq = q ^ (((q >> 2) ^ 1u) * 4u + (pl & 1u) * 2u + (d & 1u)), dr = dq ^ d;
          so = OFF_WS + ((dq - 2u) * 16u + c) * CH + u * 4096u; gate = F_LINK + (dq * 64u + 16u + c) * 128u + u * 4u;
          dl = OFF_WS + ((dr - 2u) * 16u + c) * CH + u * 4096u; fl = F_LINK + (dr * 64u + 16u + c) * 128u + u * 4u;
        } else {
          const u32 rin = ((d ^ 1u) & 1u) * 64u + pl, rout = (d & 1u) * 64u + pl;
          if (kind == E_R2) { so = OFF_REL + rin * YT; gate = F_REL + rin * 128u + sub * 4u; }
          else { so = OFF_YOUT + pl * YT; gate = F_YRDY + pl * 16u + sub * 2u; ngate = 1; }
          if (kind == E_R1) { dl = OFF_REL + rout * YT; fl = F_REL + rout * 128u; }
          else { dl = OFF_YIN + pl * YT; fl = F_LINK + (pl < 64u ? 64u + pl : d * 64u + 32u + (pl & 31u)) * 128u; }
          so += sub * 4096u; dl += sub * 4096u; fl += sub * 4u;
        }
        if (!in) wave_wait<2>(r_me, gate, ngate, lane, call, tmo);
        const rsrc_t rs = mk_rsrc((const void*)(!in ? (u64)me : kind == E_X ? (u64)gx : (u64)gw), in ? X_BYTES : WS_USED);
        const u32 ve = !in ? vl : kind == E_X ? vx0 : vwp, vo = !in ? vl : kind == E_X ? vx1 : vwp;
#pragma unroll
        for (u32 i = 0; i < 8; ++i) __builtin_amdgcn_raw_ptr_buffer_load_lds(rs, lds + j * 8192u + i * 1024u, 16, (i & 1u) ? vo : ve, so + i * st, 0, SYS);
        WAIT_VM(0);
        if (lane == 0u) full[j] = u32x4{n + 1u, dl, fl, pend};
      }
      asm volatile("fused_kernel_mark:");
      if (lane == 0u) full[j] = u32x4{n + 1u, OOB, 0u, ~0u};
    } else if (wid < 2u) {
      const rsrc_t r_dst = mk_rsrc(far, WS_USED);
      const u32 js = (lane & 7u) % NLD;
      const u32 vf = lane < 2u ? lane * 4u : OOB;
      u32 cnt = 0, call = 0, prev = fast ? F_LINK + d * 64u * 128u + grp * 8u : 0u, prevc = NLD * 4u;
      auto flag = [&]() {
        u32 old = 0;
        if (prev && !call) call = get_call();
        if (prev && lane == 0u) old = __hip_atomic_fetch_add(acked + prevc, 1u, __ATOMIC_RELAXED, __HIP_MEMORY_SCOPE_WORKGROUP);
        __builtin_amdgcn_raw_buffer_store_b32(call, r_dst, vf + (prev && (rfl(old) & 1u) ? prev : OOB >> 1), 0, SYS);
        prev = 0;
      };
      for (;;) {
        const u32x4 e = full[js];
        const u32 key = e.x != cnt ? e.w : ~0u;
        u32 j = 0, bk = __builtin_amdgcn_readlane(key, 0);
#pragma unroll
        for (u32 i = 1; i < NLD; ++i) { const u32 k2 = __builtin_amdgcn_readlane(key, i); if (k2 < bk) { bk = k2; j = i; } }
        if (bk == ~0u) {
          if (prev) { WAIT_VM(0); flag(); }
          if (__all(e.x != cnt)) break;
          __builtin_amdgcn_s_sleep(1);
          continue;
        }
        u32x4 buf[4];
#pragma unroll
        for (u32 i = 0; i < 4; ++i) buf[i] = *(volatile LDSP u32x4*)(lds + j * 8192u + wid * 1024u + i * 2048u + vl);
        const u32 dl = __builtin_amdgcn_readlane(e.y, j) + wid * 1024u, fl = __builtin_amdgcn_readlane(e.z, j);
        const u32 gen = __builtin_amdgcn_readlane(e.x, j);
        if (js == j) ++cnt;
        WAIT_LGKM0();
        if (lane == 0u) freed[j * 4u + wid] = gen;
#define ST_(i, gap) do { __builtin_amdgcn_raw_buffer_store_b128(buf[i], r_dst, vl + dl + (i) * 2048u, 0, SYS); __builtin_amdgcn_s_sleep(gap); } while (0)
        WAIT_VM(3); ST_(0, 4); WAIT_VM(3); ST_(1, 4); WAIT_VM(2); ST_(2, 2); flag(); WAIT_VM(3); ST_(3, 1);
#undef ST_
        prev = fl; prevc = j * 4u + (gen & 3u);
      }
    }
    return;
  }

  const u32 kidx = li - NPUSH;
  const u32 ga = grp >> 1, gb = grp & 1u;
  const u32 lw = wid & 3u;
  u32 call = 0;
  if (wid == 4u) {
    call = rfl(ld_agt(my_ctr)) + 1u;
    if (lane == 0) ctrl[16] = call;
    WAIT_LGKM0();
  }

  u32 rt = (ga + 4u * (kidx >> 3)) & 15u, ct = 2u * (kidx & 7u) + gb, kind = rt < 4u ? 0u : 1u, fin = 0;
  for (int round = kidx < 24u ? 0 : 1; round < 2; ++round) {
    if (round == 1) {
      if (wid == 4u) {
        u32 t = 0;
        if (lane == 0) t = __hip_atomic_fetch_add((GLBP u32*)(me + F_TICKET + grp * 128u), 1u, __ATOMIC_RELAXED, __HIP_MEMORY_SCOPE_AGENT);
        t = rfl(t) - (call - 1u) * NGEMM;
        if (lane == 0) { ctrl[0] = t; st_agt(my_ctr, call); }
        if (lane >= 1u && lane < 16u) ctrl[lane] = 0;
        WAIT_LGKM0();
      }
      BAR();
      const u32 t = rfl(ctrl[0]);
      if (t >= 16u) return;
      fin = grp * 16u + t;
      kind = t < 8u ? 2u : 3u; rt = 12u + ga; ct = 2u * (t & 7u) + gb;
    }
    const u32 gcol = ((q >> 2) ^ (kind == 3u ? 1u : 0u)) * 2048u + ct * 128u;

    if (wid >= 4u) {
      const u32 remote = kind == 3u ? 0u : kind == 1u ? 63u & ~(4u << (rt & 3u)) : 63u;
      const u32 pvo = ((remote >> (lane >> 3)) & 1u) ? F_LINK + (((lane >> 3) + 2u) * 64u + ((lane & 4u) ? 16u + ct : rt)) * 128u + (lane & 3u) * 16u : OOB;
      const u32 lbase = (u32)(size_t)lds + L_CTRL;
      const u32 vs = lw * 1024u + lane * 16u;
      if (wid == 4u) {
        const u32 pace = (K->pace[rt * 4u + (ct >> 2)] >> ((ct & 3u) * 8u)) & 63u;
        if (remote && pace < 32u)
          wave_wait<16>(r_me, F_LINK + (2u * 64u + pace) * 128u, 1, lane, call, tmo);
        lds_wr32(lbase + 96u, 0);
        lds_wr32(lbase + 104u, 0);
      }
      BAR();
      u32 rem = remote, rdy = 0, snap = 0, sx = 0, sw = 0;
      auto pick = [&]() -> bool {
        const u32 av = rdy & rem;
        if (!av) return false;
        const u32 n = (u32)__builtin_ctz(av);
        rem &= ~(1u << n);
        sx = OFF_XS + (n * 16u + rt) * CH; sw = OFF_WS + (n * 16u + ct) * CH;
        return true;
      };
      auto choose = [&]() -> bool {
        if (!rem) return false;
        rdy |= lds_rd32(lbase + 96u);
        return pick();
      };
      auto issue = [&](u32 slot) {
        u32 dofs = slot * STAGE + lw * 1024u;
        asm volatile("" : "+s"(dofs));
        LDSP char* dst = lds + dofs;
#pragma unroll
        for (u32 i = 0; i < 4; ++i) {
          __builtin_amdgcn_raw_ptr_buffer_load_lds(r_me, dst + i * 4096u, 16, vs, sx + i * 4096u, 0, SYS);
          __builtin_amdgcn_raw_ptr_buffer_load_lds(r_me, dst + 16384u + i * 4096u, 16, vs, sw + i * 4096u, 0, SYS);
        }
        sx += 16384u; sw += 16384u;
      };
      auto shoot = [&]() {
        if (wid == 4u && rem && !(rem & rdy)) { __builtin_amdgcn_raw_ptr_buffer_load_lds(r_me, lds + L_SNAP, 16, pvo, 0, 0, SYS); snap = 1; }
      };
      auto publish = [&]() {
        if (snap) { lds_wr32(lbase + 96u, seg_mask(lds_rd128(lbase + 1024u + lane * 16u), call, lane)); snap = 0; }
      };
      {
        const rsrc_t r_x = mk_rsrc(gx, X_BYTES), r_w = mk_rsrc(gw, W_BYTES);
        const u32 vx = lw * 8192u + ((lw & 1u) ? vx1 : vx0), vw = lw * 32768u + vwp;
        const u32 grow = row_base(q, rt);
#pragma unroll 1
        for (u32 s8 = 0; s8 < 8; ++s8) {
          if (s8 == 4u) { WAIT_VM(24); BAR(); }
          if (s8 >= 4u) { WAIT_VM(16); BAR(); }
          if (s8 == 5u) shoot();
          LDSP char* dst = lds + (s8 & 3u) * STAGE + lw * 1024u;
          const u32 ox = grow * 1024u + s8 * 128u, ow = s8 * 64u * 8192u + gcol * 2u;
#pragma unroll
          for (u32 i = 0; i < 4; ++i) {
            __builtin_amdgcn_raw_ptr_buffer_load_lds(r_x, dst + i * 4096u, 16, vx, ox + i * 32768u, 0, 0);
            __builtin_amdgcn_raw_ptr_buffer_load_lds(r_w, dst + 16384u + i * 4096u, 16, vw, ow + i * 131072u, 0, 0);
          }
        }
      }
      for (;;) {
        for (;;) {
          const bool pre = pick();
          WAIT_VM(16); publish(); BAR();
          if (pre || choose()) {
            issue(0); WAIT_VM(16); BAR();
            shoot(); issue(1); WAIT_VM(16); BAR();
          } else {
            shoot(); WAIT_VM(8); BAR();
            WAIT_VM(0); publish(); BAR();
            if (!choose()) break;
            shoot();
#pragma unroll 1
            for (u32 i = 0; i < 2; ++i) issue(i);
          }
          issue(2); WAIT_VM(16); BAR();
          issue(3);
        }
        if (wid == 4u) lds_wr32(lbase + 104u, rem ? 1u : 2u);
        BAR();
        if (!rem) break;
        if (wid == 4u) {
          const u64 t0 = __builtin_amdgcn_s_memrealtime();
          u32 sm;
          for (;;) {
            const u32x4 v = poll128(r_me, lane == 63u ? F_TMO : pvo);
            sm = seg_mask(v, call, lane);
            if (sm & rem) break;
            sm = 63u;
            if (__builtin_amdgcn_readlane(v.x, 63) == call) break;
            if (__builtin_amdgcn_s_memrealtime() - t0 > DEADLINE) { st_agt(tmo, call); break; }
            __builtin_amdgcn_s_sleep(2);
          }
          lds_wr32(lbase + 96u, sm);
        }
        BAR();
        if (wid == 4u) lds_wr32(lbase + 104u, 0);
        choose();
#pragma unroll 1
        for (u32 i = 0; i < 4; ++i) issue(i);
        WAIT_VM(24); BAR();
      }
      if (kind >= 2u) {
        const u32 callw = rfl(ctrl[16]);
        const u64 t0 = __builtin_amdgcn_s_memrealtime();
        volatile LDSP u32x2* const res = (volatile LDSP u32x2*)(lds + L_CTRL + 32u + lw * 8u);
        u32 o0 = F_AMAX + (lw * 64u + lane) * 16u, o1 = o0 + 4096u, m = 0, bad = 0, seen = 0;
        for (;;) {
          const u32x4 g0 = poll128(r_me, o0), g1 = poll128(r_me, o1);
          const u32 tv = poll32(r_me, lane == 0u ? F_TMO : OOB);
          if (g0.y == callw && g0.w == callw) { m = max(m, max(g0.x, g0.z)); o0 = OOB; }
          if (g1.y == callw && g1.w == callw) { m = max(m, max(g1.x, g1.z)); o1 = OOB; }
          if (rfl(tv) == callw) { bad = 1; break; }
          const u32 missing = (u32)__builtin_popcountll(__ballot((o0 & o1) != OOB));
          if (missing == 0u) break;
          if (missing <= 36u && missing != seen) {
            const u32 pm = wave_max(m);
            if (lane == 0) *res = u32x2{pm, 1u};
            seen = missing;
          }
          if (__builtin_amdgcn_s_memrealtime() - t0 > DEADLINE) { st_agt(tmo, callw); bad = 1; break; }
          if (missing > 24u) __builtin_amdgcn_s_sleep(40);
          else if (missing > 8u) __builtin_amdgcn_s_sleep(10);
          else __builtin_amdgcn_s_sleep(1);
        }
        m = wave_max(m);
        if (lane == 0) *res = u32x2{m, 2u + bad};
      }
    } else {
      const u32 wn = wid & 1u, wm = wid >> 1;
      const u32 hh = lane >> 5, r31 = lane & 31u;
      const u32 z = (lane >> 4) & 1u, tq = (lane >> 2) & 3u, tp = lane & 3u;
      const u32 aw = 16384u + hh * 2048u + wn * 512u + tq * 64u + (16u * (tp & 1u) + 8u * z + 4u * (tp >> 1)) * 2u;
      const u32 axb = (wm * 64u + r31) * 128u, sw7 = (r31 >> 1) & 7u;
      u32 ax[4];
#pragma unroll
      for (u32 s = 0; s < 4; ++s) ax[s] = axb + (((2u * s + hh) ^ sw7) << 4);
      f32x16 acc[2][2] = {};
      s16x8 fa[4][2] = {}, fb[4][2] = {};
      s16x4 alo = {};
      auto tr = [&](u32 off) -> s16x4 { return __builtin_amdgcn_ds_read_tr16_b64_v4i16((LDSP s16x4*)(lds + off)); };
      auto rd0 = [&](u32 so, int ks) {
        const s16x4 lo = tr(so + aw + ks * 4096), hi = tr(so + aw + ks * 4096 + 1024);
        fa[ks][0] = __builtin_shufflevector(lo, hi, 0, 1, 2, 3, 4, 5, 6, 7);
        fb[ks][0] = *(LDSP s16x8*)(lds + so + ax[ks]);
      };
      auto rd1 = [&](u32 so, int ks) { fb[ks][1] = *(LDSP s16x8*)(lds + so + ax[ks] + 4096); alo = tr(so + aw + ks * 4096 + 256); };
      auto rd2 = [&](u32 so, int ks) { fa[ks][1] = __builtin_shufflevector(alo, tr(so + aw + ks * 4096 + 256 + 1024), 0, 1, 2, 3, 4, 5, 6, 7); };
      auto mm = [&](int ks, int fn, int fm) {
        acc[fn][fm] = __builtin_amdgcn_mfma_f32_32x32x16_bf16(__builtin_bit_cast(bf16x8, fa[ks][fn]), __builtin_bit_cast(bf16x8, fb[ks][fm]), acc[fn][fm], 0, 0, 0);
        SCHED();
      };
      u32 statv = 0;
      auto kstep = [&](int kc, u32 so, int kn, int sync) {
        mm(kc, 0, 0);
        if (sync) {
          mm(kc, 0, 1);
          WAIT_LGKM0(); BAR(); asm volatile("" ::: "memory");
          if (sync == 2) statv = ctrl[26];
        }
        rd0(so, kn); SCHED();
        if (!sync) { mm(kc, 0, 1); rd1(so, kn); SCHED(); }
        mm(kc, 1, 0);
        if (sync) rd1(so, kn);
        rd2(so, kn); SCHED();
        mm(kc, 1, 1);
      };
      BAR();
      __builtin_amdgcn_s_waitcnt(0x0F70);
      for (;;) {
        BAR();
        asm volatile("" ::: "memory");
#pragma unroll
        for (int k = 0; k < 2; ++k) { rd0(0, k); rd1(0, k); rd2(0, k); SCHED(); }
        u32 status;
        do {
#pragma unroll
          for (int st = 0; st < 4; ++st)
#pragma unroll
            for (int k = 0; k < 4; ++k)
              kstep(k, (k < 2 ? st : (st + 1) & 3) * STAGE, (k + 2) & 3, k == 2 ? (st == 3 ? 2 : 1) : 0);
          status = rfl(statv);
        } while (status == 0u);
        if (status == 2u) break;
        BAR();
      }
      const u32 vy = wid * 8192u + lane * 16u;
      const u32 callc = rfl(ctrl[16]);
      if (kind < 2u) {
        const u32 ti = kind == 0u ? rt * 16u + ct : 64u + (rt & 3u) * 32u + ((rt - 4u) >> 2) * 16u + ct;
#pragma unroll
        for (int f = 0; f < 4; ++f)
#pragma unroll
          for (int hf = 0; hf < 2; ++hf) {
            u32x4 v;
#pragma unroll
            for (int e = 0; e < 4; ++e) {
              const f32x2 p = {acc[f >> 1][f & 1][hf * 8 + e * 2], acc[f >> 1][f & 1][hf * 8 + e * 2 + 1]};
              v[e] = __builtin_bit_cast(u32, __builtin_convertvector(p, bf16x2));
            }
            __builtin_amdgcn_raw_buffer_store_b128(v, r_me, vy + OFF_YOUT + ti * YT + (f * 2 + hf) * 1024, 0, SYS);
          }
        WAIT_VM(0);
        if (lane == 0u) st_sys(me + F_YRDY + ti * 16u + wid * 4u, callc);
      } else {
        char* const gdst = K->peer[lane & 7u] + F_AMAX + (q * 128u + fin) * 8u;
        const u32 d0 = (q ^ (((q >> 2) ^ 1u) * 4u + (ga >> 1) * 2u)) & 7u, d1 = d0 ^ 1u;
        const u32 ty = (ga & 1u) * 16u + ct;
        const u32 so0 = OFF_YIN + (kind == 2u ? ga * 16u + ct : d0 * 32u - 64u + ty) * YT, so1 = OFF_YIN + (d1 * 32u - 64u + ty) * YT;
        const u32 fo0 = F_LINK + (kind == 2u ? 64u + ga * 16u + ct : d0 * 64u + 32u + ty) * 128u, fo1 = F_LINK + (d1 * 64u + 32u + ty) * 128u;
        const u32 po = lane == 63u ? F_TMO : lane < 16u ? ((lane & 8u) ? fo1 : fo0) + (lane & 7u) * 4u : OOB;
        const rsrc_t r_out = mk_rsrc(K->out, OUT_BYTES);
        const u32 vo = (ga * 128u + wm * 64u + (lane >> 2)) * 4096u + gcol + wn * 64u + (lane & 3u) * 16u;
        const u64 t0 = __builtin_amdgcn_s_memrealtime();
        auto poll = [&](u32 todo, u32 pv) {
          u32 go;
          for (;; pv = poll32(r_me, po)) {
            const u32 okm = (u32)__ballot((int)(pv - callc) >= 0), own = okm >> (wid * 2u);
            go = (((own & 3u) == 3u ? 1u : 0u) | ((own & 0x300u) == 0x300u ? 2u : 0u)) & todo;
            if (go) break;
            go = todo;
            if (__builtin_amdgcn_readlane(pv, 63) == callc) break;
            if (__builtin_amdgcn_s_memrealtime() - t0 > DEADLINE) { st_agt(tmo, callc); break; }
            if (okm & (((todo & 1u) ? 0xFFu : 0u) | ((todo & 2u) ? 0xFF00u : 0u))) __builtin_amdgcn_s_sleep(1);
            else __builtin_amdgcn_s_sleep(12);
          }
          asm volatile("" ::: "memory");
          return go;
        };
        auto fetch = [&](u32x4 (&v)[8], u32 so) {
#pragma unroll
          for (int j = 0; j < 8; ++j) v[j] = __builtin_amdgcn_raw_buffer_load_b128(r_me, vy + j * 1024, so, SYS);
        };
        auto add = [&](const u32x4 (&v)[8]) {
#pragma unroll
          for (int j = 0; j < 8; ++j)
#pragma unroll
            for (int e = 0; e < 4; ++e) {
              acc[j >> 2][(j >> 1) & 1][(j & 1) * 8 + e * 2] += __builtin_bit_cast(float, v[j][e] << 16);
              acc[j >> 2][(j >> 1) & 1][(j & 1) * 8 + e * 2 + 1] += __builtin_bit_cast(float, v[j][e] & 0xFFFF0000u);
            }
        };
        u32x4 va[8], vb[8];
        const u32 go = poll(kind == 2u ? 1u : 3u, poll32(r_me, po));
        fetch(va, (go & 1u) ? so0 : so1);
        fetch(vb, go == 3u ? so1 : OOB);
        const u32 pv2 = poll32(r_me, po);
        add(va);
        if (kind == 3u && go != 3u) { poll(go ^ 3u, pv2); fetch(vb, (go & 1u) ? so1 : so0); }
        if (kind == 3u) add(vb);
        float mx = 0.f;
#pragma unroll
        for (int f = 0; f < 4; ++f)
#pragma unroll
          for (int e = 0; e < 16; ++e) mx = fmaxf(mx, acc[f >> 1][f & 1][e]);
        const u32 wmax = wave_max(__builtin_bit_cast(u32, mx) & 0x7FFFFFFFu);
        u32 old = 0;
        if (lane == 0) {
          ctrl[4 + wid] = wmax;
          old = __hip_atomic_fetch_add((LDSP u32*)(lds + L_CTRL) + 1, 1u, __ATOMIC_RELAXED, __HIP_MEMORY_SCOPE_WORKGROUP);
        }
        if (rfl(old) == 3u) {
          const u32x4 m4 = *(volatile LDSP u32x4*)(lds + L_CTRL + 16);
          const u32 m = max(max(m4.x, m4.y), max(m4.z, m4.w));
          if (lane < 8u) __hip_atomic_store((GLBP u64*)gdst, ((u64)callc << 32) | m, __ATOMIC_RELAXED, __HIP_MEMORY_SCOPE_SYSTEM);
        }
        u32 used = ~0u;
        for (;;) {
          const u32x4 a4 = *(volatile LDSP u32x4*)(lds + L_CTRL + 32), b4 = *(volatile LDSP u32x4*)(lds + L_CTRL + 48);
          const u32 lo = rfl(min(min(a4.y, a4.w), min(b4.y, b4.w))), hi = rfl(max(max(a4.y, a4.w), max(b4.y, b4.w)));
          if (lo == 0u) { __builtin_amdgcn_s_sleep(2); continue; }
          const u32 am = rfl(max(max(max(a4.x, a4.z), max(b4.x, b4.z)), wmax));
          const bool bad = hi == 3u;
          if (am != used || bad) {
            if (used != ~0u) WAIT_VM(0);
            used = am;
            const float amax = __builtin_bit_cast(float, am);
            const float scale = amax / 127.0f;
            const float inv = amax > 0.f ? 1.0f / scale : 0.f;
#pragma unroll
            for (int f = 0; f < 4; ++f) {
              u32x4 v;
#pragma unroll
              for (int e = 0; e < 4; ++e) {
                u32 pk = 0;
#pragma unroll
                for (int b = 0; b < 4; ++b) {
                  const float y = __builtin_amdgcn_fmed3f(__builtin_rintf(acc[f >> 1][f & 1][e * 4 + b] * inv), 0.f, 127.f);
                  pk = __builtin_amdgcn_cvt_pk_u8_f32(y, b, pk);
                }
                v[e] = bad ? 0x80808080u : pk;
              }
              *(LDSP u32x4*)(lds + wid * 4096u + ((f & 1) * 32u + r31) * 64u + ((((f >> 1) * 2u + hh) ^ ((r31 >> 1) & 3u)) << 4)) = v;
            }
#pragma unroll
            for (u32 i = 0; i < 4; ++i) {
              const u32x4 v = *(LDSP u32x4*)(lds + wid * 4096u + (i * 16u + (lane >> 2)) * 64u + (((lane & 3u) ^ ((lane >> 3) & 3u)) << 4));
              __builtin_amdgcn_raw_buffer_store_b128(v, r_out, vo + i * 16u * 4096u, 0, 16);
            }
          }
          if (lo >= 2u) break;
          __builtin_amdgcn_s_sleep(1);
        }
      }
    }
  }
}

namespace {
const char* const kCross = "T0 w0 T1 w1 T2 w2 T3 w3 w4 X0 w5 X1 w6 X2 X3 w7 X4 X5 w8 X6 X7 w11 w9 w12 w14 w10 w15 w13 O0 O1 O2 O3";
const char* const kSib = "T0 w0 T1 w1 T2 w2 T3 w3 w4 X0 w5 X1 w6 X2 w7 X3 X4 w8 X5 X6 X7 w9 w10 f11 f12 w11 f14 f15 w12 w13 w14 w15 "
                         "O0 O1 O2 O3 a12 a13 b12 b13";

int parse_order(const char* p, int* v) {
  int n = 0;
  while (*p && n < 48) {
    const char c = *p++;
    int k = 0;
    while (*p >= '0' && *p <= '9') k = k * 10 + (*p++ - '0');
    while (*p == ' ') ++p;
    v[n++] = (c == 'T' ? 0 : c == 'X' ? 4 : c == 'O' ? 12 : c == 'w' ? 16 : c == 'f' ? 32 : c == 'a' ? 48 : 64) + k;
  }
  return n;
}

void build_tables(Params& P, int rank) {
  int cs[48], ss[48];
  const int nc = parse_order(kCross, cs), ns = parse_order(kSib, ss);
  u32 split = 0, relay = 0;
  for (int i = 0; i < ns; ++i) { if ((ss[i] >> 4) == 2) split |= 1u << (ss[i] & 15); if ((ss[i] >> 4) == 3) relay |= 1u << (ss[i] & 15); }
  const double C = 1.125, LEAD = 10.0;
  double ct[4][32] = {}, sb[48] = {}, done[16][16];
  for (int l = 0; l < 4; ++l) {
    double t = 0;
    for (int i = 0; i < nc; ++i) {
      const int it = cs[i];
      if (it >= 4 && it < 12 && (it & 3) == l) continue;
      ct[l][it] = t += (it >= 16 && ((split >> (it - 16)) & 1u)) ? C / 2 : C;
    }
  }
  double t = 0;
  for (int i = 0; i < ns; ++i) {
    const int it = ss[i];
    if ((it >> 4) == 2) for (int l = 0; l < 4; ++l) t = std::max(t, ct[l][it - 16] + 2.2);
    t += C;
    if (it < 48) sb[it] = t;
  }
  unsigned char pace[16][16];
  for (int r = 0; r < 16; ++r)
    for (int c = 0; c < 16; ++c) {
      double sg[6]; int n = 0;
      sg[n++] = std::max(sb[r], sb[16 + c]); sg[n++] = sg[0];
      for (int l = 0; l < 4; ++l)
        if (!(r >= 4 && r < 12 && (r & 3) == l)) sg[n++] = std::max(std::max(ct[l][r], ct[l][16 + c]), sb[32 + c]);
      std::sort(sg, sg + n);
      double e = sg[1];
      for (int i = 0; i < n; ++i) e = std::max(e, sg[i] + (i >= 2 ? 0.9 : 0.0)) + 1.25;
      done[r][c] = e + 1e-3 * (c * 16 + r);
      pace[r][c] = 32;
      for (int i = 0; i < ns; ++i) if (ss[i] < 32 && sb[ss[i]] - C <= sg[1] - LEAD) pace[r][c] = (unsigned char)ss[i];
    }
  memcpy(P.pace, pace, sizeof pace);
  unsigned short tab[7][96] = {};
  for (int d = 1; d < 8; ++d) {
    int m = 0;
    auto put = [&](u32 kind, int pl) { if (m < 96) tab[d - 1][m++] = (unsigned short)(kind << 8 | (u32)pl); };
    const int tdst = ((rank ^ d) >> 1) & 1;
    for (int i = 0; i < (d >= 4 ? nc : d >= 2 ? ns : 0); ++i) {
      const int it = d >= 4 ? cs[i] : ss[i], k = it & 15;
      if (it < 32) {
        if (d >= 4 && it >= 4 && it < 12 && (it & 3) == d - 4) continue;
        for (int hf = 0; hf < 2; ++hf)
          if (!(d >= 4 && it >= 16 && ((split >> k) & 1u)) || hf == tdst) put(it < 16 ? E_X : E_W, k * 2 + hf);
      } else if (it < 48) { put(E_FW, k * 2); put(E_FW, k * 2 + 1); }
      else for (int r = (d ^ (it >= 64)) & 1; r < 4; r += 2) put(it >= 64 ? E_R2 : E_R1, r * 16 + k);
    }
    const int nt = d == 1 ? 64 : d >= 4 ? 32 : 0;
    int ord[64], no = 0;
    auto key = [&](int tl) { return done[d == 1 ? tl >> 4 : d + 4 * (tl >> 4)][tl & 15]; };
    for (int tl = 0; tl < nt; ++tl)
      if (d != 1 || !((relay >> (tl & 15)) & 1u)) ord[no++] = tl;
    std::sort(ord, ord + no, [&](int x, int y) { return key(x) < key(y); });
    if (d >= 4)
      std::sort(ord + no - 4, ord + no, [&](int x, int y) { return key(x) + 1e3 * ((x >> 4) ^ (rank & 1)) < key(y) + 1e3 * ((y >> 4) ^ (rank & 1)); });
    for (int i = 0; i < no; ++i) put(E_Y, d == 1 ? ord[i] : d * 32 - 64 + ord[i]);
    P.nent[d - 1] = (u32)m;
    for (int i = 0; i < 8 && d >= 2; ++i) {
      const u32 kind = tab[d - 1][i] >> 8, c = (tab[d - 1][i] & 255u) >> 1, u = (tab[d - 1][i] & 1u) * 8u;
      const u32 dst = (u32)(rank ^ d), h = k_half(dst, (u32)rank);
      P.first[d - 1][i] = u32x4{kind == E_X ? row_base(dst, c) * 1024u + h * 512u + (u >> 2) * 128u : (h * 256u + u * 16u) * 8192u + ((dst >> 2) * 2048u + c * 128u) * 2u,
                                (kind == E_X ? OFF_XS : OFF_WS) + ((d - 2u) * 16u + c) * CH + u * 4096u,
                                F_LINK + (d * 64u + kind * 16u + c) * 128u + u * 4u, kind};
    }
    for (u32 p0 = 0; p0 < 32; ++p0) {
      const u32 sub = (p0 & 3u) * 2u;
      const u32x4 e = P.first[d - 1][p0 >> 2];
      P.fso[(d - 1) * 8 + (p0 & 7u)][p0 >> 3] = (e.x + (e.w ? sub * 131072u : (sub >> 2) * 128u + (sub & 3u) * 32768u)) | e.w;
    }
  }
  memcpy(P.tab, tab, sizeof tab);
}
}

extern "C" void kernel_launch(int rank, int nranks, ncclComm_t comm, hipStream_t stream,
                              void* const* d_in, const int* in_sizes, int n_in,
                              void* d_out, int out_size, const int* in_dtypes, int out_dtype,
                              void* const* d_out_all, void* const* d_ws_all, size_t ws_bytes) {
  (void)comm; (void)in_dtypes; (void)out_dtype; (void)d_out_all;
  if (nranks != NR || (unsigned)rank >= (unsigned)NR || n_in < 2 || ws_bytes < WS_USED ||
      (size_t)in_sizes[0] * 2 != X_BYTES || (size_t)in_sizes[1] * 2 != W_BYTES || (u32)out_size != OUT_BYTES) {
    fprintf(stderr, "kernel_launch: unexpected problem shape\n");
    return;
  }
  Params P;
  memset(&P, 0, sizeof P);
  P.x = (const char*)d_in[0];
  P.w = (const char*)d_in[1];
  P.out = (char*)d_out;
  for (int p = 0; p < NR; ++p) P.peer[p] = (char*)d_ws_all[rank ^ p];
  P.rank = (u32)rank;
  build_tables(P, rank);
  hipError_t e = hipFuncSetAttribute((const void*)fused_kernel, hipFuncAttributeMaxDynamicSharedMemorySize, (int)LDS_BYTES);
  if (e != hipSuccess) { fprintf(stderr, "hipFuncSetAttribute: %s\n", hipGetErrorString(e)); return; }
  fused_kernel<<<dim3(256), dim3(512), LDS_BYTES, stream>>>(P);
  e = hipGetLastError();
  if (e != hipSuccess) fprintf(stderr, "launch: %s\n", hipGetErrorString(e));
}
```

```cpp
#include <hip/hip_runtime.h>
#include <rccl/rccl.h>
#include <algorithm>
#include <cstdio>
#include <cstring>

typedef unsigned int u32;
typedef unsigned long long u64;
typedef __attribute__((ext_vector_type(4))) unsigned int u32x4;
typedef __attribute__((ext_vector_type(2))) unsigned int u32x2;
typedef __attribute__((ext_vector_type(4))) short s16x4;
typedef __attribute__((ext_vector_type(8))) short s16x8;
typedef __attribute__((ext_vector_type(8))) __bf16 bf16x8;
typedef __attribute__((ext_vector_type(2))) __bf16 bf16x2;
typedef __attribute__((ext_vector_type(2))) float f32x2;
typedef __attribute__((ext_vector_type(16))) float f32x16;
#define LDSP __attribute__((address_space(3)))
#define GLBP __attribute__((address_space(1)))
#define CSTP __attribute__((address_space(4)))
typedef __amdgpu_buffer_rsrc_t rsrc_t;

namespace {

constexpr int NR = 8;
constexpr u32 X_BYTES = 4096u * 512u * 2u;
constexpr u32 W_BYTES = 512u * 4096u * 2u;
constexpr u32 OUT_BYTES = 512u * 4096u;
constexpr u32 CH = 65536;
constexpr u32 YT = 32768;
constexpr u32 F_CALL = 0;
constexpr u32 F_TMO = 16384;
constexpr u32 F_TICKET = F_TMO + 1024;
constexpr u32 F_UNIT = F_TICKET + 1024;
constexpr u32 F_AMAX = 20480;
constexpr u32 F_YRDY = 28672;
constexpr u32 F_LINK = 32768;
constexpr u32 OFF_XS = 1u << 20;
constexpr u32 OFF_WS = OFF_XS + 6 * 16 * CH;
constexpr u32 OFF_YOUT = OFF_WS + 6 * 16 * CH;
constexpr u32 OFF_YIN = OFF_YOUT + 192 * YT;
constexpr u32 OFF_REL = OFF_YIN + 192 * YT;
constexpr u32 WS_USED = OFF_REL + 128 * YT;
constexpr u32 OOB = 0x80000000u;
constexpr int SYS = 17;

constexpr u32 STAGE = 32768;
constexpr u32 L_CTRL = 4 * STAGE;
constexpr u32 L_SNAP = L_CTRL + 1024;
constexpr u32 LDS_BYTES = L_SNAP + 1024;
constexpr u64 DEADLINE = 50000000ull;
constexpr u32 NPUSH = 7;
constexpr u32 NGEMM = 32 - NPUSH;
constexpr u32 NLD = 3;

constexpr u32 PULL_FROM = 14;

enum : u32 { E_X = 0, E_W = 1, E_Y = 2, E_FW = 3, E_R1 = 4 };
struct Params {
  const char* x;
  const char* w;
  char* out;
  u32 rank, relay;
  char* peer[NR];
  u32 nent[8];
  u32 trig[8];
  u32 pad2[8];
  u32 tab[7][48];
  u32x4 first[8][8];
  u32x4 fso[64];
  u32 pace[64];
};

__device__ __forceinline__ rsrc_t mk_rsrc(const void* p, u32 bytes) {
  return __builtin_amdgcn_make_buffer_rsrc((void*)p, 0, bytes, 0x00020000);
}
__device__ __forceinline__ u32 rfl(u32 v) { return __builtin_amdgcn_readfirstlane(v); }
__device__ __forceinline__ u32 ld_agt(const char* p) { return __hip_atomic_load((const GLBP u32*)p, __ATOMIC_RELAXED, __HIP_MEMORY_SCOPE_AGENT); }
__device__ __forceinline__ void st_sys(char* p, u32 v) { __hip_atomic_store((GLBP u32*)p, v, __ATOMIC_RELAXED, __HIP_MEMORY_SCOPE_SYSTEM); }
__device__ __forceinline__ void st_agt(char* p, u32 v) { __hip_atomic_store((GLBP u32*)p, v, __ATOMIC_RELAXED, __HIP_MEMORY_SCOPE_AGENT); }
__device__ __forceinline__ u32 poll32(rsrc_t r, u32 off) { asm volatile("" ::: "memory"); return __builtin_amdgcn_raw_buffer_load_b32(r, off, 0, SYS); }
__device__ __forceinline__ u32x4 poll128(rsrc_t r, u32 off) { asm volatile("" ::: "memory"); return __builtin_amdgcn_raw_buffer_load_b128(r, off, 0, SYS); }
#define WAIT_VM(n) asm volatile("s_waitcnt vmcnt(" #n ")" ::: "memory")
#define WAIT_LGKM0() asm volatile("s_waitcnt lgkmcnt(0)" ::: "memory")
#define BAR() __builtin_amdgcn_s_barrier()
#define SCHED() __builtin_amdgcn_sched_barrier(0)

__host__ __device__ __forceinline__ u32 row_base(u32 q, u32 r) {
  if (r < 4) return 512u * (q ^ 1u) + 128u * r;
  if (r < 12) return 512u * (q ^ (4u + (r & 3u))) + 256u * ((q >> 1) & 1u) + 128u * ((r - 4u) >> 2);
  return 512u * q + 128u * (r - 12u);
}
__host__ __device__ __forceinline__ u32 k_half(u32 q, u32 p) { return (q ^ ((q >> 1) & p)) & 1u; }

__device__ __forceinline__ u32 lds_rd32(u32 addr) {
  u32 v;
  asm volatile("ds_read_b32 %0, %1\n\ts_waitcnt lgkmcnt(0)" : "=&v"(v) : "v"(addr) : "memory");
  return rfl(v);
}
__device__ __forceinline__ void lds_wr32(u32 addr, u32 v) {
  asm volatile("ds_write_b32 %0, %1\n\ts_waitcnt lgkmcnt(0)" :: "v"(addr), "v"(v) : "memory");
}
__device__ __forceinline__ u32x4 lds_rd128(u32 addr) {
  u32x4 v;
  asm volatile("ds_read_b128 %0, %1\n\ts_waitcnt lgkmcnt(0)" : "=&v"(v) : "v"(addr) : "memory");
  return v;
}
__device__ __forceinline__ u32 seg_mask(u32x4 v, u32 call, u32 lane) {
  const u64 b = __ballot((int)((v.x - call) | (v.y - call) | (v.z - call) | (v.w - call)) >= 0);
  return (u32)__ballot(lane < 6u && ((b >> ((lane & 7u) * 8u)) & 255u) == 255u);
}

__device__ __forceinline__ u32 wave_max(u32 v) {
  v = max(v, (u32)__builtin_amdgcn_update_dpp(0, (int)v, 0x111, 0xf, 0xf, false));
  v = max(v, (u32)__builtin_amdgcn_update_dpp(0, (int)v, 0x112, 0xf, 0xf, false));
  v = max(v, (u32)__builtin_amdgcn_update_dpp(0, (int)v, 0x114, 0xf, 0xf, false));
  v = max(v, (u32)__builtin_amdgcn_update_dpp(0, (int)v, 0x118, 0xf, 0xf, false));
  v = max(v, (u32)__builtin_amdgcn_update_dpp(0, (int)v, 0x142, 0xa, 0xf, false));
  v = max(v, (u32)__builtin_amdgcn_update_dpp(0, (int)v, 0x143, 0xc, 0xf, false));
  return __builtin_amdgcn_readlane(v, 63);
}

template <int SLP>
__device__ __forceinline__ void wave_wait(rsrc_t r, u32 off, u32 n, u32 lane, u32 call, char* tmo) {
  const u64 t0 = __builtin_amdgcn_s_memrealtime();
  const u32 o = lane == 63u ? F_TMO : lane < n ? off + lane * 4u : OOB;
  for (;;) {
    const u32 v = poll32(r, o);
    if (__all((int)(v - call) >= 0 || lane >= n)) break;
    if (__builtin_amdgcn_readlane(v, 63) == call) break;
    if (__builtin_amdgcn_s_memrealtime() - t0 > DEADLINE) { st_agt(tmo, call); break; }
    __builtin_amdgcn_s_sleep(SLP);
  }
  asm volatile("" ::: "memory");
}

}

extern "C" __global__ void __launch_bounds__(512, 2) fused_kernel(Params params_by_value) {
  (void)params_by_value;
  const CSTP Params* const K = (const CSTP Params*)__builtin_amdgcn_kernarg_segment_ptr();
  const u32 grp = blockIdx.x & 7u, li = blockIdx.x >> 3;
  const char* const gx = K->x;
  const char* const gw = K->w;
  const u32x4 fso = K->fso[blockIdx.x & 63u];
  char* const far = K->peer[(li + 1u) & 7u];
  SCHED();
  extern __shared__ __attribute__((aligned(16))) char smem[];
  LDSP char* lds = (LDSP char*)smem;
  LDSP char* const sink = lds + L_CTRL + 512u;
  {
    u64 pc; u32 code_bytes;
    asm volatile("s_getpc_b64 %0\nfused_kernel_here:\n\ts_mov_b32 %1, fused_kernel_mark-fused_kernel_here" : "=&s"(pc), "=&s"(code_bytes));
    __builtin_amdgcn_raw_ptr_buffer_load_lds(mk_rsrc((const void*)pc, li == 31u ? code_bytes : 0u), sink, 4, threadIdx.x * 128u, 0, 0, 0);
    SCHED();
  }
  volatile LDSP u32* ctrl = (volatile LDSP u32*)(lds + L_CTRL);
  const u32 tid = threadIdx.x, lane = tid & 63u, wid = rfl(tid >> 6);
  const u32 vx0 = (lane >> 3) * 1024u + (((lane & 7u) ^ (lane >> 4)) << 4), vx1 = vx0 ^ 64u;
  const u32 vwp = ((lane >> 2) & 3u) * 8192u + (32u * (lane >> 4) + 8u * (lane & 3u)) * 2u;
  const bool fast = li - 1u < 6u && wid < 2u + NLD;
  const u32 twv = tid ? (li & 7u) * 16u * CH + (tid - 1u) * (OFF_WS - OFF_XS) : 0u, twb = li >= 24u ? OFF_YOUT : 0u;
  const u32 vx01 = wid ? vx1 + 8192u : vx0, head = fast && wid < 2u;
  asm volatile("" :: "v"(twv), "s"(twb), "v"(vx01), "s"(head));
  SCHED();
  asm volatile("" :: "s"(gx), "s"(gw), "s"(fso), "s"(far));
  __builtin_amdgcn_raw_ptr_buffer_load_lds(mk_rsrc(far, twb), sink, 4, twv, 0, 0, SYS);
  if (head) {
    u32x4 b0[4];
#pragma unroll
    for (u32 i = 0; i < 4; ++i) b0[i] = __builtin_amdgcn_raw_buffer_load_b128(mk_rsrc(gx, X_BYTES), vx01, fso.x + i * 16384u, SYS);
    const u32 vd = OFF_XS + (li - 1u) * 16u * CH + grp * 8192u + tid * 16u;
#pragma unroll
    for (u32 i = 0; i < 4; ++i) {
      __builtin_amdgcn_raw_buffer_store_b128(b0[i], mk_rsrc(far, WS_USED), vd + i * 2048u, 0, SYS);
      if (i < 3) __builtin_amdgcn_s_sleep(2);
    }
  }
  if (fast && wid >= 2u) {
    const u32 f = wid == 2u ? fso.y : wid == 3u ? fso.z : fso.w;
    const bool isw = f & 1u;
    const rsrc_t rs = mk_rsrc((const void*)(isw ? (u64)gw : (u64)gx), X_BYTES);
    const u32 ve = isw ? vwp : vx0, vo = isw ? vwp + 32768u : vx1 + 8192u;
    u32 so = f & ~1u;
#pragma unroll 1
    for (u32 i = 0; i < 8192u; i += 2048u, so += isw ? 65536u : 16384u) {
      __builtin_amdgcn_raw_ptr_buffer_load_lds(rs, lds + (wid - 2u) * 8192u + i, 16, ve, so, 0, SYS);
      __builtin_amdgcn_raw_ptr_buffer_load_lds(rs, lds + (wid - 2u) * 8192u + i + 1024u, 16, vo, so, 0, SYS);
    }
    WAIT_VM(0);
    BAR();
  }
  if (wid == 7u && li == 31u)
    __builtin_amdgcn_raw_ptr_buffer_load_lds(mk_rsrc((const void*)K, sizeof(Params)), sink, 4, lane * 128u, 0, 0, 0);
  if (wid == 0u && li - 24u < 7u) {
    const u32 dd = li - 23u;
    __builtin_amdgcn_s_sleep(48);
    __builtin_amdgcn_raw_ptr_buffer_load_lds(mk_rsrc(far, WS_USED), sink, 4, dd >= 4u ? (lane < 2u ? (lane ? OFF_YOUT : OFF_YIN) + (dd * 32u - 64u) * YT : OOB)
        : dd == 1u ? (lane < 4u ? ((lane & 2u) ? OFF_YOUT : OFF_YIN) + ((lane & 1u) << 20) : OOB)
        : lane < 4u ? OFF_WS + (lane + 2u) * 16u * CH : lane < 8u ? OFF_REL + ((lane - 4u) << 20) : OOB, 0, 0, SYS);
    __builtin_amdgcn_raw_ptr_buffer_load_lds(mk_rsrc(K->peer[0], WS_USED), sink, 4, lane < 4u ? (dd * 4u - 3u + lane) << 20 : OOB, 0, 0, SYS);
  }
  const u32 q = K->rank & 7u;
  char* const me = K->peer[0];
  char* const tmo = me + F_TMO;
  char* const my_ctr = me + F_CALL + blockIdx.x * 64u;
  const rsrc_t r_me = mk_rsrc(me, WS_USED);

  if (li < NPUSH) {
    const u32 d = li + 1u;
    const u32 total = min(K->nent[li], 96u) * 4u;
    const u32 nstat = d == 1u ? NLD * 8u : NLD * 8u + 8u;
    const u32 vl = lane * 16u;
    volatile LDSP u32x4* const full = (volatile LDSP u32x4*)(lds + L_CTRL);
    volatile LDSP u32* const freed = (volatile LDSP u32*)(lds + L_CTRL + 128);
    LDSP u32* const acked = (LDSP u32*)(lds + L_CTRL + 256);
    auto get_call = [&]() { u32 c; while ((c = rfl(ctrl[127])) == 0u) __builtin_amdgcn_s_sleep(1); return c; };
    if (tid - 320u < 128u) ctrl[tid - 320u] = 0;
    if (wid == 5u && d != 1u) {
#pragma unroll
      for (u32 j = 0; j < NLD; ++j) {
        const u32 p0 = j * 8u + 8u + grp;
        const u32x4 fst = K->first[li][p0 >> 2];
        if (lane == 0u) full[j] = u32x4{1u, fst.y + (p0 & 3u) * 8192u, fst.z + (p0 & 3u) * 8u, p0};
      }
    }
    WAIT_LGKM0();
    if (!fast || wid < 2u) BAR();
    if (wid == 7u) {
      if (lane == 0u) { const u32 c = ld_agt(my_ctr) + 1u; ctrl[127] = c; st_agt(my_ctr, c); }
    } else if (wid - 2u < NLD) {
      const u32 j = wid - 2u;
      const u32 dst_rank = (q ^ d) & 7u;
      const u32 h = k_half(dst_rank, q);
      u32 pend = j * 8u + grp, n = fast ? 1u : 0u;
      const u32 call = get_call();
      for (;; ++n) {
        if (n) {
          while (!__all(freed[j * 4u + (lane & 1u)] == n)) __builtin_amdgcn_s_sleep(1);
          const u32 tkv = (u32)__builtin_amdgcn_raw_ptr_buffer_atomic_add_i32(1, r_me, lane == 0u ? F_UNIT + li * 128u : OOB, 0, 0);
          pend = rfl(tkv) - (call - 1u) * (total + NLD * 8u - nstat) + nstat;
        }
        if (pend >= total) break;
        const u32 it = pend >> 2, sub = (pend & 3u) * 2u;
        const u32 ent = K->tab[li][it >> 1] >> ((it & 1u) * 16u);
        const u32 kind = (ent >> 8) & 7u, pl = ent & 255u;
        const bool in = kind <= E_W;
        u32 so, dl, fl, st = 1024u, gate = 0, ngate = 2;
        if (in) {
          const u32 c = pl >> 1, u = (pl & 1u) * 8u + sub;
          fl = F_LINK + (d * 64u + kind * 16u + c) * 128u + u * 4u;
          if (kind == E_X) { so = row_base(dst_rank, c) * 1024u + h * 512u + (u >> 2) * 128u + (u & 3u) * 32768u; dl = OFF_XS; st = 8192u; }
          else { so = (h * 256u + u * 16u) * 8192u + ((dst_rank >> 2) * 2048u + c * 128u) * 2u; dl = OFF_WS; st = 32768u; }
          dl += ((d - 2u) * 16u + c) * CH + u * 4096u;
        } else if (kind == E_FW) {
          const u32 c = pl >> 1, u = ((q >> 1) & 1u) * 8u + sub;
          const u32 dq = q ^ (((q >> 2) ^ 1u) * 4u + (pl & 1u) * 2u + (d & 1u)), dr = dq ^ d;
          so = OFF_WS + ((dq - 2u) * 16u + c) * CH + u * 4096u; gate = F_LINK + (dq * 64u + 16u + c) * 128u + u * 4u;
          dl = OFF_WS + ((dr - 2u) * 16u + c) * CH + u * 4096u; fl = F_LINK + (dr * 64u + 16u + c) * 128u + u * 4u;
        } else {
          so = OFF_YOUT + pl * YT + sub * 4096u; gate = F_YRDY + pl * 16u + sub * 2u; ngate = 1;
          dl = (kind == E_R1 ? OFF_REL + (d & 1u) * 64u * YT : OFF_YIN) + pl * YT + sub * 4096u;
          fl = F_LINK + (pl < 64u ? 64u + pl : d * 64u + 32u + (pl & 31u)) * 128u + sub * 4u + (kind == E_R1 ? 1u : 0u);
        }
        if (!in) wave_wait<2>(r_me, gate, ngate, lane, call, tmo);
        const rsrc_t rs = mk_rsrc((const void*)(!in ? (u64)me : kind == E_X ? (u64)gx : (u64)gw), in ? X_BYTES : WS_USED);
        const u32 ve = !in ? vl : kind == E_X ? vx0 : vwp, vo = !in ? vl : kind == E_X ? vx1 : vwp;
#pragma unroll
        for (u32 i = 0; i < 8; ++i) __builtin_amdgcn_raw_ptr_buffer_load_lds(rs, lds + j * 8192u + i * 1024u, 16, (i & 1u) ? vo : ve, so + i * st, 0, SYS);
        WAIT_VM(0);
        if (lane == 0u) full[j] = u32x4{n + 1u, dl, fl, pend};
      }
      asm volatile("fused_kernel_mark:");
      if (lane == 0u) full[j] = u32x4{n + 1u, OOB, 0u, ~0u};
    } else if (wid < 2u) {
      const rsrc_t r_dst = mk_rsrc(far, WS_USED);
      const u64 afar = (u64)far, amate = (u64)K->peer[1];
      const u32 js = (lane & 7u) % NLD;
      const u32 vf = lane < 2u ? lane * 4u : OOB;
      u32 cnt = 0, call = 0, prev = fast ? F_LINK + d * 64u * 128u + grp * 8u : 0u, prevc = NLD * 4u;
      auto flag = [&]() {
        u32 old = 0;
        if (prev && !call) call = get_call();
        if (prev && lane == 0u) old = __hip_atomic_fetch_add(acked + prevc, 1u, __ATOMIC_RELAXED, __HIP_MEMORY_SCOPE_WORKGROUP);
        const u64 af = afar, am = amate;
        __builtin_amdgcn_raw_buffer_store_b32(call, mk_rsrc((const void*)((prev & 1u) ? am : af), WS_USED),
                                              vf + (prev && (rfl(old) & 1u) ? prev & ~1u : OOB >> 1), 0, SYS);
        prev = 0;
      };
      for (;;) {
        const u32x4 e = full[js];
        const u32 key = e.x != cnt ? e.w : ~0u;
        u32 j = 0, bk = __builtin_amdgcn_readlane(key, 0);
#pragma unroll
        for (u32 i = 1; i < NLD; ++i) { const u32 k2 = __builtin_amdgcn_readlane(key, i); if (k2 < bk) { bk = k2; j = i; } }
        if (bk == ~0u) {
          if (prev) { WAIT_VM(0); flag(); }
          if (__all(e.x != cnt)) break;
          __builtin_amdgcn_s_sleep(1);
          continue;
        }
        u32x4 buf[4];
#pragma unroll
        for (u32 i = 0; i < 4; ++i) buf[i] = *(volatile LDSP u32x4*)(lds + j * 8192u + wid * 1024u + i * 2048u + vl);
        const u32 dl = __builtin_amdgcn_readlane(e.y, j) + wid * 1024u, fl = __builtin_amdgcn_readlane(e.z, j);
        const u32 gen = __builtin_amdgcn_readlane(e.x, j);
        if (js == j) ++cnt;
        WAIT_LGKM0();
        if (lane == 0u) freed[j * 4u + wid] = gen;
#define ST_(i, gap) do { __builtin_amdgcn_raw_buffer_store_b128(buf[i], r_dst, vl + dl + (i) * 2048u, 0, SYS); __builtin_amdgcn_s_sleep(gap); } while (0)
        WAIT_VM(3); ST_(0, 4); WAIT_VM(3); ST_(1, 4); WAIT_VM(2); ST_(2, 2); flag(); WAIT_VM(3); ST_(3, 1);
#undef ST_
        prev = fl; prevc = j * 4u + (gen & 3u);
      }
    }
    return;
  }

  const u32 kidx = li - NPUSH;
  const u32 ga = grp >> 1, gb = grp & 1u;
  const u32 lw = wid & 3u;
  u32 call = 0;
  if (wid == 4u) {
    call = rfl(ld_agt(my_ctr)) + 1u;
    if (lane == 0) ctrl[16] = call;
    WAIT_LGKM0();
  }

  u32 rt = (ga + 4u * (kidx >> 3)) & 15u, ct = 2u * (kidx & 7u) + gb, kind = rt < 4u ? 0u : 1u, fin = 0;
  for (int round = kidx < 24u ? 0 : 1; round < 2; ++round) {
    if (round == 1) {
      if (wid == 4u) {
        u32 t = 0;
        if (lane == 0) t = __hip_atomic_fetch_add((GLBP u32*)(me + F_TICKET + grp * 128u), 1u, __ATOMIC_RELAXED, __HIP_MEMORY_SCOPE_AGENT);
        t = rfl(t) - (call - 1u) * NGEMM;
        if (lane == 0) { ctrl[0] = t; st_agt(my_ctr, call); }
        if (lane >= 1u && lane < 16u) ctrl[lane] = 0;
        WAIT_LGKM0();
      }
      BAR();
      const u32 t = rfl(ctrl[0]);
      if (t >= 16u) return;
      fin = grp * 16u + t;
      kind = t < 8u ? 2u : 3u; rt = 12u + ga; ct = 2u * (t & 7u) + gb;
    }
    const u32 gcol = ((q >> 2) ^ (kind == 3u ? 1u : 0u)) * 2048u + ct * 128u;

    if (wid >= 4u) {
      const u32 remote = kind == 3u ? 0u : kind == 1u ? 63u & ~(4u << (rt & 3u)) : 63u;
      const u32 pvo = ((remote >> (lane >> 3)) & 1u) ? F_LINK + (((lane >> 3) + 2u) * 64u + ((lane & 4u) ? 16u + ct : rt)) * 128u + (lane & 3u) * 16u : OOB;
      const u32 lbase = (u32)(size_t)lds + L_CTRL;
      const u32 vs = lw * 1024u + lane * 16u;
      if (wid == 4u) {
        const u32 pace = (K->pace[rt * 4u + (ct >> 2)] >> ((ct & 3u) * 8u)) & 63u;
        if (remote && pace < 32u)
          wave_wait<16>(r_me, F_LINK + (2u * 64u + pace) * 128u, 1, lane, call, tmo);
        lds_wr32(lbase + 96u, 0);
        lds_wr32(lbase + 104u, 0);
      }
      BAR();
      u32 rem = remote, rdy = 0, snap = 0, sx = 0, sw = 0;
      auto pick = [&]() -> bool {
        const u32 av = rdy & rem;
        if (!av) return false;
        const u32 n = (u32)__builtin_ctz(av);
        rem &= ~(1u << n);
        sx = OFF_XS + (n * 16u + rt) * CH; sw = OFF_WS + (n * 16u + ct) * CH;
        return true;
      };
      auto choose = [&]() -> bool {
        if (!rem) return false;
        rdy |= lds_rd32(lbase + 96u);
        return pick();
      };
      auto issue = [&](u32 slot) {
        u32 dofs = slot * STAGE + lw * 1024u;
        asm volatile("" : "+s"(dofs));
        LDSP char* dst = lds + dofs;
#pragma unroll
        for (u32 i = 0; i < 4; ++i) {
          __builtin_amdgcn_raw_ptr_buffer_load_lds(r_me, dst + i * 4096u, 16, vs, sx + i * 4096u, 0, SYS);
          __builtin_amdgcn_raw_ptr_buffer_load_lds(r_me, dst + 16384u + i * 4096u, 16, vs, sw + i * 4096u, 0, SYS);
        }
        sx += 16384u; sw += 16384u;
      };
      auto shoot = [&]() {
        if (wid == 4u && rem && !(rem & rdy)) { __builtin_amdgcn_raw_ptr_buffer_load_lds(r_me, lds + L_SNAP, 16, pvo, 0, 0, SYS); snap = 1; }
      };
      auto publish = [&]() {
        if (snap) { lds_wr32(lbase + 96u, seg_mask(lds_rd128(lbase + 1024u + lane * 16u), call, lane)); snap = 0; }
      };
      {
        const rsrc_t r_x = mk_rsrc(gx, X_BYTES), r_w = mk_rsrc(gw, W_BYTES);
        const u32 vx = lw * 8192u + ((lw & 1u) ? vx1 : vx0), vw = lw * 32768u + vwp;
        const u32 grow = row_base(q, rt);
#pragma unroll 1
        for (u32 s8 = 0; s8 < 8; ++s8) {
          if (s8 == 4u) { WAIT_VM(24); BAR(); }
          if (s8 >= 4u) { WAIT_VM(16); BAR(); }
          if (s8 == 5u) shoot();
          LDSP char* dst = lds + (s8 & 3u) * STAGE + lw * 1024u;
          const u32 ox = grow * 1024u + s8 * 128u, ow = s8 * 64u * 8192u + gcol * 2u;
#pragma unroll
          for (u32 i = 0; i < 4; ++i) {
            __builtin_amdgcn_raw_ptr_buffer_load_lds(r_x, dst + i * 4096u, 16, vx, ox + i * 32768u, 0, 0);
            __builtin_amdgcn_raw_ptr_buffer_load_lds(r_w, dst + 16384u + i * 4096u, 16, vw, ow + i * 131072u, 0, 0);
          }
        }
      }
      for (;;) {
        for (;;) {
          const bool pre = pick();
          WAIT_VM(16); publish(); BAR();
          if (pre || choose()) {
            issue(0); WAIT_VM(16); BAR();
            shoot(); issue(1); WAIT_VM(16); BAR();
          } else {
            shoot(); WAIT_VM(8); BAR();
            WAIT_VM(0); publish(); BAR();
            if (!choose()) break;
            shoot();
#pragma unroll 1
            for (u32 i = 0; i < 2; ++i) issue(i);
          }
          issue(2); WAIT_VM(16); BAR();
          issue(3);
        }
        if (wid == 4u) lds_wr32(lbase + 104u, rem ? 1u : 2u);
        BAR();
        if (!rem) break;
        if (wid == 4u) {
          const u64 t0 = __builtin_amdgcn_s_memrealtime();
          u32 sm;
          for (;;) {
            const u32x4 v = poll128(r_me, lane == 63u ? F_TMO : pvo);
            sm = seg_mask(v, call, lane);
            if (sm & rem) break;
            sm = 63u;
            if (__builtin_amdgcn_readlane(v.x, 63) == call) break;
            if (__builtin_amdgcn_s_memrealtime() - t0 > DEADLINE) { st_agt(tmo, call); break; }
            __builtin_amdgcn_s_sleep(2);
          }
          lds_wr32(lbase + 96u, sm);
        }
        BAR();
        if (wid == 4u) lds_wr32(lbase + 104u, 0);
        choose();
#pragma unroll 1
        for (u32 i = 0; i < 4; ++i) issue(i);
        WAIT_VM(24); BAR();
      }
      if (kind >= 2u) {
        const u32 callw = rfl(ctrl[16]);
        const u64 t0 = __builtin_amdgcn_s_memrealtime();
        volatile LDSP u32x2* const res = (volatile LDSP u32x2*)(lds + L_CTRL + 32u + lw * 8u);
        u32 o0 = F_AMAX + (lw * 64u + lane) * 16u, o1 = o0 + 4096u, m = 0, bad = 0, seen = 0;
        for (;;) {
          const u32x4 g0 = poll128(r_me, o0), g1 = poll128(r_me, o1);
          const u32 tv = poll32(r_me, lane == 0u ? F_TMO : OOB);
          if (g0.y == callw && g0.w == callw) { m = max(m, max(g0.x, g0.z)); o0 = OOB; }
          if (g1.y == callw && g1.w == callw) { m = max(m, max(g1.x, g1.z)); o1 = OOB; }
          if (rfl(tv) == callw) { bad = 1; break; }
          const u32 missing = (u32)__builtin_popcountll(__ballot((o0 & o1) != OOB));
          if (missing == 0u) break;
          if (missing <= 36u && missing != seen) {
            const u32 pm = wave_max(m);
            if (lane == 0) *res = u32x2{pm, 1u};
            seen = missing;
          }
          if (__builtin_amdgcn_s_memrealtime() - t0 > DEADLINE) { st_agt(tmo, callw); bad = 1; break; }
          if (missing > 24u) __builtin_amdgcn_s_sleep(40);
          else if (missing > 8u) __builtin_amdgcn_s_sleep(10);
          else __builtin_amdgcn_s_sleep(1);
        }
        m = wave_max(m);
        if (lane == 0) *res = u32x2{m, 2u + bad};
      }
    } else {
      const u32 wn = wid & 1u, wm = wid >> 1;
      const u32 hh = lane >> 5, r31 = lane & 31u;
      const u32 z = (lane >> 4) & 1u, tq = (lane >> 2) & 3u, tp = lane & 3u;
      const u32 aw = 16384u + hh * 2048u + wn * 512u + tq * 64u + (16u * (tp & 1u) + 8u * z + 4u * (tp >> 1)) * 2u;
      const u32 axb = (wm * 64u + r31) * 128u, sw7 = (r31 >> 1) & 7u;
      u32 ax[4];
#pragma unroll
      for (u32 s = 0; s < 4; ++s) ax[s] = axb + (((2u * s + hh) ^ sw7) << 4);
      f32x16 acc[2][2] = {};
      s16x8 fa[4][2] = {}, fb[4][2] = {};
      s16x4 alo = {};
      auto tr = [&](u32 off) -> s16x4 { return __builtin_amdgcn_ds_read_tr16_b64_v4i16((LDSP s16x4*)(lds + off)); };
      auto rd0 = [&](u32 so, int ks) {
        const s16x4 lo = tr(so + aw + ks * 4096), hi = tr(so + aw + ks * 4096 + 1024);
        fa[ks][0] = __builtin_shufflevector(lo, hi, 0, 1, 2, 3, 4, 5, 6, 7);
        fb[ks][0] = *(LDSP s16x8*)(lds + so + ax[ks]);
      };
      auto rd1 = [&](u32 so, int ks) { fb[ks][1] = *(LDSP s16x8*)(lds + so + ax[ks] + 4096); alo = tr(so + aw + ks * 4096 + 256); };
      auto rd2 = [&](u32 so, int ks) { fa[ks][1] = __builtin_shufflevector(alo, tr(so + aw + ks * 4096 + 256 + 1024), 0, 1, 2, 3, 4, 5, 6, 7); };
      auto mm = [&](int ks, int fn, int fm) {
        acc[fn][fm] = __builtin_amdgcn_mfma_f32_32x32x16_bf16(__builtin_bit_cast(bf16x8, fa[ks][fn]), __builtin_bit_cast(bf16x8, fb[ks][fm]), acc[fn][fm], 0, 0, 0);
        SCHED();
      };
      u32 statv = 0;
      auto kstep = [&](int kc, u32 so, int kn, int sync) {
        mm(kc, 0, 0);
        if (sync) {
          mm(kc, 0, 1);
          WAIT_LGKM0(); BAR(); asm volatile("" ::: "memory");
          if (sync == 2) statv = ctrl[26];
        }
        rd0(so, kn); SCHED();
        if (!sync) { mm(kc, 0, 1); rd1(so, kn); SCHED(); }
        mm(kc, 1, 0);
        if (sync) rd1(so, kn);
        rd2(so, kn); SCHED();
        mm(kc, 1, 1);
      };
      BAR();
      __builtin_amdgcn_s_waitcnt(0x0F70);
      for (;;) {
        BAR();
        asm volatile("" ::: "memory");
#pragma unroll
        for (int k = 0; k < 2; ++k) { rd0(0, k); rd1(0, k); rd2(0, k); SCHED(); }
        u32 status;
        do {
#pragma unroll
          for (int st = 0; st < 4; ++st)
#pragma unroll
            for (int k = 0; k < 4; ++k)
              kstep(k, (k < 2 ? st : (st + 1) & 3) * STAGE, (k + 2) & 3, k == 2 ? (st == 3 ? 2 : 1) : 0);
          status = rfl(statv);
        } while (status == 0u);
        if (status == 2u) break;
        BAR();
      }
      const u32 vy = wid * 8192u + lane * 16u;
      const u32 callc = rfl(ctrl[16]);
      if (kind < 2u) {
        const u32 ti = kind == 0u ? rt * 16u + ct : 64u + (rt & 3u) * 32u + ((rt - 4u) >> 2) * 16u + ct;
#pragma unroll
        for (int f = 0; f < 4; ++f)
#pragma unroll
          for (int hf = 0; hf < 2; ++hf) {
            u32x4 v;
#pragma unroll
            for (int e = 0; e < 4; ++e) {
              const f32x2 p = {acc[f >> 1][f & 1][hf * 8 + e * 2], acc[f >> 1][f & 1][hf * 8 + e * 2 + 1]};
              v[e] = __builtin_bit_cast(u32, __builtin_convertvector(p, bf16x2));
            }
            __builtin_amdgcn_raw_buffer_store_b128(v, r_me, vy + OFF_YOUT + ti * YT + (f * 2 + hf) * 1024, 0, SYS);
          }
        WAIT_VM(0);
        if (ct >= PULL_FROM) {
          const u32 dd = kind == 0u ? 1u : 4u + (rt & 3u);
          if (lane < 2u) st_sys(K->peer[dd] + F_LINK + (kind == 0u ? 64u + ti : dd * 64u + 32u + (ti & 31u)) * 128u + wid * 8u + lane * 4u, callc);
        } else if (lane == 0u) st_sys(me + F_YRDY + ti * 16u + wid * 4u, callc);
      } else {
        char* const gdst = K->peer[lane & 7u] + F_AMAX + (q * 128u + fin) * 8u;
        const u32 d0 = (q ^ (((q >> 2) ^ 1u) * 4u + (ga >> 1) * 2u)) & 7u, d1 = d0 ^ 1u;
        const u32 ty = (ga & 1u) * 16u + ct;
        const u32 so0 = OFF_YIN + (kind == 2u ? ga * 16u + ct : d0 * 32u - 64u + ty) * YT, so1 = OFF_YIN + (d1 * 32u - 64u + ty) * YT;
        const u32 fo0 = F_LINK + (kind == 2u ? 64u + ga * 16u + ct : d0 * 64u + 32u + ty) * 128u, fo1 = F_LINK + (d1 * 64u + 32u + ty) * 128u;
        const bool parked = kind == 2u && ((K->relay >> ct) & 1u);
        const bool pull = parked || ct >= PULL_FROM;
        const u32 tg0 = parked ? fo0 + wid * 8u : K->trig[kind == 2u ? 1u : d0], tg1 = K->trig[d1];
        const u32 po = lane == 63u ? F_TMO : lane < 16u ? ((lane & 8u) ? fo1 : fo0) + (lane & 7u) * 4u
                     : pull && lane < 18u ? ((lane & 1u) ? tg1 : tg0) : OOB;
        const rsrc_t r_out = mk_rsrc(K->out, OUT_BYTES);
        const u32 vo = (ga * 128u + wm * 64u + (lane >> 2)) * 4096u + gcol + wn * 64u + (lane & 3u) * 16u;
        const u64 t0 = __builtin_amdgcn_s_memrealtime();
        auto poll = [&](u32 todo, u32 pv) {
          u32 go;
          for (;; pv = poll32(r_me, po)) {
            const u32 okm = (u32)__ballot((int)(pv - callc) >= 0), own = okm >> (wid * 2u);
            go = (((own & 3u) == 3u ? 1u : 0u) | ((own & 0x300u) == 0x300u ? 2u : 0u)) & todo;
            if (pull && (go & (okm >> 16)) != todo) go = 0;
            if (go) break;
            go = todo;
            if (__builtin_amdgcn_readlane(pv, 63) == callc) break;
            if (__builtin_amdgcn_s_memrealtime() - t0 > DEADLINE) { st_agt(tmo, callc); break; }
            if (okm & (((todo & 1u) ? 0xFFu : 0u) | ((todo & 2u) ? 0xFF00u : 0u))) __builtin_amdgcn_s_sleep(1);
            else __builtin_amdgcn_s_sleep(12);
          }
          asm volatile("" ::: "memory");
          return go;
        };
        auto fetch = [&](u32x4 (&v)[8], u32 so) {
#pragma unroll
          for (int j = 0; j < 8; ++j) v[j] = __builtin_amdgcn_raw_buffer_load_b128(r_me, vy + j * 1024, so, SYS);
        };
        auto add = [&](const u32x4 (&v)[8]) {
#pragma unroll
          for (int j = 0; j < 8; ++j)
#pragma unroll
            for (int e = 0; e < 4; ++e) {
              acc[j >> 2][(j >> 1) & 1][(j & 1) * 8 + e * 2] += __builtin_bit_cast(float, v[j][e] << 16);
              acc[j >> 2][(j >> 1) & 1][(j & 1) * 8 + e * 2 + 1] += __builtin_bit_cast(float, v[j][e] & 0xFFFF0000u);
            }
        };
        u32x4 va[8], vb[8];
        const u32 go = poll(kind == 2u ? 1u : 3u, poll32(r_me, po));
        if (pull) {
          const rsrc_t ra = mk_rsrc(K->peer[parked ? 3u - (ga & 1u) : kind == 2u ? 1u : d0], WS_USED);
          const rsrc_t rb = mk_rsrc(K->peer[d1], kind == 3u ? WS_USED : 0u);
          const u32 sa = parked ? OFF_REL + ((ga & 1u) * 64u + ga * 16u + ct) * YT : so0 - (OFF_YIN - OFF_YOUT);
#pragma unroll
          for (int j = 0; j < 8; ++j) {
            if (j >= 3) WAIT_VM(4);
            va[j] = __builtin_amdgcn_raw_buffer_load_b128(ra, vy + j * 1024, sa, SYS);
            vb[j] = __builtin_amdgcn_raw_buffer_load_b128(rb, vy + j * 1024, so1 - (OFF_YIN - OFF_YOUT), SYS);
          }
        } else {
          fetch(va, (go & 1u) ? so0 : so1);
          fetch(vb, go == 3u ? so1 : OOB);
        }
        const u32 pv2 = poll32(r_me, po);
        add(va);
        if (kind == 3u && go != 3u) { poll(go ^ 3u, pv2); fetch(vb, (go & 1u) ? so1 : so0); }
        if (kind == 3u) add(vb);
        float mx = 0.f;
#pragma unroll
        for (int f = 0; f < 4; ++f)
#pragma unroll
          for (int e = 0; e < 16; ++e) mx = fmaxf(mx, acc[f >> 1][f & 1][e]);
        const u32 wmax = wave_max(__builtin_bit_cast(u32, mx) & 0x7FFFFFFFu);
        u32 old = 0;
        if (lane == 0) {
          ctrl[4 + wid] = wmax;
          old = __hip_atomic_fetch_add((LDSP u32*)(lds + L_CTRL) + 1, 1u, __ATOMIC_RELAXED, __HIP_MEMORY_SCOPE_WORKGROUP);
        }
        if (rfl(old) == 3u) {
          const u32x4 m4 = *(volatile LDSP u32x4*)(lds + L_CTRL + 16);
          const u32 m = max(max(m4.x, m4.y), max(m4.z, m4.w));
          if (lane < 8u) __hip_atomic_store((GLBP u64*)gdst, ((u64)callc << 32) | m, __ATOMIC_RELAXED, __HIP_MEMORY_SCOPE_SYSTEM);
        }
        u32 used = ~0u;
        for (;;) {
          const u32x4 a4 = *(volatile LDSP u32x4*)(lds + L_CTRL + 32), b4 = *(volatile LDSP u32x4*)(lds + L_CTRL + 48);
          const u32 lo = rfl(min(min(a4.y, a4.w), min(b4.y, b4.w))), hi = rfl(max(max(a4.y, a4.w), max(b4.y, b4.w)));
          if (lo == 0u) { __builtin_amdgcn_s_sleep(2); continue; }
          const u32 am = rfl(max(max(max(a4.x, a4.z), max(b4.x, b4.z)), wmax));
          const bool bad = hi == 3u;
          if (am != used || bad) {
            if (used != ~0u) WAIT_VM(0);
            used = am;
            const float amax = __builtin_bit_cast(float, am);
            const float scale = amax / 127.0f;
            const float inv = amax > 0.f ? 1.0f / scale : 0.f;
#pragma unroll
            for (int f = 0; f < 4; ++f) {
              u32x4 v;
#pragma unroll
              for (int e = 0; e < 4; ++e) {
                u32 pk = 0;
#pragma unroll
                for (int b = 0; b < 4; ++b) {
                  const float y = __builtin_amdgcn_fmed3f(__builtin_rintf(acc[f >> 1][f & 1][e * 4 + b] * inv), 0.f, 127.f);
                  pk = __builtin_amdgcn_cvt_pk_u8_f32(y, b, pk);
                }
                v[e] = bad ? 0x80808080u : pk;
              }
              *(LDSP u32x4*)(lds + wid * 4096u + ((f & 1) * 32u + r31) * 64u + ((((f >> 1) * 2u + hh) ^ ((r31 >> 1) & 3u)) << 4)) = v;
            }
#pragma unroll
            for (u32 i = 0; i < 4; ++i) {
              const u32x4 v = *(LDSP u32x4*)(lds + wid * 4096u + (i * 16u + (lane >> 2)) * 64u + (((lane & 3u) ^ ((lane >> 3) & 3u)) << 4));
              __builtin_amdgcn_raw_buffer_store_b128(v, r_out, vo + i * 16u * 4096u, 0, 16);
            }
          }
          if (lo >= 2u) break;
          __builtin_amdgcn_s_sleep(1);
        }
      }
    }
  }
}

namespace {
const char* const kCross = "T0 w0 T1 w1 T2 w2 T3 w3 w4 X0 w5 X1 w6 X2 X3 w7 X4 X5 w8 X6 X7 w11 w9 w12 w14 w10 w15 w13 O0 O1 O2 O3";
const char* const kSib = "T0 w0 T1 w1 T2 w2 T3 w3 w4 X0 w5 X1 w6 X2 w7 X3 X4 w8 X5 X6 X7 w9 w10 f11 f12 w11 f14 f15 w12 w13 w14 w15 "
                         "O0 O1 O2 O3 a12 a13";

int parse_order(const char* p, int* v) {
  int n = 0;
  while (*p && n < 48) {
    const char c = *p++;
    int k = 0;
    while (*p >= '0' && *p <= '9') k = k * 10 + (*p++ - '0');
    while (*p == ' ') ++p;
    v[n++] = (c == 'T' ? 0 : c == 'X' ? 4 : c == 'O' ? 12 : c == 'w' ? 16 : c == 'f' ? 32 : 48) + k;
  }
  return n;
}

void build_tables(Params& P, int rank) {
  int cs[48], ss[48];
  const int nc = parse_order(kCross, cs), ns = parse_order(kSib, ss);
  u32 split = 0, relay = 0;
  for (int i = 0; i < ns; ++i) { if ((ss[i] >> 4) == 2) split |= 1u << (ss[i] & 15); if ((ss[i] >> 4) == 3) relay |= 1u << (ss[i] & 15); }
  const double C = 1.125, LEAD = 10.0;
  double ct[4][32] = {}, sb[48] = {}, done[16][16];
  for (int l = 0; l < 4; ++l) {
    double t = 0;
    for (int i = 0; i < nc; ++i) {
      const int it = cs[i];
      if (it >= 4 && it < 12 && (it & 3) == l) continue;
      ct[l][it] = t += (it >= 16 && ((split >> (it - 16)) & 1u)) ? C / 2 : C;
    }
  }
  double t = 0;
  for (int i = 0; i < ns; ++i) {
    const int it = ss[i];
    if ((it >> 4) == 2) for (int l = 0; l < 4; ++l) t = std::max(t, ct[l][it - 16] + 2.2);
    t += C;
    if (it < 48) sb[it] = t;
  }
  unsigned char pace[16][16];
  for (int r = 0; r < 16; ++r)
    for (int c = 0; c < 16; ++c) {
      double sg[6]; int n = 0;
      sg[n++] = std::max(sb[r], sb[16 + c]); sg[n++] = sg[0];
      for (int l = 0; l < 4; ++l)
        if (!(r >= 4 && r < 12 && (r & 3) == l)) sg[n++] = std::max(std::max(ct[l][r], ct[l][16 + c]), sb[32 + c]);
      std::sort(sg, sg + n);
      double e = sg[1];
      for (int i = 0; i < n; ++i) e = std::max(e, sg[i] + (i >= 2 ? 0.9 : 0.0)) + 1.25;
      done[r][c] = e + 1e-3 * (c * 16 + r);
      pace[r][c] = 32;
      for (int i = 0; i < ns; ++i) if (ss[i] < 32 && sb[ss[i]] - C <= sg[1] - LEAD) pace[r][c] = (unsigned char)ss[i];
    }
  memcpy(P.pace, pace, sizeof pace);
  P.relay = relay;
  unsigned short tab[7][96] = {};
  for (int d = 1; d < 8; ++d) {
    int m = 0;
    auto put = [&](u32 kind, int pl) { if (m < 96) tab[d - 1][m++] = (unsigned short)(kind << 8 | (u32)pl); };
    const int tdst = ((rank ^ d) >> 1) & 1;
    for (int i = 0; i < (d >= 4 ? nc : d >= 2 ? ns : 0); ++i) {
      const int it = d >= 4 ? cs[i] : ss[i], k = it & 15;
      if (it < 32) {
        if (d >= 4 && it >= 4 && it < 12 && (it & 3) == d - 4) continue;
        for (int hf = 0; hf < 2; ++hf)
          if (!(d >= 4 && it >= 16 && ((split >> k) & 1u)) || hf == tdst) put(it < 16 ? E_X : E_W, k * 2 + hf);
      } else if (it < 48) { put(E_FW, k * 2); put(E_FW, k * 2 + 1); }
      else for (int r = d & 1; r < 4; r += 2) put(E_R1, r * 16 + k);
    }
    const int nt = d == 1 ? 64 : d >= 4 ? 32 : 0;
    int ord[64], no = 0;
    auto key = [&](int tl) { return done[d == 1 ? tl >> 4 : d + 4 * (tl >> 4)][tl & 15]; };
    for (int tl = 0; tl < nt; ++tl)
      if ((u32)(tl & 15) < PULL_FROM && (d != 1 || !((relay >> (tl & 15)) & 1u))) ord[no++] = tl;
    std::sort(ord, ord + no, [&](int x, int y) { return key(x) < key(y); });
    for (int side = 1; side >= 0 && no; --side) {
      const int e = (rank ^ (side ? d : 0)) & 1;
      if (d >= 4)
        std::sort(ord + no - 4, ord + no, [&](int x, int y) { return key(x) + 1e3 * ((x >> 4) ^ e) < key(y) + 1e3 * ((y >> 4) ^ e); });
      if (side) P.trig[d] = F_LINK + (u32)(d == 1 ? 64 + ord[no - 4] : d * 64 + 32 + ord[no - 4]) * 128u;
    }
    for (int i = 0; i < no; ++i) put(E_Y, d == 1 ? ord[i] : d * 32 - 64 + ord[i]);
    P.nent[d - 1] = (u32)m;
    for (int i = 0; i < 8 && d >= 2; ++i) {
      const u32 kind = tab[d - 1][i] >> 8, c = (tab[d - 1][i] & 255u) >> 1, u = (tab[d - 1][i] & 1u) * 8u;
      const u32 dst = (u32)(rank ^ d), h = k_half(dst, (u32)rank);
      P.first[d - 1][i] = u32x4{kind == E_X ? row_base(dst, c) * 1024u + h * 512u + (u >> 2) * 128u : (h * 256u + u * 16u) * 8192u + ((dst >> 2) * 2048u + c * 128u) * 2u,
                                (kind == E_X ? OFF_XS : OFF_WS) + ((d - 2u) * 16u + c) * CH + u * 4096u,
                                F_LINK + (d * 64u + kind * 16u + c) * 128u + u * 4u, kind};
    }
    for (u32 p0 = 0; p0 < 32; ++p0) {
      const u32 sub = (p0 & 3u) * 2u;
      const u32x4 e = P.first[d - 1][p0 >> 2];
      P.fso[(d - 1) * 8 + (p0 & 7u)][p0 >> 3] = (e.x + (e.w ? sub * 131072u : (sub >> 2) * 128u + (sub & 3u) * 32768u)) | e.w;
    }
  }
  memcpy(P.tab, tab, sizeof tab);
}
}

extern "C" void kernel_launch(int rank, int nranks, ncclComm_t comm, hipStream_t stream,
                              void* const* d_in, const int* in_sizes, int n_in,
                              void* d_out, int out_size, const int* in_dtypes, int out_dtype,
                              void* const* d_out_all, void* const* d_ws_all, size_t ws_bytes) {
  (void)comm; (void)in_dtypes; (void)out_dtype; (void)d_out_all;
  if (nranks != NR || (unsigned)rank >= (unsigned)NR || n_in < 2 || ws_bytes < WS_USED ||
      (size_t)in_sizes[0] * 2 != X_BYTES || (size_t)in_sizes[1] * 2 != W_BYTES || (u32)out_size != OUT_BYTES) {
    fprintf(stderr, "kernel_launch: unexpected problem shape\n");
    return;
  }
  Params P;
  memset(&P, 0, sizeof P);
  P.x = (const char*)d_in[0];
  P.w = (const char*)d_in[1];
  P.out = (char*)d_out;
  for (int p = 0; p < NR; ++p) P.peer[p] = (char*)d_ws_all[rank ^ p];
  P.rank = (u32)rank;
  build_tables(P, rank);
  hipError_t e = hipFuncSetAttribute((const void*)fused_kernel, hipFuncAttributeMaxDynamicSharedMemorySize, (int)LDS_BYTES);
  if (e != hipSuccess) { fprintf(stderr, "hipFuncSetAttribute: %s\n", hipGetErrorString(e)); return; }
  fused_kernel<<<dim3(256), dim3(512), LDS_BYTES, stream>>>(P);
  e = hipGetLastError();
  if (e != hipSuccess) fprintf(stderr, "launch: %s\n", hipGetErrorString(e));
}
```

```cpp
#include <hip/hip_runtime.h>
#include <rccl/rccl.h>
#include <algorithm>
#include <cstdio>
#include <cstring>

typedef unsigned int u32;
typedef unsigned long long u64;
typedef __attribute__((ext_vector_type(4))) unsigned int u32x4;
typedef __attribute__((ext_vector_type(2))) unsigned int u32x2;
typedef __attribute__((ext_vector_type(4))) short s16x4;
typedef __attribute__((ext_vector_type(8))) short s16x8;
typedef __attribute__((ext_vector_type(8))) __bf16 bf16x8;
typedef __attribute__((ext_vector_type(2))) __bf16 bf16x2;
typedef __attribute__((ext_vector_type(2))) float f32x2;
typedef __attribute__((ext_vector_type(16))) float f32x16;
#define LDSP __attribute__((address_space(3)))
#define GLBP __attribute__((address_space(1)))
#define CSTP __attribute__((address_space(4)))
typedef __amdgpu_buffer_rsrc_t rsrc_t;

namespace {

constexpr int NR = 8;
constexpr u32 X_BYTES = 4096u * 512u * 2u;
constexpr u32 W_BYTES = 512u * 4096u * 2u;
constexpr u32 OUT_BYTES = 512u * 4096u;
constexpr u32 CH = 65536;
constexpr u32 YT = 32768;
constexpr u32 F_CALL = 0;
constexpr u32 F_TMO = 16384;
constexpr u32 F_TICKET = F_TMO + 1024;
constexpr u32 F_UNIT = F_TICKET + 1024;
constexpr u32 F_AMAX = 20480;
constexpr u32 F_YRDY = 28672;
constexpr u32 F_LINK = 32768;
constexpr u32 OFF_XS = 1u << 20;
constexpr u32 OFF_WS = OFF_XS + 6 * 16 * CH;
constexpr u32 OFF_YOUT = OFF_WS + 6 * 16 * CH;
constexpr u32 OFF_YIN = OFF_YOUT + 192 * YT;
constexpr u32 OFF_REL = OFF_YIN + 192 * YT;
constexpr u32 WS_USED = OFF_REL + 128 * YT;
constexpr u32 OOB = 0x80000000u;
constexpr int SYS = 17;

constexpr u32 STAGE = 32768;
constexpr u32 L_CTRL = 4 * STAGE;
constexpr u32 L_SNAP = L_CTRL + 1024;
constexpr u32 LDS_BYTES = L_SNAP + 1024;
constexpr u64 DEADLINE = 50000000ull;
constexpr u32 NPUSH = 7;
constexpr u32 NGEMM = 32 - NPUSH;
constexpr u32 NLD = 3;

constexpr u32 PULL_FROM = 14;

enum : u32 { E_X = 0, E_W = 1, E_Y = 2, E_FW = 3, E_R1 = 4 };
struct Params {
  const char* x;
  const char* w;
  char* out;
  u32 rank, relay;
  char* peer[NR];
  u32 nent[8];
  u32 trig[8];
  u32 pad2[8];
  u32 tab[7][48];
  u32x4 first[8][8];
  u32x4 fso[64];
  u32 pace[64];
};

__device__ __forceinline__ rsrc_t mk_rsrc(const void* p, u32 bytes) {
  return __builtin_amdgcn_make_buffer_rsrc((void*)p, 0, bytes, 0x00020000);
}
__device__ __forceinline__ u32 rfl(u32 v) { return __builtin_amdgcn_readfirstlane(v); }
__device__ __forceinline__ u32 ld_agt(const char* p) { return __hip_atomic_load((const GLBP u32*)p, __ATOMIC_RELAXED, __HIP_MEMORY_SCOPE_AGENT); }
__device__ __forceinline__ void st_sys(char* p, u32 v) { __hip_atomic_store((GLBP u32*)p, v, __ATOMIC_RELAXED, __HIP_MEMORY_SCOPE_SYSTEM); }
__device__ __forceinline__ void st_agt(char* p, u32 v) { __hip_atomic_store((GLBP u32*)p, v, __ATOMIC_RELAXED, __HIP_MEMORY_SCOPE_AGENT); }
__device__ __forceinline__ u32 poll32(rsrc_t r, u32 off) { asm volatile("" ::: "memory"); return __builtin_amdgcn_raw_buffer_load_b32(r, off, 0, SYS); }
__device__ __forceinline__ u32x4 poll128(rsrc_t r, u32 off) { asm volatile("" ::: "memory"); return __builtin_amdgcn_raw_buffer_load_b128(r, off, 0, SYS); }
#define WAIT_VM(n) asm volatile("s_waitcnt vmcnt(" #n ")" ::: "memory")
#define WAIT_LGKM0() asm volatile("s_waitcnt lgkmcnt(0)" ::: "memory")
#define BAR() __builtin_amdgcn_s_barrier()
#define SCHED() __builtin_amdgcn_sched_barrier(0)

__host__ __device__ __forceinline__ u32 row_base(u32 q, u32 r) {
  if (r < 4) return 512u * (q ^ 1u) + 128u * r;
  if (r < 12) return 512u * (q ^ (4u + (r & 3u))) + 256u * ((q >> 1) & 1u) + 128u * ((r - 4u) >> 2);
  return 512u * q + 128u * (r - 12u);
}
__host__ __device__ __forceinline__ u32 k_half(u32 q, u32 p) { return (q ^ ((q >> 1) & p)) & 1u; }

__device__ __forceinline__ u32 lds_rd32(u32 addr) {
  u32 v;
  asm volatile("ds_read_b32 %0, %1\n\ts_waitcnt lgkmcnt(0)" : "=&v"(v) : "v"(addr) : "memory");
  return rfl(v);
}
__device__ __forceinline__ void lds_wr32(u32 addr, u32 v) {
  asm volatile("ds_write_b32 %0, %1\n\ts_waitcnt lgkmcnt(0)" :: "v"(addr), "v"(v) : "memory");
}
__device__ __forceinline__ u32x4 lds_rd128(u32 addr) {
  u32x4 v;
  asm volatile("ds_read_b128 %0, %1\n\ts_waitcnt lgkmcnt(0)" : "=&v"(v) : "v"(addr) : "memory");
  return v;
}
__device__ __forceinline__ u32 seg_mask(u32x4 v, u32 call, u32 lane) {
  const u64 b = __ballot((int)((v.x - call) | (v.y - call) | (v.z - call) | (v.w - call)) >= 0);
  return (u32)__ballot(lane < 6u && ((b >> ((lane & 7u) * 8u)) & 255u) == 255u);
}

__device__ __forceinline__ u32 wave_max(u32 v) {
  v = max(v, (u32)__builtin_amdgcn_update_dpp(0, (int)v, 0x111, 0xf, 0xf, false));
  v = max(v, (u32)__builtin_amdgcn_update_dpp(0, (int)v, 0x112, 0xf, 0xf, false));
  v = max(v, (u32)__builtin_amdgcn_update_dpp(0, (int)v, 0x114, 0xf, 0xf, false));
  v = max(v, (u32)__builtin_amdgcn_update_dpp(0, (int)v, 0x118, 0xf, 0xf, false));
  v = max(v, (u32)__builtin_amdgcn_update_dpp(0, (int)v, 0x142, 0xa, 0xf, false));
  v = max(v, (u32)__builtin_amdgcn_update_dpp(0, (int)v, 0x143, 0xc, 0xf, false));
  return __builtin_amdgcn_readlane(v, 63);
}

template <int SLP>
__device__ __forceinline__ void wave_wait(rsrc_t r, u32 off, u32 n, u32 lane, u32 call, char* tmo) {
  const u64 t0 = __builtin_amdgcn_s_memrealtime();
  const u32 o = lane == 63u ? F_TMO : lane < n ? off + lane * 4u : OOB;
  for (;;) {
    const u32 v = poll32(r, o);
    if (__all((int)(v - call) >= 0 || lane >= n)) break;
    if (__builtin_amdgcn_readlane(v, 63) == call) break;
    if (__builtin_amdgcn_s_memrealtime() - t0 > DEADLINE) { st_agt(tmo, call); break; }
    __builtin_amdgcn_s_sleep(SLP);
  }
  asm volatile("" ::: "memory");
}

}

extern "C" __global__ void __launch_bounds__(512, 2) fused_kernel(Params params_by_value) {
  (void)params_by_value;
  const CSTP Params* const K = (const CSTP Params*)__builtin_amdgcn_kernarg_segment_ptr();
  const u32 grp = blockIdx.x & 7u, li = blockIdx.x >> 3;
  const char* const gx = K->x;
  const char* const gw = K->w;
  const u32x4 fso = K->fso[blockIdx.x & 63u];
  char* const far = K->peer[(li + 1u) & 7u];
  SCHED();
  extern __shared__ __attribute__((aligned(16))) char smem[];
  LDSP char* lds = (LDSP char*)smem;
  LDSP char* const sink = lds + L_CTRL + 512u;
  {
    u64 pc; u32 code_bytes;
    asm volatile("s_getpc_b64 %0\nfused_kernel_here:\n\ts_mov_b32 %1, fused_kernel_mark-fused_kernel_here" : "=&s"(pc), "=&s"(code_bytes));
    __builtin_amdgcn_raw_ptr_buffer_load_lds(mk_rsrc((const void*)pc, li == 31u ? code_bytes : 0u), sink, 4, threadIdx.x * 128u, 0, 0, 0);
    SCHED();
  }
  volatile LDSP u32* ctrl = (volatile LDSP u32*)(lds + L_CTRL);
  const u32 tid = threadIdx.x, lane = tid & 63u, wid = rfl(tid >> 6);
  const u32 vx0 = (lane >> 3) * 1024u + (((lane & 7u) ^ (lane >> 4)) << 4), vx1 = vx0 ^ 64u;
  const u32 vwp = ((lane >> 2) & 3u) * 8192u + (32u * (lane >> 4) + 8u * (lane & 3u)) * 2u;
  const bool fast = li - 1u < 6u && wid < 2u + NLD;
  const u32 twv = tid ? (li & 7u) * 16u * CH + (tid - 1u) * (OFF_WS - OFF_XS) : 0u, twb = li >= 24u ? OFF_YOUT : 0u;
  const u32 vx01 = wid ? vx1 + 8192u : vx0, head = fast && wid < 2u;
  asm volatile("" :: "v"(twv), "s"(twb), "v"(vx01), "s"(head));
  SCHED();
  asm volatile("" :: "s"(gx), "s"(gw), "s"(fso), "s"(far));
  __builtin_amdgcn_raw_ptr_buffer_load_lds(mk_rsrc(far, twb), sink, 4, twv, 0, 0, SYS);
  if (head) {
    u32x4 b0[4];
#pragma unroll
    for (u32 i = 0; i < 4; ++i) b0[i] = __builtin_amdgcn_raw_buffer_load_b128(mk_rsrc(gx, X_BYTES), vx01, fso.x + i * 16384u, SYS);
    const u32 vd = OFF_XS + (li - 1u) * 16u * CH + grp * 8192u + tid * 16u;
#pragma unroll
    for (u32 i = 0; i < 4; ++i) {
      __builtin_amdgcn_raw_buffer_store_b128(b0[i], mk_rsrc(far, WS_USED), vd + i * 2048u, 0, SYS);
      if (i < 3) __builtin_amdgcn_s_sleep(2);
    }
  }
  if (fast && wid >= 2u) {
    const u32 f = wid == 2u ? fso.y : wid == 3u ? fso.z : fso.w;
    const bool isw = f & 1u;
    const rsrc_t rs = mk_rsrc((const void*)(isw ? (u64)gw : (u64)gx), X_BYTES);
    const u32 ve = isw ? vwp : vx0, vo = isw ? vwp + 32768u : vx1 + 8192u;
    u32 so = f & ~1u;
#pragma unroll 1
    for (u32 i = 0; i < 8192u; i += 2048u, so += isw ? 65536u : 16384u) {
      __builtin_amdgcn_raw_ptr_buffer_load_lds(rs, lds + (wid - 2u) * 8192u + i, 16, ve, so, 0, SYS);
      __builtin_amdgcn_raw_ptr_buffer_load_lds(rs, lds + (wid - 2u) * 8192u + i + 1024u, 16, vo, so, 0, SYS);
    }
    WAIT_VM(0);
    BAR();
  }
  if (wid == 7u && li == 31u)
    __builtin_amdgcn_raw_ptr_buffer_load_lds(mk_rsrc((const void*)K, sizeof(Params)), sink, 4, lane * 128u, 0, 0, 0);
  if (wid == 0u && li - 24u < 7u) {
    const u32 dd = li - 23u;
    __builtin_amdgcn_s_sleep(48);
    __builtin_amdgcn_raw_ptr_buffer_load_lds(mk_rsrc(far, WS_USED), sink, 4, dd >= 4u ? (lane < 2u ? (lane ? OFF_YOUT : OFF_YIN) + (dd * 32u - 64u) * YT : OOB)
        : dd == 1u ? (lane < 4u ? ((lane & 2u) ? OFF_YOUT : OFF_YIN) + ((lane & 1u) << 20) : OOB)
        : lane < 4u ? OFF_WS + (lane + 2u) * 16u * CH : lane < 8u ? OFF_REL + ((lane - 4u) << 20) : OOB, 0, 0, SYS);
    __builtin_amdgcn_raw_ptr_buffer_load_lds(mk_rsrc(K->peer[0], WS_USED), sink, 4, lane < 4u ? (dd * 4u - 3u + lane) << 20 : OOB, 0, 0, SYS);
  }
  const u32 q = K->rank & 7u;
  char* const me = K->peer[0];
  char* const tmo = me + F_TMO;
  char* const my_ctr = me + F_CALL + blockIdx.x * 64u;
  const rsrc_t r_me = mk_rsrc(me, WS_USED);

  if (li < NPUSH) {
    const u32 d = li + 1u;
    const u32 total = min(K->nent[li], 96u) * 4u;
    const u32 nstat = d == 1u ? NLD * 8u : NLD * 8u + 8u;
    const u32 vl = lane * 16u;
    volatile LDSP u32x4* const full = (volatile LDSP u32x4*)(lds + L_CTRL);
    volatile LDSP u32* const freed = (volatile LDSP u32*)(lds + L_CTRL + 128);
    LDSP u32* const acked = (LDSP u32*)(lds + L_CTRL + 256);
    auto get_call = [&]() { u32 c; while ((c = rfl(ctrl[127])) == 0u) __builtin_amdgcn_s_sleep(1); return c; };
    if (tid - 320u < 128u) ctrl[tid - 320u] = 0;
    if (wid == 5u && d != 1u) {
#pragma unroll
      for (u32 j = 0; j < NLD; ++j) {
        const u32 p0 = j * 8u + 8u + grp;
        const u32x4 fst = K->first[li][p0 >> 2];
        if (lane == 0u) full[j] = u32x4{1u, fst.y + (p0 & 3u) * 8192u, fst.z + (p0 & 3u) * 8u, p0};
      }
    }
    WAIT_LGKM0();
    if (!fast || wid < 2u) BAR();
    if (wid == 7u) {
      if (lane == 0u) { const u32 c = ld_agt(my_ctr) + 1u; ctrl[127] = c; st_agt(my_ctr, c); }
    } else if (wid - 2u < NLD) {
      const u32 j = wid - 2u;
      const u32 dst_rank = (q ^ d) & 7u;
      const u32 h = k_half(dst_rank, q);
      u32 pend = j * 8u + grp, n = fast ? 1u : 0u;
      const u32 call = get_call();
      for (;; ++n) {
        if (n) {
          while (!__all(freed[j * 4u + (lane & 1u)] == n)) __builtin_amdgcn_s_sleep(1);
          const u32 tkv = (u32)__builtin_amdgcn_raw_ptr_buffer_atomic_add_i32(1, r_me, lane == 0u ? F_UNIT + li * 128u : OOB, 0, 0);
          pend = rfl(tkv) - (call - 1u) * (total + NLD * 8u - nstat) + nstat;
        }
        if (pend >= total) break;
        const u32 it = pend >> 2, sub = (pend & 3u) * 2u;
        const u32 ent = K->tab[li][it >> 1] >> ((it & 1u) * 16u);
        const u32 kind = (ent >> 8) & 7u, pl = ent & 255u;
        const bool in = kind <= E_W;
        u32 so, dl, fl, st = 1024u, gate = 0, ngate = 2;
        if (in) {
          const u32 c = pl >> 1, u = (pl & 1u) * 8u + sub;
          fl = F_LINK + (d * 64u + kind * 16u + c) * 128u + u * 4u;
          if (kind == E_X) { so = row_base(dst_rank, c) * 1024u + h * 512u + (u >> 2) * 128u + (u & 3u) * 32768u; dl = OFF_XS; st = 8192u; }
          else { so = (h * 256u + u * 16u) * 8192u + ((dst_rank >> 2) * 2048u + c * 128u) * 2u; dl = OFF_WS; st = 32768u; }
          dl += ((d - 2u) * 16u + c) * CH + u * 4096u;
        } else if (kind == E_FW) {
          const u32 c = pl >> 1, u = ((q >> 1) & 1u) * 8u + sub;
          const u32 dq = q ^ (((q >> 2) ^ 1u) * 4u + (pl & 1u) * 2u + (d & 1u)), dr = dq ^ d;
          so = OFF_WS + ((dq - 2u) * 16u + c) * CH + u * 4096u; gate = F_LINK + (dq * 64u + 16u + c) * 128u + u * 4u;
          dl = OFF_WS + ((dr - 2u) * 16u + c) * CH + u * 4096u; fl = F_LINK + (dr * 64u + 16u + c) * 128u + u * 4u;
        } else {
          so = OFF_YOUT + pl * YT + sub * 4096u; gate = F_YRDY + pl * 16u + sub * 2u; ngate = 1;
          dl = (kind == E_R1 ? OFF_REL + (d & 1u) * 64u * YT : OFF_YIN) + pl * YT + sub * 4096u;
          fl = F_LINK + (pl < 64u ? 64u + pl : d * 64u + 32u + (pl & 31u)) * 128u + sub * 4u + (kind == E_R1 ? 1u : 0u);
        }
        if (!in) wave_wait<2>(r_me, gate, ngate, lane, call, tmo);
        const rsrc_t rs = mk_rsrc((const void*)(!in ? (u64)me : kind == E_X ? (u64)gx : (u64)gw), in ? X_BYTES : WS_USED);
        const u32 ve = !in ? vl : kind == E_X ? vx0 : vwp, vo = !in ? vl : kind == E_X ? vx1 : vwp;
#pragma unroll
        for (u32 i = 0; i < 8; ++i) __builtin_amdgcn_raw_ptr_buffer_load_lds(rs, lds + j * 8192u + i * 1024u, 16, (i & 1u) ? vo : ve, so + i * st, 0, SYS);
        WAIT_VM(0);
        if (lane == 0u) full[j] = u32x4{n + 1u, dl, fl, pend};
      }
      asm volatile("fused_kernel_mark:");
      if (lane == 0u) full[j] = u32x4{n + 1u, OOB, 0u, ~0u};
    } else if (wid < 2u) {
      const rsrc_t r_dst = mk_rsrc(far, WS_USED);
      const u64 afar = (u64)far, amate = (u64)K->peer[1];
      const u32 js = (lane & 7u) % NLD;
      const u32 vf = lane < 2u ? lane * 4u : OOB;
      u32 cnt = 0, call = 0, prev = fast ? F_LINK + d * 64u * 128u + grp * 8u : 0u, prevc = NLD * 4u;
      auto flag = [&]() {
        u32 old = 0;
        if (prev && !call) call = get_call();
        if (prev && lane == 0u) old = __hip_atomic_fetch_add(acked + prevc, 1u, __ATOMIC_RELAXED, __HIP_MEMORY_SCOPE_WORKGROUP);
        const u64 af = afar, am = amate;
        __builtin_amdgcn_raw_buffer_store_b32(call, mk_rsrc((const void*)((prev & 1u) ? am : af), WS_USED),
                                              vf + (prev && (rfl(old) & 1u) ? prev & ~1u : OOB >> 1), 0, SYS);
        prev = 0;
      };
      for (;;) {
        const u32x4 e = full[js];
        const u32 key = e.x != cnt ? e.w : ~0u;
        u32 j = 0, bk = __builtin_amdgcn_readlane(key, 0);
#pragma unroll
        for (u32 i = 1; i < NLD; ++i) { const u32 k2 = __builtin_amdgcn_readlane(key, i); if (k2 < bk) { bk = k2; j = i; } }
        if (bk == ~0u) {
          if (prev) { WAIT_VM(0); flag(); }
          if (__all(e.x != cnt)) break;
          __builtin_amdgcn_s_sleep(1);
          continue;
        }
        u32x4 buf[4];
#pragma unroll
        for (u32 i = 0; i < 4; ++i) buf[i] = *(volatile LDSP u32x4*)(lds + j * 8192u + wid * 1024u + i * 2048u + vl);
        const u32 dl = __builtin_amdgcn_readlane(e.y, j) + wid * 1024u, fl = __builtin_amdgcn_readlane(e.z, j);
        const u32 gen = __builtin_amdgcn_readlane(e.x, j);
        if (js == j) ++cnt;
        WAIT_LGKM0();
        if (lane == 0u) freed[j * 4u + wid] = gen;
#define ST_(i, gap) do { __builtin_amdgcn_raw_buffer_store_b128(buf[i], r_dst, vl + dl + (i) * 2048u, 0, SYS); __builtin_amdgcn_s_sleep(gap); } while (0)
        WAIT_VM(3); ST_(0, 4); WAIT_VM(3); ST_(1, 4); WAIT_VM(2); ST_(2, 2); flag(); WAIT_VM(3); ST_(3, 1);
#undef ST_
        prev = fl; prevc = j * 4u + (gen & 3u);
      }
    }
    return;
  }

  const u32 kidx = li - NPUSH;
  const u32 ga = grp >> 1, gb = grp & 1u;
  const u32 lw = wid & 3u;
  u32 call = 0;
  if (wid == 4u) {
    call = rfl(ld_agt(my_ctr)) + 1u;
    if (lane == 0) ctrl[16] = call;
    WAIT_LGKM0();
  }

  u32 rt = (ga + 4u * (kidx >> 3)) & 15u, ct = 2u * (kidx & 7u) + gb, kind = rt < 4u ? 0u : 1u, fin = 0;
  for (int round = kidx < 24u ? 0 : 1; round < 2; ++round) {
    if (round == 1) {
      if (wid == 4u) {
        u32 t = 0;
        if (lane == 0) t = __hip_atomic_fetch_add((GLBP u32*)(me + F_TICKET + grp * 128u), 1u, __ATOMIC_RELAXED, __HIP_MEMORY_SCOPE_AGENT);
        t = rfl(t) - (call - 1u) * NGEMM;
        if (lane == 0) { ctrl[0] = t; st_agt(my_ctr, call); }
        if (lane >= 1u && lane < 16u) ctrl[lane] = 0;
        WAIT_LGKM0();
      }
      BAR();
      const u32 t = rfl(ctrl[0]);
      if (t >= 16u) return;
      fin = grp * 16u + t;
      kind = t < 8u ? 2u : 3u; rt = 12u + ga; ct = 2u * (t & 7u) + gb;
    }
    const u32 gcol = ((q >> 2) ^ (kind == 3u ? 1u : 0u)) * 2048u + ct * 128u;

    if (wid >= 4u) {
      const u32 remote = kind == 3u ? 0u : kind == 1u ? 63u & ~(4u << (rt & 3u)) : 63u;
      const u32 pvo = ((remote >> (lane >> 3)) & 1u) ? F_LINK + (((lane >> 3) + 2u) * 64u + ((lane & 4u) ? 16u + ct : rt)) * 128u + (lane & 3u) * 16u : OOB;
      const u32 lbase = (u32)(size_t)lds + L_CTRL;
      const u32 vs = lw * 1024u + lane * 16u;
      if (wid == 4u) {
        const u32 pace = (K->pace[rt * 4u + (ct >> 2)] >> ((ct & 3u) * 8u)) & 63u;
        if (remote && pace < 32u)
          wave_wait<16>(r_me, F_LINK + (2u * 64u + pace) * 128u, 1, lane, call, tmo);
        lds_wr32(lbase + 96u, 0);
        lds_wr32(lbase + 104u, 0);
      }
      BAR();
      u32 rem = remote, rdy = 0, snap = 0, sx = 0, sw = 0;
      auto pick = [&]() -> bool {
        const u32 av = rdy & rem;
        if (!av) return false;
        const u32 n = (u32)__builtin_ctz(av);
        rem &= ~(1u << n);
        sx = OFF_XS + (n * 16u + rt) * CH; sw = OFF_WS + (n * 16u + ct) * CH;
        return true;
      };
      auto choose = [&]() -> bool {
        if (!rem) return false;
        rdy |= lds_rd32(lbase + 96u);
        return pick();
      };
      auto issue = [&](u32 slot) {
        u32 dofs = slot * STAGE + lw * 1024u;
        asm volatile("" : "+s"(dofs));
        LDSP char* dst = lds + dofs;
#pragma unroll
        for (u32 i = 0; i < 4; ++i) {
          __builtin_amdgcn_raw_ptr_buffer_load_lds(r_me, dst + i * 4096u, 16, vs, sx + i * 4096u, 0, SYS);
          __builtin_amdgcn_raw_ptr_buffer_load_lds(r_me, dst + 16384u + i * 4096u, 16, vs, sw + i * 4096u, 0, SYS);
        }
        sx += 16384u; sw += 16384u;
      };
      auto shoot = [&]() {
        if (wid == 4u && rem && !(rem & rdy)) { __builtin_amdgcn_raw_ptr_buffer_load_lds(r_me, lds + L_SNAP, 16, pvo, 0, 0, SYS); snap = 1; }
      };
      auto publish = [&]() {
        if (snap) { lds_wr32(lbase + 96u, seg_mask(lds_rd128(lbase + 1024u + lane * 16u), call, lane)); snap = 0; }
      };
      {
        const rsrc_t r_x = mk_rsrc(gx, X_BYTES), r_w = mk_rsrc(gw, W_BYTES);
        const u32 vx = lw * 8192u + ((lw & 1u) ? vx1 : vx0), vw = lw * 32768u + vwp;
        const u32 grow = row_base(q, rt);
#pragma unroll 1
        for (u32 s8 = 0; s8 < 8; ++s8) {
          if (s8 == 4u) { WAIT_VM(24); BAR(); }
          if (s8 >= 4u) { WAIT_VM(16); BAR(); }
          if (s8 == 5u) shoot();
          LDSP char* dst = lds + (s8 & 3u) * STAGE + lw * 1024u;
          const u32 ox = grow * 1024u + s8 * 128u, ow = s8 * 64u * 8192u + gcol * 2u;
#pragma unroll
          for (u32 i = 0; i < 4; ++i) {
            __builtin_amdgcn_raw_ptr_buffer_load_lds(r_x, dst + i * 4096u, 16, vx, ox + i * 32768u, 0, 0);
            __builtin_amdgcn_raw_ptr_buffer_load_lds(r_w, dst + 16384u + i * 4096u, 16, vw, ow + i * 131072u, 0, 0);
          }
        }
      }
      for (;;) {
        for (;;) {
          const bool pre = pick();
          WAIT_VM(16); publish(); BAR();
          if (pre || choose()) {
            issue(0); WAIT_VM(16); BAR();
            shoot(); issue(1); WAIT_VM(16); BAR();
          } else {
            shoot(); WAIT_VM(8); BAR();
            WAIT_VM(0); publish(); BAR();
            if (!choose()) break;
            shoot();
#pragma unroll 1
            for (u32 i = 0; i < 2; ++i) issue(i);
          }
          issue(2); WAIT_VM(16); BAR();
          issue(3);
        }
        if (wid == 4u) lds_wr32(lbase + 104u, rem ? 1u : 2u);
        BAR();
        if (!rem) break;
        if (wid == 4u) {
          const u64 t0 = __builtin_amdgcn_s_memrealtime();
          u32 sm;
          for (;;) {
            const u32x4 v = poll128(r_me, lane == 63u ? F_TMO : pvo);
            sm = seg_mask(v, call, lane);
            if (sm & rem) break;
            sm = 63u;
            if (__builtin_amdgcn_readlane(v.x, 63) == call) break;
            if (__builtin_amdgcn_s_memrealtime() - t0 > DEADLINE) { st_agt(tmo, call); break; }
            __builtin_amdgcn_s_sleep(2);
          }
          lds_wr32(lbase + 96u, sm);
        }
        BAR();
        if (wid == 4u) lds_wr32(lbase + 104u, 0);
        choose();
#pragma unroll 1
        for (u32 i = 0; i < 4; ++i) issue(i);
        WAIT_VM(24); BAR();
      }
      if (kind >= 2u) {
        const u32 callw = rfl(ctrl[16]);
        const u64 t0 = __builtin_amdgcn_s_memrealtime();
        volatile LDSP u32x2* const res = (volatile LDSP u32x2*)(lds + L_CTRL + 32u + lw * 8u);
        u32 o0 = F_AMAX + (lw * 64u + lane) * 16u, o1 = o0 + 4096u, m = 0, bad = 0, seen = 0;
        for (;;) {
          const u32x4 g0 = poll128(r_me, o0), g1 = poll128(r_me, o1);
          const u32 tv = poll32(r_me, lane == 0u ? F_TMO : OOB);
          if (g0.y == callw && g0.w == callw) { m = max(m, max(g0.x, g0.z)); o0 = OOB; }
          if (g1.y == callw && g1.w == callw) { m = max(m, max(g1.x, g1.z)); o1 = OOB; }
          if (rfl(tv) == callw) { bad = 1; break; }
          const u32 missing = (u32)__builtin_popcountll(__ballot((o0 & o1) != OOB));
          if (missing == 0u) break;
          if (missing <= 36u && missing != seen) {
            const u32 pm = wave_max(m);
            if (lane == 0) *res = u32x2{pm, 1u};
            seen = missing;
          }
          if (__builtin_amdgcn_s_memrealtime() - t0 > DEADLINE) { st_agt(tmo, callw); bad = 1; break; }
          if (missing > 24u) __builtin_amdgcn_s_sleep(40);
          else if (missing > 8u) __builtin_amdgcn_s_sleep(10);
          else __builtin_amdgcn_s_sleep(1);
        }
        m = wave_max(m);
        if (lane == 0) *res = u32x2{m, 2u + bad};
      }
    } else {
      const u32 wn = wid & 1u, wm = wid >> 1;
      const u32 hh = lane >> 5, r31 = lane & 31u;
      const u32 z = (lane >> 4) & 1u, tq = (lane >> 2) & 3u, tp = lane & 3u;
      const u32 aw = 16384u + hh * 2048u + wn * 512u + tq * 64u + (16u * (tp & 1u) + 8u * z + 4u * (tp >> 1)) * 2u;
      const u32 axb = (wm * 64u + r31) * 128u, sw7 = (r31 >> 1) & 7u;
      u32 ax[4];
#pragma unroll
      for (u32 s = 0; s < 4; ++s) ax[s] = axb + (((2u * s + hh) ^ sw7) << 4);
      f32x16 acc[2][2] = {};
      s16x8 fa[4][2] = {}, fb[4][2] = {};
      s16x4 alo = {};
      auto tr = [&](u32 off) -> s16x4 { return __builtin_amdgcn_ds_read_tr16_b64_v4i16((LDSP s16x4*)(lds + off)); };
      auto rd0 = [&](u32 so, int ks) {
        const s16x4 lo = tr(so + aw + ks * 4096), hi = tr(so + aw + ks * 4096 + 1024);
        fa[ks][0] = __builtin_shufflevector(lo, hi, 0, 1, 2, 3, 4, 5, 6, 7);
        fb[ks][0] = *(LDSP s16x8*)(lds + so + ax[ks]);
      };
      auto rd1 = [&](u32 so, int ks) { fb[ks][1] = *(LDSP s16x8*)(lds + so + ax[ks] + 4096); alo = tr(so + aw + ks * 4096 + 256); };
      auto rd2 = [&](u32 so, int ks) { fa[ks][1] = __builtin_shufflevector(alo, tr(so + aw + ks * 4096 + 256 + 1024), 0, 1, 2, 3, 4, 5, 6, 7); };
      auto mm = [&](int ks, int fn, int fm) {
        acc[fn][fm] = __builtin_amdgcn_mfma_f32_32x32x16_bf16(__builtin_bit_cast(bf16x8, fa[ks][fn]), __builtin_bit_cast(bf16x8, fb[ks][fm]), acc[fn][fm], 0, 0, 0);
        SCHED();
      };
      u32 statv = 0;
      auto kstep = [&](int kc, u32 so, int kn, int sync) {
        mm(kc, 0, 0);
        if (sync) {
          mm(kc, 0, 1);
          WAIT_LGKM0(); BAR(); asm volatile("" ::: "memory");
          if (sync == 2) statv = ctrl[26];
        }
        rd0(so, kn); SCHED();
        if (!sync) { mm(kc, 0, 1); rd1(so, kn); SCHED(); }
        mm(kc, 1, 0);
        if (sync) rd1(so, kn);
        rd2(so, kn); SCHED();
        mm(kc, 1, 1);
      };
      BAR();
      __builtin_amdgcn_s_waitcnt(0x0F70);
      for (;;) {
        BAR();
        asm volatile("" ::: "memory");
#pragma unroll
        for (int k = 0; k < 2; ++k) { rd0(0, k); rd1(0, k); rd2(0, k); SCHED(); }
        u32 status;
        do {
#pragma unroll
          for (int st = 0; st < 4; ++st)
#pragma unroll
            for (int k = 0; k < 4; ++k)
              kstep(k, (k < 2 ? st : (st + 1) & 3) * STAGE, (k + 2) & 3, k == 2 ? (st == 3 ? 2 : 1) : 0);
          status = rfl(statv);
        } while (status == 0u);
        if (status == 2u) break;
        BAR();
      }
      const u32 vy = wid * 8192u + lane * 16u;
      const u32 callc = rfl(ctrl[16]);
      if (kind < 2u) {
        const u32 ti = kind == 0u ? rt * 16u + ct : 64u + (rt & 3u) * 32u + ((rt - 4u) >> 2) * 16u + ct;
#pragma unroll
        for (int f = 0; f < 4; ++f)
#pragma unroll
          for (int hf = 0; hf < 2; ++hf) {
            u32x4 v;
#pragma unroll
            for (int e = 0; e < 4; ++e) {
              const f32x2 p = {acc[f >> 1][f & 1][hf * 8 + e * 2], acc[f >> 1][f & 1][hf * 8 + e * 2 + 1]};
              v[e] = __builtin_bit_cast(u32, __builtin_convertvector(p, bf16x2));
            }
            __builtin_amdgcn_raw_buffer_store_b128(v, r_me, vy + OFF_YOUT + ti * YT + (f * 2 + hf) * 1024, 0, SYS);
          }
        WAIT_VM(0);
        if (ct >= PULL_FROM) {
          const u32 dd = kind == 0u ? 1u : 4u + (rt & 3u);
          if (lane < 2u) st_sys(K->peer[dd] + F_LINK + (kind == 0u ? 64u + ti : dd * 64u + 32u + (ti & 31u)) * 128u + wid * 8u + lane * 4u, callc);
        } else if (lane == 0u) st_sys(me + F_YRDY + ti * 16u + wid * 4u, callc);
      } else {
        char* const gdst = K->peer[lane & 7u] + F_AMAX + (q * 128u + fin) * 8u;
        const u32 d0 = (q ^ (((q >> 2) ^ 1u) * 4u + (ga >> 1) * 2u)) & 7u, d1 = d0 ^ 1u;
        const u32 ty = (ga & 1u) * 16u + ct;
        const u32 so0 = OFF_YIN + (kind == 2u ? ga * 16u + ct : d0 * 32u - 64u + ty) * YT, so1 = OFF_YIN + (d1 * 32u - 64u + ty) * YT;
        const u32 fo0 = F_LINK + (kind == 2u ? 64u + ga * 16u + ct : d0 * 64u + 32u + ty) * 128u, fo1 = F_LINK + (d1 * 64u + 32u + ty) * 128u;
        const bool parked = kind == 2u && ((K->relay >> ct) & 1u);
        const bool pull = parked || ct >= PULL_FROM;
        const u32 tg0 = parked ? fo0 : K->trig[kind == 2u ? 1u : d0], tg1 = K->trig[d1];
        const u32 po = lane == 63u ? F_TMO : lane < 16u ? ((lane & 8u) ? fo1 : fo0) + (lane & 7u) * 4u
                     : pull && lane < 24u ? ((lane & 4u) ? tg1 : tg0) + (lane & 3u) * 8u : OOB;
        const rsrc_t r_out = mk_rsrc(K->out, OUT_BYTES);
        const u32 vo = (ga * 128u + wm * 64u + (lane >> 2)) * 4096u + gcol + wn * 64u + (lane & 3u) * 16u;
        const u64 t0 = __builtin_amdgcn_s_memrealtime();
        auto poll = [&](u32 todo, u32 pv) {
          u32 go;
          for (;; pv = poll32(r_me, po)) {
            const u32 okm = (u32)__ballot((int)(pv - callc) >= 0), own = okm >> (wid * 2u);
            go = (((own & 3u) == 3u ? 1u : 0u) | ((own & 0x300u) == 0x300u ? 2u : 0u)) & todo;
            if (pull && (go & (((okm & 0x0F0000u) ? 1u : 0u) | ((okm & 0xF00000u) ? 2u : 0u))) != todo) go = 0;
            if (go) break;
            go = todo;
            if (__builtin_amdgcn_readlane(pv, 63) == callc) break;
            if (__builtin_amdgcn_s_memrealtime() - t0 > DEADLINE) { st_agt(tmo, callc); break; }
            if (okm & (((todo & 1u) ? 0xFFu : 0u) | ((todo & 2u) ? 0xFF00u : 0u))) __builtin_amdgcn_s_sleep(1);
            else __builtin_amdgcn_s_sleep(12);
          }
          asm volatile("" ::: "memory");
          return go;
        };
        auto fetch = [&](u32x4 (&v)[8], u32 so) {
#pragma unroll
          for (int j = 0; j < 8; ++j) v[j] = __builtin_amdgcn_raw_buffer_load_b128(r_me, vy + j * 1024, so, SYS);
        };
        auto add = [&](const u32x4 (&v)[8]) {
#pragma unroll
          for (int j = 0; j < 8; ++j)
#pragma unroll
            for (int e = 0; e < 4; ++e) {
              acc[j >> 2][(j >> 1) & 1][(j & 1) * 8 + e * 2] += __builtin_bit_cast(float, v[j][e] << 16);
              acc[j >> 2][(j >> 1) & 1][(j & 1) * 8 + e * 2 + 1] += __builtin_bit_cast(float, v[j][e] & 0xFFFF0000u);
            }
        };
        u32x4 va[8], vb[8];
        const u32 go = poll(kind == 2u ? 1u : 3u, poll32(r_me, po));
        if (pull) {
          const rsrc_t ra = mk_rsrc(K->peer[parked ? 3u - (ga & 1u) : kind == 2u ? 1u : d0], WS_USED);
          const rsrc_t rb = mk_rsrc(K->peer[d1], kind == 3u ? WS_USED : 0u);
          const u32 sa = parked ? OFF_REL + ((ga & 1u) * 64u + ga * 16u + ct) * YT : so0 - (OFF_YIN - OFF_YOUT);
#pragma unroll
          for (int j = 0; j < 8; ++j) {
            if (j >= 2 && kind == 2u) WAIT_VM(2);
            if (j >= 3) WAIT_VM(4);
            va[j] = __builtin_amdgcn_raw_buffer_load_b128(ra, vy + j * 1024, sa, SYS);
            vb[j] = __builtin_amdgcn_raw_buffer_load_b128(rb, vy + j * 1024, so1 - (OFF_YIN - OFF_YOUT), SYS);
            if (j < 7) __builtin_amdgcn_s_sleep(5);
          }
        } else {
          fetch(va, (go & 1u) ? so0 : so1);
          fetch(vb, go == 3u ? so1 : OOB);
        }
        const u32 pv2 = poll32(r_me, po);
        add(va);
        if (kind == 3u && go != 3u) { poll(go ^ 3u, pv2); fetch(vb, (go & 1u) ? so1 : so0); }
        if (kind == 3u) add(vb);
        float mx = 0.f;
#pragma unroll
        for (int f = 0; f < 4; ++f)
#pragma unroll
          for (int e = 0; e < 16; ++e) mx = fmaxf(mx, acc[f >> 1][f & 1][e]);
        const u32 wmax = wave_max(__builtin_bit_cast(u32, mx) & 0x7FFFFFFFu);
        u32 old = 0;
        if (lane == 0) {
          ctrl[4 + wid] = wmax;
          old = __hip_atomic_fetch_add((LDSP u32*)(lds + L_CTRL) + 1, 1u, __ATOMIC_RELAXED, __HIP_MEMORY_SCOPE_WORKGROUP);
        }
        if (rfl(old) == 3u) {
          const u32x4 m4 = *(volatile LDSP u32x4*)(lds + L_CTRL + 16);
          const u32 m = max(max(m4.x, m4.y), max(m4.z, m4.w));
          if (lane < 8u) __hip_atomic_store((GLBP u64*)gdst, ((u64)callc << 32) | m, __ATOMIC_RELAXED, __HIP_MEMORY_SCOPE_SYSTEM);
        }
        u32 used = ~0u;
        for (;;) {
          const u32x4 a4 = *(volatile LDSP u32x4*)(lds + L_CTRL + 32), b4 = *(volatile LDSP u32x4*)(lds + L_CTRL + 48);
          const u32 lo = rfl(min(min(a4.y, a4.w), min(b4.y, b4.w))), hi = rfl(max(max(a4.y, a4.w), max(b4.y, b4.w)));
          if (lo == 0u) { __builtin_amdgcn_s_sleep(2); continue; }
          const u32 am = rfl(max(max(max(a4.x, a4.z), max(b4.x, b4.z)), wmax));
          const bool bad = hi == 3u;
          if (am != used || bad) {
            if (used != ~0u) WAIT_VM(0);
            used = am;
            const float amax = __builtin_bit_cast(float, am);
            const float scale = amax / 127.0f;
            const float inv = amax > 0.f ? 1.0f / scale : 0.f;
#pragma unroll
            for (int f = 0; f < 4; ++f) {
              u32x4 v;
#pragma unroll
              for (int e = 0; e < 4; ++e) {
                u32 pk = 0;
#pragma unroll
                for (int b = 0; b < 4; ++b) {
                  const float y = __builtin_amdgcn_fmed3f(__builtin_rintf(acc[f >> 1][f & 1][e * 4 + b] * inv), 0.f, 127.f);
                  pk = __builtin_amdgcn_cvt_pk_u8_f32(y, b, pk);
                }
                v[e] = bad ? 0x80808080u : pk;
              }
              *(LDSP u32x4*)(lds + wid * 4096u + ((f & 1) * 32u + r31) * 64u + ((((f >> 1) * 2u + hh) ^ ((r31 >> 1) & 3u)) << 4)) = v;
            }
#pragma unroll
            for (u32 i = 0; i < 4; ++i) {
              const u32x4 v = *(LDSP u32x4*)(lds + wid * 4096u + (i * 16u + (lane >> 2)) * 64u + (((lane & 3u) ^ ((lane >> 3) & 3u)) << 4));
              __builtin_amdgcn_raw_buffer_store_b128(v, r_out, vo + i * 16u * 4096u, 0, 16);
            }
          }
          if (lo >= 2u) break;
          __builtin_amdgcn_s_sleep(1);
        }
      }
    }
  }
}

namespace {
const char* const kCross = "T0 w0 T1 w1 T2 w2 T3 w3 w4 X0 w5 X1 w6 X2 X3 w7 X4 X5 w8 X6 X7 w11 w9 w12 w14 w10 w15 w13 O0 O1 O2 O3";
const char* const kSib = "T0 w0 T1 w1 T2 w2 T3 w3 w4 X0 w5 X1 w6 X2 w7 X3 X4 w8 X5 X6 X7 w9 w10 f11 f12 w11 f14 f15 w12 w13 w14 w15 "
                         "O0 O1 O2 O3 a12 a13";

int parse_order(const char* p, int* v) {
  int n = 0;
  while (*p && n < 48) {
    const char c = *p++;
    int k = 0;
    while (*p >= '0' && *p <= '9') k = k * 10 + (*p++ - '0');
    while (*p == ' ') ++p;
    v[n++] = (c == 'T' ? 0 : c == 'X' ? 4 : c == 'O' ? 12 : c == 'w' ? 16 : c == 'f' ? 32 : 48) + k;
  }
  return n;
}

void build_tables(Params& P, int rank) {
  int cs[48], ss[48];
  const int nc = parse_order(kCross, cs), ns = parse_order(kSib, ss);
  u32 split = 0, relay = 0;
  for (int i = 0; i < ns; ++i) { if ((ss[i] >> 4) == 2) split |= 1u << (ss[i] & 15); if ((ss[i] >> 4) == 3) relay |= 1u << (ss[i] & 15); }
  const double C = 1.125, LEAD = 10.0;
  double ct[4][32] = {}, sb[48] = {}, done[16][16];
  for (int l = 0; l < 4; ++l) {
    double t = 0;
    for (int i = 0; i < nc; ++i) {
      const int it = cs[i];
      if (it >= 4 && it < 12 && (it & 3) == l) continue;
      ct[l][it] = t += (it >= 16 && ((split >> (it - 16)) & 1u)) ? C / 2 : C;
    }
  }
  double t = 0;
  for (int i = 0; i < ns; ++i) {
    const int it = ss[i];
    if ((it >> 4) == 2) for (int l = 0; l < 4; ++l) t = std::max(t, ct[l][it - 16] + 2.2);
    t += C;
    if (it < 48) sb[it] = t;
  }
  unsigned char pace[16][16];
  for (int r = 0; r < 16; ++r)
    for (int c = 0; c < 16; ++c) {
      double sg[6]; int n = 0;
      sg[n++] = std::max(sb[r], sb[16 + c]); sg[n++] = sg[0];
      for (int l = 0; l < 4; ++l)
        if (!(r >= 4 && r < 12 && (r & 3) == l)) sg[n++] = std::max(std::max(ct[l][r], ct[l][16 + c]), sb[32 + c]);
      std::sort(sg, sg + n);
      double e = sg[1];
      for (int i = 0; i < n; ++i) e = std::max(e, sg[i] + (i >= 2 ? 0.9 : 0.0)) + 1.25;
      done[r][c] = e + 1e-3 * (c * 16 + r);
      pace[r][c] = 32;
      for (int i = 0; i < ns; ++i) if (ss[i] < 32 && sb[ss[i]] - C <= sg[1] - LEAD) pace[r][c] = (unsigned char)ss[i];
    }
  memcpy(P.pace, pace, sizeof pace);
  P.relay = relay;
  unsigned short tab[7][96] = {};
  for (int d = 1; d < 8; ++d) {
    int m = 0;
    auto put = [&](u32 kind, int pl) { if (m < 96) tab[d - 1][m++] = (unsigned short)(kind << 8 | (u32)pl); };
    const int tdst = ((rank ^ d) >> 1) & 1;
    for (int i = 0; i < (d >= 4 ? nc : d >= 2 ? ns : 0); ++i) {
      const int it = d >= 4 ? cs[i] : ss[i], k = it & 15;
      if (it < 32) {
        if (d >= 4 && it >= 4 && it < 12 && (it & 3) == d - 4) continue;
        for (int hf = 0; hf < 2; ++hf)
          if (!(d >= 4 && it >= 16 && ((split >> k) & 1u)) || hf == tdst) put(it < 16 ? E_X : E_W, k * 2 + hf);
      } else if (it < 48) { put(E_FW, k * 2); put(E_FW, k * 2 + 1); }
      else for (int r = d & 1; r < 4; r += 2) put(E_R1, r * 16 + k);
    }
    const int nt = d == 1 ? 64 : d >= 4 ? 32 : 0;
    int ord[64], no = 0;
    auto key = [&](int tl) { return done[d == 1 ? tl >> 4 : d + 4 * (tl >> 4)][tl & 15]; };
    for (int tl = 0; tl < nt; ++tl)
      if ((u32)(tl & 15) < PULL_FROM && (d != 1 || !((relay >> (tl & 15)) & 1u))) ord[no++] = tl;
    std::sort(ord, ord + no, [&](int x, int y) { return key(x) < key(y); });
    for (int side = 1; side >= 0 && no; --side) {
      const int e = (rank ^ (side ? d : 0)) & 1;
      if (d >= 4)
        std::sort(ord + no - 4, ord + no, [&](int x, int y) { return key(x) + 1e3 * ((x >> 4) ^ e) < key(y) + 1e3 * ((y >> 4) ^ e); });
      if (side) P.trig[d] = F_LINK + (u32)(d == 1 ? 64 + ord[no - 4] : d * 64 + 32 + ord[no - 4]) * 128u;
    }
    for (int i = 0; i < no; ++i) put(E_Y, d == 1 ? ord[i] : d * 32 - 64 + ord[i]);
    P.nent[d - 1] = (u32)m;
    for (int i = 0; i < 8 && d >= 2; ++i) {
      const u32 kind = tab[d - 1][i] >> 8, c = (tab[d - 1][i] & 255u) >> 1, u = (tab[d - 1][i] & 1u) * 8u;
      const u32 dst = (u32)(rank ^ d), h = k_half(dst, (u32)rank);
      P.first[d - 1][i] = u32x4{kind == E_X ? row_base(dst, c) * 1024u + h * 512u + (u >> 2) * 128u : (h * 256u + u * 16u) * 8192u + ((dst >> 2) * 2048u + c * 128u) * 2u,
                                (kind == E_X ? OFF_XS : OFF_WS) + ((d - 2u) * 16u + c) * CH + u * 4096u,
                                F_LINK + (d * 64u + kind * 16u + c) * 128u + u * 4u, kind};
    }
    for (u32 p0 = 0; p0 < 32; ++p0) {
      const u32 sub = (p0 & 3u) * 2u;
      const u32x4 e = P.first[d - 1][p0 >> 2];
      P.fso[(d - 1) * 8 + (p0 & 7u)][p0 >> 3] = (e.x + (e.w ? sub * 131072u : (sub >> 2) * 128u + (sub & 3u) * 32768u)) | e.w;
    }
  }
  memcpy(P.tab, tab, sizeof tab);
}
}

extern "C" void kernel_launch(int rank, int nranks, ncclComm_t comm, hipStream_t stream,
                              void* const* d_in, const int* in_sizes, int n_in,
                              void* d_out, int out_size, const int* in_dtypes, int out_dtype,
                              void* const* d_out_all, void* const* d_ws_all, size_t ws_bytes) {
  (void)comm; (void)in_dtypes; (void)out_dtype; (void)d_out_all;
  if (nranks != NR || (unsigned)rank >= (unsigned)NR || n_in < 2 || ws_bytes < WS_USED ||
      (size_t)in_sizes[0] * 2 != X_BYTES || (size_t)in_sizes[1] * 2 != W_BYTES || (u32)out_size != OUT_BYTES) {
    fprintf(stderr, "kernel_launch: unexpected problem shape\n");
    return;
  }
  Params P;
  memset(&P, 0, sizeof P);
  P.x = (const char*)d_in[0];
  P.w = (const char*)d_in[1];
  P.out = (char*)d_out;
  for (int p = 0; p < NR; ++p) P.peer[p] = (char*)d_ws_all[rank ^ p];
  P.rank = (u32)rank;
  build_tables(P, rank);
  hipError_t e = hipFuncSetAttribute((const void*)fused_kernel, hipFuncAttributeMaxDynamicSharedMemorySize, (int)LDS_BYTES);
  if (e != hipSuccess) { fprintf(stderr, "hipFuncSetAttribute: %s\n", hipGetErrorString(e)); return; }
  fused_kernel<<<dim3(256), dim3(512), LDS_BYTES, stream>>>(P);
  e = hipGetLastError();
  if (e != hipSuccess) fprintf(stderr, "launch: %s\n", hipGetErrorString(e));
}
```
